```python
import numpy as np

kernel_name = "gpu_gemm_a2a_m4096_k4096_n16384_e4m3e4m3_dequant"


ABSMAX_RTOL = 0.03

COMPUTE_DTYPE = "float32"
EPILOGUE = "dequant"
IN_QUANT = None
OUT_QUANT = None
IN_FP8 = "e4m3e4m3"
IN_DTYPES = {"x": "e4m3", "w_mat": "e4m3",
             "scale_x": "float32", "scale_w": "float32"}
WORLD_SIZE = 8

def _snap_fp8(x, mant_bits, max_abs, min_normal):
    a = np.abs(x.astype(np.float64))
    _m, _e = np.frexp(a)
    q = np.where(a >= min_normal, np.ldexp(1.0, _e - 1 - mant_bits),
                 min_normal * 2.0 ** -mant_bits)
    snapped = np.minimum(np.round(a / q) * q, max_abs)
    return (np.sign(x) * snapped).astype(np.float32)

def setup_inputs():

    x = _snap_fp8(np.random.randn(4096, 4096).astype(np.float32), 3, 448.0, 2.0 ** -6)
    w_mat = _snap_fp8(np.random.randn(4096, 16384).astype(np.float32), 3, 448.0, 2.0 ** -6)
    scale_x = (0.02 * (1.0 + np.random.rand(1))).astype(np.float32)
    scale_w = (0.02 * (1.0 + np.random.rand(1))).astype(np.float32)
    return {"x": x, "w_mat": w_mat, "scale_x": scale_x, "scale_w": scale_w}

def reference(x, w_mat, scale_x, scale_w):
    acc = (x @ w_mat).astype(np.float32)
    y = (acc * (scale_x * scale_w)).astype(np.float32)
    return y
```

```cpp
#include <hip/hip_runtime.h>
#include <rccl/rccl.h>
#include <cstdio>

#define HIP_CHECK(expr) do { hipError_t _e = (expr); if (_e != hipSuccess) { \
    fprintf(stderr, "HIP error %s at %s:%d\n", hipGetErrorString(_e), __FILE__, __LINE__); \
    return; } } while (0)

namespace {
typedef unsigned u32x4 __attribute__((ext_vector_type(4)));
typedef unsigned u32x8 __attribute__((ext_vector_type(8)));
typedef float f32x4 __attribute__((ext_vector_type(4)));
#define LAS __attribute__((address_space(3)))
#define DI __device__ __forceinline__

constexpr int NR = 8;
constexpr int KDIM = 4096, NFULL = 16384, NSH = 2048, MSH = 512;
constexpr unsigned SHARD = MSH * KDIM;
constexpr int NBLK = MSH / 16;
constexpr int NUNIT = 512;
constexpr int WPP = 16;
constexpr int NSEND_WG = WPP * 7 / 4;
constexpr size_t WS_X = 0;
constexpr size_t WS_FLAG = 2ull * NR * SHARD;
constexpr size_t WS_TICK = WS_FLAG + NR * 2048;
constexpr size_t WS_TMO = WS_TICK + 1024;
constexpr size_t WS_CALL = WS_TMO + 1024;
constexpr size_t WS_DBG = WS_CALL + 1024 * 64;
constexpr size_t WS_END = WS_DBG + 256 * 64 * 8;
constexpr unsigned long long DEADLINE_TICKS = 30000000ull;

#define MAPMODE 0
#define OUT_AUX 0

struct Params {
    const unsigned char* x; const unsigned char* w; const float* sx; const float* sw; float* out;
    char* ws[NR]; int rank; int pad;
};

#define ACLOB "a0","a1","a2","a3","a4","a5","a6","a7","a8","a9","a10","a11","a12","a13","a14","a15","a16","a17","a18","a19","a20","a21","a22","a23","a24","a25","a26","a27","a28","a29","a30","a31","a32","a33","a34","a35","a36","a37","a38","a39","a40","a41","a42","a43","a44","a45","a46","a47","a48","a49","a50","a51","a52","a53","a54","a55","a56","a57","a58","a59","a60","a61","a62","a63","a64","a65","a66","a67","a68","a69","a70","a71","a72","a73","a74","a75","a76","a77","a78","a79","a80","a81","a82","a83","a84","a85","a86","a87","a88","a89","a90","a91","a92","a93","a94","a95","a96","a97","a98","a99","a100","a101","a102","a103","a104","a105","a106","a107","a108","a109","a110","a111","a112","a113","a114","a115","a116","a117","a118","a119","a120","a121","a122","a123","a124","a125","a126","a127","a128","a129","a130","a131","a132","a133","a134","a135","a136","a137","a138","a139","a140","a141","a142","a143","a144","a145","a146","a147","a148","a149","a150","a151","a152","a153","a154","a155","a156","a157","a158","a159","a160","a161","a162","a163","a164","a165","a166","a167","a168","a169","a170","a171","a172","a173","a174","a175","a176","a177","a178","a179","a180","a181","a182","a183","a184","a185","a186","a187","a188","a189","a190","a191","a192","a193","a194","a195","a196","a197","a198","a199","a200","a201","a202","a203","a204","a205","a206","a207","a208","a209","a210","a211","a212","a213","a214","a215","a216","a217","a218","a219","a220","a221","a222","a223","a224","a225","a226","a227","a228","a229","a230","a231","a232","a233","a234","a235","a236","a237","a238","a239","a240","a241","a242","a243","a244","a245","a246","a247","a248","a249","a250","a251","a252","a253","a254","a255"

template <int N> DI void wait_vm() { asm volatile("s_waitcnt vmcnt(%0)" :: "n"(N) : "memory"); }
DI void wait_lgkm0() { asm volatile("s_waitcnt lgkmcnt(0)" ::: "memory"); }
template <int B> DI void snd_load(const unsigned char* base, unsigned voff) {
    asm volatile("global_load_dwordx4 a[%c2:%c2+3], %0, %1\n\t"
                 "global_load_dwordx4 a[%c2+4:%c2+7], %0, %1 offset:64\n\t"
                 "global_load_dwordx4 a[%c2+8:%c2+11], %0, %1 offset:128\n\t"
                 "global_load_dwordx4 a[%c2+12:%c2+15], %0, %1 offset:192" :: "v"(voff), "s"(base), "i"(B) : "memory");
}
template <int B> DI void snd_store(char* base, unsigned voff) {
    asm volatile("global_store_dwordx4 %0, a[%c2:%c2+3], %1 sc0 sc1\n\t"
                 "global_store_dwordx4 %0, a[%c2+4:%c2+7], %1 offset:1024 sc0 sc1\n\t"
                 "global_store_dwordx4 %0, a[%c2+8:%c2+11], %1 offset:2048 sc0 sc1\n\t"
                 "global_store_dwordx4 %0, a[%c2+12:%c2+15], %1 offset:3072 sc0 sc1\n\ts_nop 1" :: "v"(voff), "s"(base), "i"(B) : "memory");
}
DI void snd_ticket(char* ctr, unsigned vzero) {
    unsigned long long keep;
    asm volatile("s_mov_b64 %0, exec\n\ts_mov_b64 exec, 1\n\tglobal_atomic_add a32, %1, a33, %2 sc0\n\ts_mov_b64 exec, %0"
                 : "=&s"(keep) : "v"(vzero), "s"(ctr) : "memory");
}
DI void snd_flag(char* flag, unsigned vzero) {
    unsigned long long keep;
    asm volatile("s_mov_b64 %0, exec\n\ts_mov_b64 exec, 1\n\tglobal_store_dword %1, a34, %2 sc0 sc1\n\ts_mov_b64 exec, %0"
                 : "=&s"(keep) : "v"(vzero), "s"(flag) : "memory");
}
template <int N> DI unsigned snd_wait_ticket() {
    unsigned v;
    asm volatile("s_waitcnt vmcnt(%1)\n\tv_accvgpr_read_b32 %0, a32\n\ts_nop 1" : "=v"(v) : "n"(N) : "memory");
    return __builtin_amdgcn_readfirstlane(v);
}

#define STAMP(k) do { } while (0)

template <int S, int SLOT> DI void w_read(unsigned a0, unsigned a1, unsigned a2, unsigned a3) {
    constexpr int B = S * 32, O = SLOT * 8192;
#define TRR(c, t, A) "ds_read_b64_tr_b8 a[%c4+" #c "*8+" #t "*2:%c4+" #c "*8+" #t "*2+1], " A " offset:%c5+4096*(" #t "/2)+512*(" #t "%%2)\n\t"
    asm volatile(TRR(0, 0, "%0") TRR(0, 1, "%0") TRR(0, 2, "%0") TRR(0, 3, "%0")
                 TRR(1, 0, "%1") TRR(1, 1, "%1") TRR(1, 2, "%1") TRR(1, 3, "%1")
                 TRR(2, 0, "%2") TRR(2, 1, "%2") TRR(2, 2, "%2") TRR(2, 3, "%2")
                 TRR(3, 0, "%3") TRR(3, 1, "%3") TRR(3, 2, "%3") TRR(3, 3, "%3")
                 "s_waitcnt lgkmcnt(0)"
                 :: "v"(a0), "v"(a1), "v"(a2), "v"(a3), "i"(B), "i"(O) : "memory", ACLOB);
#undef TRR
}
DI void w_stage(__amdgpu_buffer_rsrc_t rs, unsigned ve, unsigned vo, unsigned m0base, unsigned soff) {
    unsigned keep, so;
#define WST(V) "s_nop 0\n\tbuffer_load_dwordx4 " V ", %5, %1 offen lds\n\t"
#define WNX "s_add_u32 m0, m0, 0x400\n\ts_add_u32 %1, %1, 0x40000\n\t"
    asm volatile("s_mov_b32 %0, m0\n\ts_mov_b32 m0, %2\n\ts_mov_b32 %1, %6\n\t"
                 WST("%3") WNX WST("%4") WNX WST("%3") WNX WST("%4") WNX WST("%3") WNX WST("%4") WNX WST("%3") WNX WST("%4")
                 "s_mov_b32 m0, %0"
                 : "=&s"(keep), "=&s"(so) : "s"(m0base), "v"(ve), "v"(vo), "s"(rs), "s"(soff) : "scc", "memory");
#undef WST
#undef WNX
    static_assert(16 * NFULL == 0x40000, "");
}
template <int S> DI void mfma4(f32x4& c0, f32x4& c1, f32x4& c2, f32x4& c3, const u32x8& x, const f32x4& cinit) {
    constexpr int B = S * 32;
    if constexpr (S == 0) {
        asm volatile("s_nop 1\n\t"
                     "v_mfma_f32_16x16x128_f8f6f4 %0, a[%c6:%c6+7], %4, %5\n\t"
                     "v_mfma_f32_16x16x128_f8f6f4 %1, a[%c6+8:%c6+15], %4, %5\n\t"
                     "v_mfma_f32_16x16x128_f8f6f4 %2, a[%c6+16:%c6+23], %4, %5\n\t"
                     "v_mfma_f32_16x16x128_f8f6f4 %3, a[%c6+24:%c6+31], %4, %5"
                     : "=&v"(c0), "=&v"(c1), "=&v"(c2), "=&v"(c3) : "v"(x), "v"(cinit), "i"(B));
    } else if constexpr (S < 7) {
        asm volatile("s_nop 1\n\t"
                     "v_mfma_f32_16x16x128_f8f6f4 %0, a[%c5:%c5+7], %4, %0\n\t"
                     "v_mfma_f32_16x16x128_f8f6f4 %1, a[%c5+8:%c5+15], %4, %1\n\t"
                     "v_mfma_f32_16x16x128_f8f6f4 %2, a[%c5+16:%c5+23], %4, %2\n\t"
                     "v_mfma_f32_16x16x128_f8f6f4 %3, a[%c5+24:%c5+31], %4, %3"
                     : "+v"(c0), "+v"(c1), "+v"(c2), "+v"(c3) : "v"(x), "i"(B));
    } else {
        asm volatile("s_nop 1\n\t"
                     "v_mfma_f32_16x16x128_f8f6f4 %1, a[%c5+8:%c5+15], %4, %1\n\t"
                     "v_mfma_f32_16x16x128_f8f6f4 %2, a[%c5+16:%c5+23], %4, %2\n\t"
                     "v_mfma_f32_16x16x128_f8f6f4 %3, a[%c5+24:%c5+31], %4, %3\n\t"
                     "v_mfma_f32_16x16x128_f8f6f4 %0, a[%c5:%c5+7], %4, %0\n\t"
                     "s_nop 7\n\ts_nop 3"
                     : "+v"(c0), "+v"(c1), "+v"(c2), "+v"(c3) : "v"(x), "i"(B));
    }
}

struct XBlk { u32x4 h[16]; };

__global__ __launch_bounds__(256, 1) void fused_kernel(Params p) {
    __shared__ __attribute__((aligned(16))) unsigned char lds[4 * 32768];
    const int lane = threadIdx.x & 63;
    const int wid = __builtin_amdgcn_readfirstlane(threadIdx.x >> 6);
    const int rank = p.rank;
    char* const me = p.ws[rank];
    const int bid = blockIdx.x;
    STAMP(0);

    char* const callp = me + WS_CALL + (size_t)(bid * 4 + wid) * 64;

    if (bid < 32) {
        if (bid >= NSEND_WG) return;
        const int w = bid * 4 + wid, pi = w % 7, j = w / 7;
        const unsigned peer = (unsigned)(rank + 1 + pi) & 7u;
        const unsigned vsrc = (lane & 15) * KDIM + (lane >> 4) * 16, vdst = lane * 16;
        unsigned vzero = 0;
        asm volatile("v_accvgpr_write_b32 a33, 1\n\ts_nop 1" : "+v"(vzero) :: "a0", "a31", "a32", "a33", "a34");
        auto srcp = [&](unsigned u) { u = u < NUNIT ? u : NUNIT - 1; return p.x + (u >> 4) * 65536u + (u & 15u) * 256u; };
        unsigned tk0 = j, tk1 = j + WPP, prev = NUNIT;
        snd_ticket(callp, vzero);
        snd_load<0>(srcp(tk0), vsrc);
        const unsigned call = snd_wait_ticket<4>() + 1u;
        asm volatile("s_nop 1\n\tv_accvgpr_write_b32 a34, %0\n\ts_nop 1" :: "s"(call));
        const unsigned tbase = (call - 1u) * NUNIT - 2u * WPP;
        char* const dst = p.ws[peer] + WS_X + (size_t)((call & 1u) * NR + rank) * SHARD;
        char* const fl = p.ws[peer] + WS_FLAG + rank * 2048;
        char* const ctr = me + WS_TICK + pi * 128;
        char* const dummy = me + WS_TMO + 128 + w * 4;
        auto flagp = [&](unsigned u) { return u < NUNIT ? fl + (((u >> 2) & 3u) * 128u + (u >> 4) * 4u + (u & 3u)) * 4u : dummy; };
#define SEND_STEP(BA, BB, NLOAD) { \
            snd_ticket(ctr, vzero); \
            snd_load<BB>(srcp(tk1), vsrc); \
            wait_vm<NLOAD>();                                            \
            snd_store<BA>(dst + tk0 * 4096u, vdst); \
            const unsigned tk2 = snd_wait_ticket<8>() - tbase;           \
            snd_flag(flagp(prev), vzero); \
            prev = tk0; tk0 = tk1; tk1 = tk2; }
        SEND_STEP(0, 16, 5)
        for (;;) {
            SEND_STEP(16, 0, 10) if (tk0 >= NUNIT) break;
            SEND_STEP(0, 16, 10) if (tk0 >= NUNIT) break;
        }
#undef SEND_STEP
        wait_vm<0>();
        snd_flag(flagp(prev), vzero);
        STAMP(1);
        return;
    }

    int cg, rg;
    {
        const int xcd = bid & 7, jj = (bid >> 3) - 4;
        cg = xcd * 4 + (jj & 3); rg = jj >> 2;
    }
    const int q = wid;
    const float scale = p.sx[0] * p.sw[0];
    unsigned call;
    const unsigned ldsw = (unsigned)(size_t)(LAS unsigned char*)lds + q * 32768u;
    {
        const __amdgpu_buffer_rsrc_t rw = __builtin_amdgcn_make_buffer_rsrc((void*)p.w, 0, (unsigned)KDIM * NFULL, 0x00020000);
        const unsigned ve = (lane >> 2) * NFULL + (((lane & 3) ^ ((lane >> 4) & 1)) * 16), vo = ve ^ 32u;
        const unsigned so = (unsigned)(q * 1024) * NFULL + rank * NSH + cg * 64;
        const unsigned ab = ldsw + ((lane >> 4) * 16 + ((lane & 15) >> 1)) * 64 + (lane & 1) * 8, f = (lane >> 3) & 3;
        const unsigned a0 = ab + (((0 ^ q) ^ f) << 4), a1 = ab + (((1 ^ q) ^ f) << 4), a2 = ab + (((2 ^ q) ^ f) << 4), a3 = ab + (((3 ^ q) ^ f) << 4);
        constexpr unsigned SS = 128u * NFULL;
        unsigned vzero = 0;
        asm volatile("v_accvgpr_write_b32 a33, 1\n\ts_nop 1" : "+v"(vzero) :: "a32", "a33");
        snd_ticket(callp, vzero);
        w_stage(rw, ve, vo, ldsw + 0 * 8192, so + 0 * SS); w_stage(rw, ve, vo, ldsw + 1 * 8192, so + 1 * SS);
        w_stage(rw, ve, vo, ldsw + 2 * 8192, so + 2 * SS); w_stage(rw, ve, vo, ldsw + 3 * 8192, so + 3 * SS);
        call = snd_wait_ticket<24>() + 1u;
        w_read<0, 0>(a0, a1, a2, a3); w_stage(rw, ve, vo, ldsw + 0 * 8192, so + 4 * SS);
        wait_vm<24>(); w_read<1, 1>(a0, a1, a2, a3); w_stage(rw, ve, vo, ldsw + 1 * 8192, so + 5 * SS);
        wait_vm<24>(); w_read<2, 2>(a0, a1, a2, a3); w_stage(rw, ve, vo, ldsw + 2 * 8192, so + 6 * SS);
        wait_vm<24>(); w_read<3, 3>(a0, a1, a2, a3); w_stage(rw, ve, vo, ldsw + 3 * 8192, so + 7 * SS);
        wait_vm<24>(); w_read<4, 0>(a0, a1, a2, a3);
        wait_vm<16>(); w_read<5, 1>(a0, a1, a2, a3);
        wait_vm<8>();  w_read<6, 2>(a0, a1, a2, a3);
        wait_vm<0>();  w_read<7, 3>(a0, a1, a2, a3);
    }
    STAMP(1);
    const unsigned peer = (unsigned)(rank + 1 + rg) & 7u;
    constexpr int nloc = 5, n = nloc + NBLK;
    const char* const xrem = me + WS_X + (size_t)((call & 1u) * NR + peer) * SHARD;
    const unsigned voffL = (lane & 15) * KDIM + (lane >> 4) * 16 + q * 1024, voffR = lane * 16 + q * 16384;
    const char* const pollp = lane < 32 ? me + WS_FLAG + peer * 2048 + q * 512 + lane * 16 : me + WS_TMO;
    const unsigned voffO = (lane & 15) * (NSH * 4) + (cg * 64 + q * 16 + (lane >> 4) * 4) * 4;
    LAS unsigned char* const lw = (LAS unsigned char*)lds + q * 32768 + lane * 16;
    LAS unsigned char* const lr1 = (LAS unsigned char*)lds + (q ^ 1) * 32768 + 1024 + lane * 16;
    LAS unsigned char* const lr2 = (LAS unsigned char*)lds + (q ^ 2) * 32768 + 2048 + lane * 16;
    LAS unsigned char* const lr3 = (LAS unsigned char*)lds + (q ^ 3) * 32768 + 3072 + lane * 16;
    int ready = nloc; bool dead = false;
    f32x4 cinit = {0.f, 0.f, 0.f, 0.f};

    auto ensure = [&](int i) {
        if (i >= ready && i < n) {
            const unsigned long long t0 = __builtin_amdgcn_s_memrealtime();
            for (;;) {
                u32x4 f;
                asm volatile("global_load_dwordx4 %0, %1, off sc0 sc1\n\ts_waitcnt vmcnt(0)" : "=&v"(f) : "v"(pollp) : "memory");
                const bool ok = f.x >= call && f.y >= call && f.z >= call && f.w >= call;
                const unsigned long long m = ~(__ballot(ok) & 0xffffffffull);
                ready = nloc + __builtin_ctzll(m);
                if (i < ready) break;
                if (__builtin_amdgcn_readlane(f.x, 32) == call || __builtin_amdgcn_s_memrealtime() - t0 > DEADLINE_TICKS) {
                    if (lane == 0) __hip_atomic_store((unsigned*)(me + WS_TMO), call, __ATOMIC_RELAXED, __HIP_MEMORY_SCOPE_AGENT);
                    dead = true; ready = n; const float qn = __builtin_nanf(""); cinit = (f32x4){qn, qn, qn, qn};
                    break;
                }
                __builtin_amdgcn_s_sleep(1);
            }
        }
    };
    struct Src { __amdgpu_buffer_rsrc_t rs; unsigned voff, soff, pstr; };
    auto source = [&](int i) {
        const bool loc = i < nloc;
        Src s;
        s.rs = __builtin_amdgcn_make_buffer_rsrc(loc ? (void*)p.x : (void*)xrem, 0, (i < n && !dead && !(loc && rg + 7 * i >= NBLK)) ? SHARD : 0u, 0x00020000);
        s.voff = loc ? voffL : voffR; s.pstr = loc ? 64u : 1024u;
        s.soff = (unsigned)((loc ? rg + 7 * i : i - nloc) & 31) * 65536u;
        return s;
    };
#define XLOAD(X, s, k) (X).h[k] = __builtin_bit_cast(u32x4, __builtin_amdgcn_raw_buffer_load_b128((s).rs, (s).voff, (s).soff + (k) * (s).pstr, 17))
#define XFRAG(X, S) __builtin_shufflevector((X).h[2 * (S)], (X).h[2 * (S) + 1], 0, 1, 2, 3, 4, 5, 6, 7)
#define KSTEP(CUR, NXT, S) XLOAD(NXT, sn, 2 * (S)); XLOAD(NXT, sn, 2 * (S) + 1); mfma4<S>(c0, c1, c2, c3, XFRAG(CUR, S), cinit); __builtin_amdgcn_sched_barrier(0);
#define BODY(CUR, NXT) { \
        ensure(i + 2); \
        const Src sn = source(i + 2); \
        f32x4 c0, c1, c2, c3; \
        KSTEP(CUR, NXT, 0) KSTEP(CUR, NXT, 1) KSTEP(CUR, NXT, 2) KSTEP(CUR, NXT, 3) \
        KSTEP(CUR, NXT, 4) KSTEP(CUR, NXT, 5) KSTEP(CUR, NXT, 6) KSTEP(CUR, NXT, 7) \
        const unsigned po = (unsigned)(i & 1) * 4096u; \
        *(LAS f32x4*)(lw + po + 1024) = c1; *(LAS f32x4*)(lw + po + 2048) = c2; *(LAS f32x4*)(lw + po + 3072) = c3; \
        wait_lgkm0(); __builtin_amdgcn_s_barrier(); asm volatile("" ::: "memory"); \
        const f32x4 r = ((c0 + *(const LAS f32x4*)(lr1 + po)) + (*(const LAS f32x4*)(lr2 + po) + *(const LAS f32x4*)(lr3 + po))) * scale; \
        const unsigned row0 = i < nloc ? (unsigned)(rank * MSH + (rg + 7 * i) * 16) : peer * MSH + (unsigned)(i - nloc) * 16u; \
        const __amdgpu_buffer_rsrc_t ro = __builtin_amdgcn_make_buffer_rsrc((void*)p.out, 0, (i < nloc && rg + 7 * i >= NBLK) ? 0u : (unsigned)(NR * MSH) * NSH * 4u, 0x00020000); \
        __builtin_amdgcn_raw_buffer_store_b128(__builtin_bit_cast(u32x4, r), ro, voffO, row0 * (NSH * 4u), OUT_AUX); \
        STAMP(2 + i); \
        ++i; }

    XBlk X0, X1, X2;
    {
        const Src s0 = source(0), s1 = source(1);
#pragma unroll
        for (int k = 0; k < 16; ++k) XLOAD(X2, s0, k);
#pragma unroll
        for (int k = 0; k < 16; ++k) XLOAD(X0, s1, k);
    }
    static_assert((n - 1) % 3 == 0, "");
    int i = 0;
    BODY(X2, X1)
    while (i < n) { BODY(X0, X2) BODY(X1, X0) BODY(X2, X1) }
#undef BODY
#undef KSTEP
#undef XFRAG
#undef XLOAD
}
}

extern "C" void kernel_launch(int rank, int nranks, ncclComm_t comm,
                              hipStream_t stream,
                              void* const* d_in, const int* in_sizes, int n_in,
                              void* d_out, int out_size,
                              const int* in_dtypes, int out_dtype,
                              void* const* d_out_all,
                              void* const* d_ws_all, size_t ws_bytes) {
    (void)comm; (void)in_dtypes; (void)out_dtype; (void)d_out_all;
    if (nranks != NR || (unsigned)rank >= (unsigned)NR || n_in != 4 || ws_bytes < WS_END ||
        in_sizes[0] != MSH * KDIM || in_sizes[1] != KDIM * NFULL || out_size != NR * MSH * NSH) {
        fprintf(stderr, "kernel_launch: unexpected problem shape\n");
        return;
    }
    Params p;
    p.x = (const unsigned char*)d_in[0]; p.w = (const unsigned char*)d_in[1];
    p.sx = (const float*)d_in[2]; p.sw = (const float*)d_in[3]; p.out = (float*)d_out;
    for (int i = 0; i < NR; ++i) p.ws[i] = (char*)d_ws_all[i];
    p.rank = rank; p.pad = 0;
    fused_kernel<<<256, 256, 0, stream>>>(p);
    HIP_CHECK(hipGetLastError());
}
```

```cpp
#include <hip/hip_runtime.h>
#include <rccl/rccl.h>
#include <cstdio>

#define HIP_CHECK(expr) do { hipError_t _e = (expr); if (_e != hipSuccess) { \
    fprintf(stderr, "HIP error %s at %s:%d\n", hipGetErrorString(_e), __FILE__, __LINE__); \
    return; } } while (0)

namespace {
typedef unsigned u32x2 __attribute__((ext_vector_type(2)));
typedef unsigned u32x4 __attribute__((ext_vector_type(4)));
typedef unsigned u32x8 __attribute__((ext_vector_type(8)));
typedef float f32x4 __attribute__((ext_vector_type(4)));
#define LAS __attribute__((address_space(3)))
#define DI __device__ __forceinline__

constexpr int NR = 8;
constexpr int KDIM = 4096, NFULL = 16384, NSH = 2048, MSH = 512;
constexpr unsigned SHARD = MSH * KDIM;
constexpr int NBLK = MSH / 16;
constexpr int NUNIT = 512;
constexpr int WPP = 16;
constexpr size_t WS_X = 0;
constexpr size_t WS_FLAG = 2ull * NR * SHARD;
constexpr size_t WS_TICK = WS_FLAG + NR * 2048;
constexpr size_t WS_TMO = WS_TICK + 1024;
constexpr size_t WS_CALL = WS_TMO + 1024;
constexpr size_t WS_DBG = WS_CALL + 1024 * 64;
constexpr size_t WS_END = WS_DBG + 2 * 256 * 64 * 8;
constexpr unsigned long long DEADLINE_TICKS = 30000000ull;

#define MAPMODE 2
#define OWNMASK 0x1full
#define POLL_AHEAD 3
#define SPIN_SLEEP 1
#define ROT rg
#define OUT_AUX 0

struct Params {
    const unsigned char* x; const unsigned char* w; const float* sx; const float* sw; float* out;
    char* ws[NR]; int rank; int pad;
};

#define ACLOB "a0","a1","a2","a3","a4","a5","a6","a7","a8","a9","a10","a11","a12","a13","a14","a15","a16","a17","a18","a19","a20","a21","a22","a23","a24","a25","a26","a27","a28","a29","a30","a31","a32","a33","a34","a35","a36","a37","a38","a39","a40","a41","a42","a43","a44","a45","a46","a47","a48","a49","a50","a51","a52","a53","a54","a55","a56","a57","a58","a59","a60","a61","a62","a63","a64","a65","a66","a67","a68","a69","a70","a71","a72","a73","a74","a75","a76","a77","a78","a79","a80","a81","a82","a83","a84","a85","a86","a87","a88","a89","a90","a91","a92","a93","a94","a95","a96","a97","a98","a99","a100","a101","a102","a103","a104","a105","a106","a107","a108","a109","a110","a111","a112","a113","a114","a115","a116","a117","a118","a119","a120","a121","a122","a123","a124","a125","a126","a127","a128","a129","a130","a131","a132","a133","a134","a135","a136","a137","a138","a139","a140","a141","a142","a143","a144","a145","a146","a147","a148","a149","a150","a151","a152","a153","a154","a155","a156","a157","a158","a159","a160","a161","a162","a163","a164","a165","a166","a167","a168","a169","a170","a171","a172","a173","a174","a175","a176","a177","a178","a179","a180","a181","a182","a183","a184","a185","a186","a187","a188","a189","a190","a191","a192","a193","a194","a195","a196","a197","a198","a199","a200","a201","a202","a203","a204","a205","a206","a207","a208","a209","a210","a211","a212","a213","a214","a215","a216","a217","a218","a219","a220","a221","a222","a223","a224","a225","a226","a227","a228","a229","a230","a231","a232","a233","a234","a235","a236","a237","a238","a239","a240","a241","a242","a243","a244","a245","a246","a247","a248","a249","a250","a251","a252","a253","a254","a255"

template <int N> DI void wait_vm() { asm volatile("s_waitcnt vmcnt(%0)" :: "n"(N) : "memory"); }
DI void wait_lgkm0() { asm volatile("s_waitcnt lgkmcnt(0)" ::: "memory"); }
template <int B> DI void snd_load(const unsigned char* base, unsigned voff) {
    asm volatile("global_load_dwordx4 a[%c2:%c2+3], %0, %1\n\t"
                 "global_load_dwordx4 a[%c2+4:%c2+7], %0, %1 offset:64\n\t"
                 "global_load_dwordx4 a[%c2+8:%c2+11], %0, %1 offset:128\n\t"
                 "global_load_dwordx4 a[%c2+12:%c2+15], %0, %1 offset:192" :: "v"(voff), "s"(base), "i"(B) : "memory");
}
template <int B> DI void snd_store(char* base, unsigned voff) {
    asm volatile("global_store_dwordx4 %0, a[%c2:%c2+3], %1 sc0 sc1\n\t"
                 "global_store_dwordx4 %0, a[%c2+4:%c2+7], %1 offset:1024 sc0 sc1\n\t"
                 "global_store_dwordx4 %0, a[%c2+8:%c2+11], %1 offset:2048 sc0 sc1\n\t"
                 "global_store_dwordx4 %0, a[%c2+12:%c2+15], %1 offset:3072 sc0 sc1\n\ts_nop 1" :: "v"(voff), "s"(base), "i"(B) : "memory");
}
DI void snd_ticket(char* ctr, unsigned vzero) {
    unsigned long long keep;
    asm volatile("s_mov_b64 %0, exec\n\ts_mov_b64 exec, 1\n\tglobal_atomic_add a32, %1, a33, %2 sc0\n\ts_mov_b64 exec, %0"
                 : "=&s"(keep) : "v"(vzero), "s"(ctr) : "memory");
}
template <int A> DI void snd_flag(char* flag, unsigned vzero) {
    unsigned long long keep;
    asm volatile("s_mov_b64 %0, exec\n\ts_mov_b64 exec, 1\n\tglobal_store_dword %1, a[%c3], %2 sc0 sc1\n\ts_mov_b64 exec, %0"
                 : "=&s"(keep) : "v"(vzero), "s"(flag), "i"(A) : "memory");
}
template <int A> DI void acc_set(unsigned v) { asm volatile("s_nop 1\n\tv_accvgpr_write_b32 a[%c1], %0\n\ts_nop 1" :: "s"(v), "i"(A)); }
DI unsigned unit_src(unsigned u) { return (u >> 4) * 65536u + (u & 15u) * 256u; }
DI unsigned unit_flag(unsigned u) { return (((u >> 2) & 3u) * 128u + (u >> 4) * 4u + (u & 3u)) * 4u; }
template <int N> DI unsigned snd_wait_ticket() {
    unsigned v;
    asm volatile("s_waitcnt vmcnt(%1)\n\tv_accvgpr_read_b32 %0, a32\n\ts_nop 1" : "=v"(v) : "n"(N) : "memory");
    return __builtin_amdgcn_readfirstlane(v);
}

#define STAMP(k) do { } while (0)
#define BSTAMP(i) do { } while (0)

template <int S, int SLOT> DI void w_read(unsigned a0, unsigned a1, unsigned a2, unsigned a3) {
    constexpr int B = S * 32, O = SLOT * 8192;
#define TRR(c, t, A) "ds_read_b64_tr_b8 a[%c4+" #c "*8+" #t "*2:%c4+" #c "*8+" #t "*2+1], " A " offset:%c5+4096*(" #t "/2)+512*(" #t "%%2)\n\t"
    asm volatile(TRR(0, 0, "%0") TRR(0, 1, "%0") TRR(0, 2, "%0") TRR(0, 3, "%0")
                 TRR(1, 0, "%1") TRR(1, 1, "%1") TRR(1, 2, "%1") TRR(1, 3, "%1")
                 TRR(2, 0, "%2") TRR(2, 1, "%2") TRR(2, 2, "%2") TRR(2, 3, "%2")
                 TRR(3, 0, "%3") TRR(3, 1, "%3") TRR(3, 2, "%3") TRR(3, 3, "%3")
                 "s_waitcnt lgkmcnt(0)"
                 :: "v"(a0), "v"(a1), "v"(a2), "v"(a3), "i"(B), "i"(O) : "memory", ACLOB);
#undef TRR
}
DI void w_stage(__amdgpu_buffer_rsrc_t rs, unsigned ve, unsigned vo, unsigned m0base, unsigned soff) {
    unsigned keep, so;
#define WST(V) "s_nop 0\n\tbuffer_load_dwordx4 " V ", %5, %1 offen lds\n\t"
#define WNX "s_add_u32 m0, m0, 0x400\n\ts_add_u32 %1, %1, 0x40000\n\t"
    asm volatile("s_mov_b32 %0, m0\n\ts_mov_b32 m0, %2\n\ts_mov_b32 %1, %6\n\t"
                 WST("%3") WNX WST("%4") WNX WST("%3") WNX WST("%4") WNX WST("%3") WNX WST("%4") WNX WST("%3") WNX WST("%4")
                 "s_mov_b32 m0, %0"
                 : "=&s"(keep), "=&s"(so) : "s"(m0base), "v"(ve), "v"(vo), "s"(rs), "s"(soff) : "scc", "memory");
#undef WST
#undef WNX
    static_assert(16 * NFULL == 0x40000, "");
}
template <int S, int C> DI void mf(f32x4& c, const u32x8& x, const f32x4& cinit) {
    constexpr int B = S * 32 + C * 8;
    if constexpr (S == 0) asm volatile("v_mfma_f32_16x16x128_f8f6f4 %0, a[%c3:%c3+7], %1, %2" : "+v"(c) : "v"(x), "v"(cinit), "i"(B));
    else asm volatile("v_mfma_f32_16x16x128_f8f6f4 %0, a[%c2:%c2+7], %1, %0" : "+v"(c) : "v"(x), "i"(B));
}
DI float fadd(float a, float b) { float r; asm("v_add_f32_e32 %0, %1, %2" : "=v"(r) : "v"(a), "v"(b)); return r; }
DI float fmul(float a, float b) { float r; asm("v_mul_f32_e32 %0, %1, %2" : "=v"(r) : "v"(a), "v"(b)); return r; }
DI f32x4 fadd4(f32x4 a, f32x4 b) { return (f32x4){fadd(a[0], b[0]), fadd(a[1], b[1]), fadd(a[2], b[2]), fadd(a[3], b[3])}; }
DI f32x4 fmul4(f32x4 a, float b) { return (f32x4){fmul(a[0], b), fmul(a[1], b), fmul(a[2], b), fmul(a[3], b)}; }
DI f32x4 copy4(const f32x4& a) {
    f32x4 r;
#pragma unroll
    for (int j = 0; j < 4; ++j) { float t; asm volatile("v_mov_b32 %0, %1" : "=v"(t) : "v"(a[j])); r[j] = t; }
    return r;
}

#define SCHED 0
constexpr int ld_at(int j) { return (j & 1) ? j >> 1 : -1; }
constexpr int S_POLL = 4, S_BAR = 6, S_RD = 8, S_A1 = 12, S_A2 = 14, S_A3 = 16, S_MU = 18, S_ST = 20, S_K1 = 8, S_K2 = 10, S_K3 = 22, S_K4 = 24, S_K5 = 26, S_K6 = 28, S_K7 = 30;

struct XBlk { u32x4 h[16]; };

__global__ __launch_bounds__(256, 1) void fused_kernel(Params p) {
    __shared__ __attribute__((aligned(16))) unsigned char lds[4 * 32768];
    const int lane = threadIdx.x & 63;
    const int wid = __builtin_amdgcn_readfirstlane(threadIdx.x >> 6);
    const int rank = p.rank;
    char* const me = p.ws[rank];
    const int bid = blockIdx.x;
    STAMP(0);

    char* const callp = me + WS_CALL + (size_t)(bid * 4 + wid) * 64;

    const unsigned vsrc = (lane & 15) * KDIM + (lane >> 4) * 16, vdst = lane * 16;

    if (bid < 32) {
        const int w = bid * 4 + wid, pi = w % 7, j = w / 7;
        const unsigned peer = (unsigned)(rank + 1 + pi) & 7u;
        unsigned vzero = 0;
        asm volatile("v_accvgpr_write_b32 a33, 1\n\ts_nop 1" : "+v"(vzero) :: "a0", "a31", "a32", "a33", "a34");
        if (w >= 7 * WPP) return;
        auto srcp = [&](unsigned u) { return p.x + unit_src(u < NUNIT ? u : NUNIT - 1); };
        unsigned tk0 = j, tk1 = j + WPP, prev = NUNIT;
        snd_ticket(callp, vzero);
        snd_load<0>(srcp(tk0), vsrc);
        const unsigned call = snd_wait_ticket<4>() + 1u;
        acc_set<34>(call);
        const unsigned tbase = (call - 1u) * NUNIT - 2u * WPP;
        char* const dst = p.ws[peer] + WS_X + (size_t)((call & 1u) * NR + rank) * SHARD;
        char* const fl = p.ws[peer] + WS_FLAG + rank * 2048;
        char* const ctr = me + WS_TICK + pi * 128;
        char* const dummy = me + WS_TMO + 128 + w * 4;
        auto flagp = [&](unsigned u) { return u < NUNIT ? fl + unit_flag(u) : dummy; };
#define SEND_STEP(BA, BB, NLOAD) { \
            snd_ticket(ctr, vzero); \
            snd_load<BB>(srcp(tk1), vsrc); \
            wait_vm<NLOAD>();                                            \
            snd_store<BA>(dst + tk0 * 4096u, vdst); \
            const unsigned tk2 = snd_wait_ticket<8>() - tbase;           \
            snd_flag<34>(flagp(prev), vzero); \
            prev = tk0; tk0 = tk1; tk1 = tk2; }
        SEND_STEP(0, 16, 5)
        for (;;) {
            SEND_STEP(16, 0, 10) if (tk0 >= NUNIT) break;
            SEND_STEP(0, 16, 10) if (tk0 >= NUNIT) break;
        }
#undef SEND_STEP
        wait_vm<0>();
        snd_flag<34>(flagp(prev), vzero);
        STAMP(1);
        return;
    }

    int cg, rg;
    {
        const int xcd = bid & 7, jj = (bid >> 3) - 4;
        const int i = (xcd & 1) * 28 + jj; rg = i >> 3; cg = (xcd >> 1) * 8 + (i & 7);
    }
    const int q = wid;
    const float scale = p.sx[0] * p.sw[0];
    unsigned call;
    const unsigned ldsw = (unsigned)(size_t)(LAS unsigned char*)lds + q * 32768u;
    {
        const __amdgpu_buffer_rsrc_t rw = __builtin_amdgcn_make_buffer_rsrc((void*)p.w, 0, (unsigned)KDIM * NFULL, 0x00020000);
        const unsigned ve = (lane >> 2) * NFULL + (((lane & 3) ^ ((lane >> 4) & 1)) * 16), vo = ve ^ 32u;
        const unsigned so = (unsigned)(q * 1024) * NFULL + rank * NSH + cg * 64;
        const unsigned ab = ldsw + ((lane >> 4) * 16 + ((lane & 15) >> 1)) * 64 + (lane & 1) * 8, f = (lane >> 3) & 3;
        const unsigned a0 = ab + (((0 ^ q) ^ f) << 4), a1 = ab + (((1 ^ q) ^ f) << 4), a2 = ab + (((2 ^ q) ^ f) << 4), a3 = ab + (((3 ^ q) ^ f) << 4);
        constexpr unsigned SS = 128u * NFULL;
        unsigned vzero = 0;
        asm volatile("v_accvgpr_write_b32 a33, 1\n\ts_nop 1" : "+v"(vzero) :: "a32", "a33");
        snd_ticket(callp, vzero);
#define WOFF(S) (so + (unsigned)(((S) + ROT) & 7) * SS)
#define WFIRST w_stage(rw, ve, vo, ldsw + 0 * 8192, WOFF(0)); w_stage(rw, ve, vo, ldsw + 1 * 8192, WOFF(1)); \
               w_stage(rw, ve, vo, ldsw + 2 * 8192, WOFF(2)); w_stage(rw, ve, vo, ldsw + 3 * 8192, WOFF(3));
        if (rg >= 3) {
            const unsigned uo = (unsigned)(((rg - 3) * 32 + cg) * 4 + q);
            snd_load<224>(p.x + unit_src(uo), vsrc);
            call = snd_wait_ticket<0>() + 1u;
            acc_set<240>(call);
            snd_store<224>(me + WS_X + (size_t)((call & 1u) * NR + rank) * SHARD + uo * 4096u, vdst);
            WFIRST
            wait_vm<32>();
            snd_flag<240>(me + WS_FLAG + rank * 2048 + unit_flag(uo), vzero);
            wait_vm<24>();
        } else {
            WFIRST
            call = snd_wait_ticket<24>() + 1u;
        }
#undef WFIRST
        w_read<0, 0>(a0, a1, a2, a3); w_stage(rw, ve, vo, ldsw + 0 * 8192, WOFF(4));
        wait_vm<24>(); w_read<1, 1>(a0, a1, a2, a3); w_stage(rw, ve, vo, ldsw + 1 * 8192, WOFF(5));
        wait_vm<24>(); w_read<2, 2>(a0, a1, a2, a3); w_stage(rw, ve, vo, ldsw + 2 * 8192, WOFF(6));
        wait_vm<24>(); w_read<3, 3>(a0, a1, a2, a3); w_stage(rw, ve, vo, ldsw + 3 * 8192, WOFF(7));
#undef WOFF
        wait_vm<24>(); w_read<4, 0>(a0, a1, a2, a3);
        wait_vm<16>(); w_read<5, 1>(a0, a1, a2, a3);
        wait_vm<8>();  w_read<6, 2>(a0, a1, a2, a3);
        wait_vm<0>();  w_read<7, 3>(a0, a1, a2, a3);
    }
    STAMP(1);
    const unsigned peer = (unsigned)(rank + 1 + rg) & 7u;
    constexpr int n = NBLK + 5;
    constexpr unsigned long long OWNPOS = OWNMASK;
    static_assert(__builtin_popcountll(OWNPOS) == 5 && (OWNPOS >> n) == 0, "");
    void* const xall = me + WS_X + (size_t)(call & 1u) * NR * SHARD;
    const unsigned sfl = peer * 2048 + q * 512, sflo = rank * 2048 + q * 512, vfl = lane * 8;
    const unsigned voffO = (lane & 15) * (NSH * 4) + (cg * 64 + q * 16 + (lane >> 4) * 4) * 4;
    LAS unsigned char* const lw = (LAS unsigned char*)lds + q * 32768 + lane * 16;
    LAS unsigned char* const lr1 = (LAS unsigned char*)lds + (q ^ 1) * 32768 + 1024 + lane * 16;
    LAS unsigned char* const lr2 = (LAS unsigned char*)lds + (q ^ 2) * 32768 + 2048 + lane * 16;
    LAS unsigned char* const lr3 = (LAS unsigned char*)lds + (q ^ 3) * 32768 + 3072 + lane * 16;
    int lim = NBLK;
    f32x4 cinit = {0.f, 0.f, 0.f, 0.f};
    unsigned xszn = NR * SHARD;

    auto count = [&](u32x2 f) { const unsigned long long m = ~__ballot(f.x >= call && f.y >= call); return m ? (int)(__builtin_ctzll(m) >> 1) : 2 * NBLK; };
    auto spin = [&](unsigned soff, int need) {
        const char* const pf = me + WS_FLAG + soff;
        const unsigned long long t0 = __builtin_amdgcn_s_memrealtime();
        asm volatile("" :: "s"(t0));
        for (;;) {
            u32x2 f; unsigned tmo;
            asm volatile("global_load_dwordx2 %0, %2, %3 sc0 sc1\n\tglobal_load_dword %1, %4, %5 sc1\n\ts_waitcnt vmcnt(0)"
                         : "=&v"(f), "=&v"(tmo) : "v"(vfl), "s"(pf), "v"(0u), "s"(me + WS_TMO) : "memory");
            const int c = count(f);
            if (c > need) return c;
            if (__builtin_amdgcn_readfirstlane(tmo) == call || __builtin_amdgcn_s_memrealtime() - t0 > DEADLINE_TICKS) {
                if (lane == 0) __hip_atomic_store((unsigned*)(me + WS_TMO), call, __ATOMIC_RELAXED, __HIP_MEMORY_SCOPE_AGENT);
                lim = 0; xszn = 0; const float qn = __builtin_nanf(""); cinit = (f32x4){qn, qn, qn, qn};
                asm volatile("s_nop 1" : "+v"(cinit));
                return 2 * NBLK;
            }
            __builtin_amdgcn_s_sleep(SPIN_SLEEP);
        }
    };
    unsigned voffR[8];
#pragma unroll
    for (int k = 0; k < 8; ++k) voffR[k] = vdst + q * 16384 + (unsigned)((k + ROT) & 7) * 2048u;
    f32x4 c0 = cinit, c1 = cinit, c2 = cinit, c3 = cinit;
    asm volatile("s_nop 1" : "+v"(cinit), "+v"(c0), "+v"(c1), "+v"(c2), "+v"(c3));
#define XLOAD(X, k) (X).h[k] = __builtin_bit_cast(u32x4, __builtin_amdgcn_raw_buffer_load_b128(rx, voffR[(k) >> 1], sx + ((k) & 1) * 1024u, 17))
#define POLL() __builtin_bit_cast(u32x2, __builtin_amdgcn_raw_buffer_load_b64(__builtin_amdgcn_make_buffer_rsrc(me + WS_FLAG, 0, psz, 0x00020000), vfl, sfl, 17))
    XBlk X0, X1, X2;
    int rdy = 0, rdyo = 0;
    int nr = 0, no = rg, blk, have;
    unsigned sx, sxn, g0, g1, g2, own, oso = 0, osz = 0, psz = 0;
    u32x2 P0 = {0u, 0u}, P1, P2;
#define FIRST(X, POS, G) { \
        if constexpr ((OWNPOS >> POS) & 1) { if (no >= rdyo) rdyo = spin(sflo, no); G = rank * NBLK + no; no += 7; } \
        else { if (nr >= rdy) rdy = spin(sfl, nr); G = peer * NBLK + nr; nr += 1; } \
        const __amdgpu_buffer_rsrc_t rx = __builtin_amdgcn_make_buffer_rsrc(xall, 0, xszn, 0x00020000); \
        sx = G * 65536u; \
        _Pragma("unroll") for (int k = 0; k < 16; ++k) XLOAD(X, k); }
    FIRST(X2, 0, g0) FIRST(X0, 1, g1)
#undef FIRST
    if constexpr ((OWNPOS >> 2) & 1) { own = 1; blk = no; have = rdyo; g2 = rank * NBLK + no; no += 7; } else { own = 0; blk = nr; have = rdy; g2 = peer * NBLK + nr; nr += 1; }
    sxn = g2 * 65536u;
    auto store_rows = [&](const f32x4& r) {
        const __amdgpu_buffer_rsrc_t ro = __builtin_amdgcn_make_buffer_rsrc((void*)p.out, 0, osz, 0x00020000);
        __builtin_amdgcn_raw_buffer_store_b128(__builtin_bit_cast(u32x4, r), ro, voffO, oso, OUT_AUX);
    };
#define SB __builtin_amdgcn_sched_barrier(0);
#define XFRAG(X, S) __builtin_shufflevector((X).h[2 * (S)], (X).h[2 * (S) + 1], 0, 1, 2, 3, 4, 5, 6, 7)
#define PIN(v) asm volatile("" : "+s"(v));
#define SLOT(CUR, NXT, PP, PC, PO, PN, J, C) \
        mf<(J) / 4, C>(c##C, XFRAG(CUR, (J) / 4), cinit); SB \
        if constexpr ((J) == 0) *(LAS f32x4*)(lw + PP * 4096 + 2048) = c2; \
        if constexpr ((J) == 1) *(LAS f32x4*)(lw + PP * 4096 + 3072) = c3; \
        if constexpr ((J) == 2) keep = copy4(c0); \
        if constexpr ((J) == 31) *(LAS f32x4*)(lw + PC * 4096 + 1024) = c1; \
        if constexpr ((J) == S_POLL) PN = POLL(); \
        if constexpr (ld_at(J) >= 0) XLOAD(NXT, ld_at(J) & 15); \
        if constexpr ((J) == S_BAR) { wait_lgkm0(); __builtin_amdgcn_s_barrier(); asm volatile("" ::: "memory"); } \
        if constexpr ((J) == S_RD) { r1 = *(const LAS f32x4*)(lr1 + PP * 4096); r2 = *(const LAS f32x4*)(lr2 + PP * 4096); r3 = *(const LAS f32x4*)(lr3 + PP * 4096); } \
        if constexpr ((J) == S_A1) r1 = fadd4(keep, r1); \
        if constexpr ((J) == S_A2) r2 = fadd4(r2, r3); \
        if constexpr ((J) == S_A3) r1 = fadd4(r1, r2); \
        if constexpr ((J) == S_MU) r1 = fmul4(r1, scale); \
        if constexpr ((J) == S_ST) store_rows(r1); \
        if constexpr ((J) == S_K1) { PIN(i) own = (unsigned)(OWNPOS >> (i + 3)) & 1u; PIN(own) } \
        if constexpr ((J) == S_K2) { PIN(own) blk = own ? no : nr; src = own ? rank : peer; int no7 = no + 7, nr1 = nr + 1; PIN(no7) PIN(nr1) no = own ? no7 : no; nr = own ? nr : nr1; PIN(blk) PIN(src) PIN(no) PIN(nr) } \
        if constexpr ((J) == S_K3) { PIN(blk) PIN(src) sxn = (src * NBLK + (blk & 31)) * 65536u; xszn = blk < lim ? NR * SHARD : 0u; PIN(sxn) PIN(xszn) } \
        if constexpr ((J) == S_K4) { PIN(g0) oso = g0 * 131072u; osz = (int)g0 < 0 ? 0u : (unsigned)(NR * MSH) * NSH * 4u; PIN(oso) PIN(osz) } \
        if constexpr ((J) == S_K5) { PIN(sxn) PIN(blk) g0 = g1; g1 = g2; g2 = (sxn >> 16) | (blk < NBLK ? 0u : 0x80000000u); PIN(g2) } \
        if constexpr ((J) == S_K6) { asm volatile("" : "+v"(PO)); const int c = count(PO); rdy = c > rdy ? c : rdy; PIN(rdy) } \
        if constexpr ((J) == S_K7) { PIN(own) PIN(blk) have = own ? rdyo : rdy; PIN(have) have = blk < NBLK ? have : 2 * NBLK; psz = rdy - nr < POLL_AHEAD ? NR * 2048 : 0u; BSTAMP(i); ++i; PIN(have) PIN(psz) PIN(i) } \
        SB
#define STEP4(CUR, NXT, PP, PC, PO, PN, S) SLOT(CUR, NXT, PP, PC, PO, PN, 4 * S, 1) SLOT(CUR, NXT, PP, PC, PO, PN, 4 * S + 1, 2) SLOT(CUR, NXT, PP, PC, PO, PN, 4 * S + 2, 3) SLOT(CUR, NXT, PP, PC, PO, PN, 4 * S + 3, 0)
#define BODY(CUR, NXT, PP, PC, PO, PN) { \
        if (blk >= have) { const int c = spin(own ? sflo : sfl, blk); rdyo = own ? c : rdyo; rdy = own ? rdy : c; } \
        sx = sxn; \
        const __amdgpu_buffer_rsrc_t rx = __builtin_amdgcn_make_buffer_rsrc(xall, 0, xszn, 0x00020000); \
        f32x4 keep, r1, r2, r3; unsigned src; \
        STEP4(CUR, NXT, PP, PC, PO, PN, 0) STEP4(CUR, NXT, PP, PC, PO, PN, 1) STEP4(CUR, NXT, PP, PC, PO, PN, 2) STEP4(CUR, NXT, PP, PC, PO, PN, 3) \
        STEP4(CUR, NXT, PP, PC, PO, PN, 4) STEP4(CUR, NXT, PP, PC, PO, PN, 5) STEP4(CUR, NXT, PP, PC, PO, PN, 6) STEP4(CUR, NXT, PP, PC, PO, PN, 7) }

    static_assert((n - 1) % 3 == 0, "");
    int i = 0;
    BODY(X2, X1, 1, 2, P0, P2)
    do { BODY(X0, X2, 2, 0, P2, P0) BODY(X1, X0, 0, 1, P0, P1) BODY(X2, X1, 1, 2, P1, P2) } while (i < n);
    {
        asm volatile("s_nop 7\n\ts_nop 7" : "+v"(c0), "+v"(c2), "+v"(c3));
        *(LAS f32x4*)(lw + 2 * 4096 + 2048) = c2; *(LAS f32x4*)(lw + 2 * 4096 + 3072) = c3;
        wait_lgkm0(); __builtin_amdgcn_s_barrier(); asm volatile("" ::: "memory");
        store_rows(fmul4(fadd4(fadd4(c0, *(const LAS f32x4*)(lr1 + 2 * 4096)), fadd4(*(const LAS f32x4*)(lr2 + 2 * 4096), *(const LAS f32x4*)(lr3 + 2 * 4096))), scale));
        STAMP(2 + n);
    }
#undef BODY
#undef STEP4
#undef SLOT
#undef XFRAG
#undef SB
#undef XLOAD
#undef POLL
#undef PIN
}
}

extern "C" void kernel_launch(int rank, int nranks, ncclComm_t comm,
                              hipStream_t stream,
                              void* const* d_in, const int* in_sizes, int n_in,
                              void* d_out, int out_size,
                              const int* in_dtypes, int out_dtype,
                              void* const* d_out_all,
                              void* const* d_ws_all, size_t ws_bytes) {
    (void)comm; (void)in_dtypes; (void)out_dtype; (void)d_out_all;
    if (nranks != NR || (unsigned)rank >= (unsigned)NR || n_in != 4 || ws_bytes < WS_END ||
        in_sizes[0] != MSH * KDIM || in_sizes[1] != KDIM * NFULL || out_size != NR * MSH * NSH) {
        fprintf(stderr, "kernel_launch: unexpected problem shape\n");
        return;
    }
    Params p;
    p.x = (const unsigned char*)d_in[0]; p.w = (const unsigned char*)d_in[1];
    p.sx = (const float*)d_in[2]; p.sw = (const float*)d_in[3]; p.out = (float*)d_out;
    for (int i = 0; i < NR; ++i) p.ws[i] = (char*)d_ws_all[i];
    p.rank = rank; p.pad = 0;
    fused_kernel<<<256, 256, 0, stream>>>(p);
    HIP_CHECK(hipGetLastError());
}
```

```cpp
#include <hip/hip_runtime.h>
#include <rccl/rccl.h>
#include <cstdio>

#define HIP_CHECK(expr) do { hipError_t _e = (expr); if (_e != hipSuccess) { \
    fprintf(stderr, "HIP error %s at %s:%d\n", hipGetErrorString(_e), __FILE__, __LINE__); \
    return; } } while (0)

namespace {
typedef unsigned u32x2 __attribute__((ext_vector_type(2)));
typedef unsigned u32x4 __attribute__((ext_vector_type(4)));
typedef unsigned u32x8 __attribute__((ext_vector_type(8)));
typedef float f32x4 __attribute__((ext_vector_type(4)));
#define LAS __attribute__((address_space(3)))
#define DI __device__ __forceinline__

constexpr int NR = 8;
constexpr int KDIM = 4096, NFULL = 16384, NSH = 2048, MSH = 512;
constexpr unsigned SHARD = MSH * KDIM;
constexpr int NBLK = MSH / 16;
constexpr int NUNIT = 512;
constexpr int WPP = 16;
constexpr size_t WS_X = 0;
constexpr size_t WS_FLAG = 2ull * NR * SHARD;
constexpr size_t WS_TICK = WS_FLAG + NR * 2048;
constexpr size_t WS_TMO = WS_TICK + 1024;
constexpr size_t WS_CALL = WS_TMO + 1024;
constexpr size_t WS_DBG = WS_CALL + 1024 * 64;
constexpr size_t WS_END = WS_DBG + 2 * 256 * 64 * 8;
constexpr unsigned long long DEADLINE_TICKS = 30000000ull;

#define MAPMODE 2
#define OWNMASK 0x2aaull
#define POLL_AHEAD 3
#define SPIN_SLEEP 1
#define XAUX 17
#define XSX sx
#define OSZ osz
#define BATCH3 0
#define ABL 0
#define DIST 2
#define ROT rg
#define OUT_AUX 2

struct Params {
    const unsigned char* x; const unsigned char* w; const float* sx; const float* sw; float* out;
    char* ws[NR]; int rank; int pad;
};

#define ACLOB "a0","a1","a2","a3","a4","a5","a6","a7","a8","a9","a10","a11","a12","a13","a14","a15","a16","a17","a18","a19","a20","a21","a22","a23","a24","a25","a26","a27","a28","a29","a30","a31","a32","a33","a34","a35","a36","a37","a38","a39","a40","a41","a42","a43","a44","a45","a46","a47","a48","a49","a50","a51","a52","a53","a54","a55","a56","a57","a58","a59","a60","a61","a62","a63","a64","a65","a66","a67","a68","a69","a70","a71","a72","a73","a74","a75","a76","a77","a78","a79","a80","a81","a82","a83","a84","a85","a86","a87","a88","a89","a90","a91","a92","a93","a94","a95","a96","a97","a98","a99","a100","a101","a102","a103","a104","a105","a106","a107","a108","a109","a110","a111","a112","a113","a114","a115","a116","a117","a118","a119","a120","a121","a122","a123","a124","a125","a126","a127","a128","a129","a130","a131","a132","a133","a134","a135","a136","a137","a138","a139","a140","a141","a142","a143","a144","a145","a146","a147","a148","a149","a150","a151","a152","a153","a154","a155","a156","a157","a158","a159","a160","a161","a162","a163","a164","a165","a166","a167","a168","a169","a170","a171","a172","a173","a174","a175","a176","a177","a178","a179","a180","a181","a182","a183","a184","a185","a186","a187","a188","a189","a190","a191","a192","a193","a194","a195","a196","a197","a198","a199","a200","a201","a202","a203","a204","a205","a206","a207","a208","a209","a210","a211","a212","a213","a214","a215","a216","a217","a218","a219","a220","a221","a222","a223","a224","a225","a226","a227","a228","a229","a230","a231","a232","a233","a234","a235","a236","a237","a238","a239","a240","a241","a242","a243","a244","a245","a246","a247","a248","a249","a250","a251","a252","a253","a254","a255"

template <int N> DI void wait_vm() { asm volatile("s_waitcnt vmcnt(%0)" :: "n"(N) : "memory"); }
DI void wait_lgkm0() { asm volatile("s_waitcnt lgkmcnt(0)" ::: "memory"); }
template <int B> DI void snd_load(const unsigned char* base, unsigned voff) {
    asm volatile("global_load_dwordx4 a[%c2:%c2+3], %0, %1\n\t"
                 "global_load_dwordx4 a[%c2+4:%c2+7], %0, %1 offset:64\n\t"
                 "global_load_dwordx4 a[%c2+8:%c2+11], %0, %1 offset:128\n\t"
                 "global_load_dwordx4 a[%c2+12:%c2+15], %0, %1 offset:192" :: "v"(voff), "s"(base), "i"(B) : "memory");
}
template <int B> DI void snd_store(char* base, unsigned voff) {
    asm volatile("global_store_dwordx4 %0, a[%c2:%c2+3], %1 sc0 sc1\n\t"
                 "global_store_dwordx4 %0, a[%c2+4:%c2+7], %1 offset:1024 sc0 sc1\n\t"
                 "global_store_dwordx4 %0, a[%c2+8:%c2+11], %1 offset:2048 sc0 sc1\n\t"
                 "global_store_dwordx4 %0, a[%c2+12:%c2+15], %1 offset:3072 sc0 sc1\n\ts_nop 1" :: "v"(voff), "s"(base), "i"(B) : "memory");
}
DI void snd_ticket(char* ctr, unsigned vzero) {
    unsigned long long keep;
    asm volatile("s_mov_b64 %0, exec\n\ts_mov_b64 exec, 1\n\tglobal_atomic_add a32, %1, a33, %2 sc0\n\ts_mov_b64 exec, %0"
                 : "=&s"(keep) : "v"(vzero), "s"(ctr) : "memory");
}
template <int A> DI void snd_flag(char* flag, unsigned vzero) {
    unsigned long long keep;
    asm volatile("s_mov_b64 %0, exec\n\ts_mov_b64 exec, 1\n\tglobal_store_dword %1, a[%c3], %2 sc0 sc1\n\ts_mov_b64 exec, %0"
                 : "=&s"(keep) : "v"(vzero), "s"(flag), "i"(A) : "memory");
}
template <int A> DI void acc_set(unsigned v) { asm volatile("s_nop 1\n\tv_accvgpr_write_b32 a[%c1], %0\n\ts_nop 1" :: "s"(v), "i"(A)); }
DI unsigned unit_src(unsigned u) { return (u >> 4) * 65536u + (u & 15u) * 256u; }
DI unsigned unit_flag(unsigned u) { return (((u >> 2) & 3u) * 128u + (u >> 4) * 4u + (u & 3u)) * 4u; }
template <int N> DI unsigned snd_wait_ticket() {
    unsigned v;
    asm volatile("s_waitcnt vmcnt(%1)\n\tv_accvgpr_read_b32 %0, a32\n\ts_nop 1" : "=v"(v) : "n"(N) : "memory");
    return __builtin_amdgcn_readfirstlane(v);
}

#define STAMP(k) do { } while (0)
#define BSTAMP(i) do { } while (0)

template <int S, int SLOT> DI void w_read(unsigned a0, unsigned a1, unsigned a2, unsigned a3) {
    constexpr int B = S * 32, O = SLOT * 8192;
#define TRR(c, t, A) "ds_read_b64_tr_b8 a[%c4+" #c "*8+" #t "*2:%c4+" #c "*8+" #t "*2+1], " A " offset:%c5+4096*(" #t "/2)+512*(" #t "%%2)\n\t"
    asm volatile(TRR(0, 0, "%0") TRR(0, 1, "%0") TRR(0, 2, "%0") TRR(0, 3, "%0")
                 TRR(1, 0, "%1") TRR(1, 1, "%1") TRR(1, 2, "%1") TRR(1, 3, "%1")
                 TRR(2, 0, "%2") TRR(2, 1, "%2") TRR(2, 2, "%2") TRR(2, 3, "%2")
                 TRR(3, 0, "%3") TRR(3, 1, "%3") TRR(3, 2, "%3") TRR(3, 3, "%3")
                 "s_waitcnt lgkmcnt(0)"
                 :: "v"(a0), "v"(a1), "v"(a2), "v"(a3), "i"(B), "i"(O) : "memory", ACLOB);
#undef TRR
}
DI void w_stage(__amdgpu_buffer_rsrc_t rs, unsigned ve, unsigned vo, unsigned m0base, unsigned soff) {
    unsigned keep, so;
#define WST(V) "s_nop 0\n\tbuffer_load_dwordx4 " V ", %5, %1 offen lds\n\t"
#define WNX "s_add_u32 m0, m0, 0x400\n\ts_add_u32 %1, %1, 0x40000\n\t"
    asm volatile("s_mov_b32 %0, m0\n\ts_mov_b32 m0, %2\n\ts_mov_b32 %1, %6\n\t"
                 WST("%3") WNX WST("%4") WNX WST("%3") WNX WST("%4") WNX WST("%3") WNX WST("%4") WNX WST("%3") WNX WST("%4")
                 "s_mov_b32 m0, %0"
                 : "=&s"(keep), "=&s"(so) : "s"(m0base), "v"(ve), "v"(vo), "s"(rs), "s"(soff) : "scc", "memory");
#undef WST
#undef WNX
    static_assert(16 * NFULL == 0x40000, "");
}
template <int S, int C> DI void mf(f32x4& c, const u32x8& x, const f32x4& cinit) {
    constexpr int B = S * 32 + C * 8;
    if constexpr (S == 0) asm volatile("v_mfma_f32_16x16x128_f8f6f4 %0, a[%c3:%c3+7], %1, %2" : "+v"(c) : "v"(x), "v"(cinit), "i"(B));
    else asm volatile("v_mfma_f32_16x16x128_f8f6f4 %0, a[%c2:%c2+7], %1, %0" : "+v"(c) : "v"(x), "i"(B));
}
DI float fadd(float a, float b) { float r; asm("v_add_f32_e32 %0, %1, %2" : "=v"(r) : "v"(a), "v"(b)); return r; }
DI float fmul(float a, float b) { float r; asm("v_mul_f32_e32 %0, %1, %2" : "=v"(r) : "v"(a), "v"(b)); return r; }
DI f32x4 fadd4(f32x4 a, f32x4 b) { return (f32x4){fadd(a[0], b[0]), fadd(a[1], b[1]), fadd(a[2], b[2]), fadd(a[3], b[3])}; }
DI f32x4 fmul4(f32x4 a, float b) { return (f32x4){fmul(a[0], b), fmul(a[1], b), fmul(a[2], b), fmul(a[3], b)}; }

#define SCHED 0
constexpr int ld_at(int j) { return (j & 1) ? j >> 1 : -1; }
#define SPOLL 16
constexpr int S_POLL = SPOLL, S_BAR = 6, S_RD = 8, S_A1 = 11, S_A2 = 16, S_A3 = 18, S_MU = 20, S_ST = 22, S_K1 = 4, S_K2 = 10, S_K3 = 14, S_K4 = 24, S_K5 = 26, S_K6 = 29, S_K7 = 30;

struct XBlk { u32x4 h[16]; };

__global__ __launch_bounds__(256, 1) void fused_kernel(Params p) {
    __shared__ __attribute__((aligned(16))) unsigned char lds[4 * 32768];
    const int lane = threadIdx.x & 63;
    const int wid = __builtin_amdgcn_readfirstlane(threadIdx.x >> 6);
    const int rank = p.rank;
    char* const me = p.ws[rank];
    const int bid = blockIdx.x;
    STAMP(0);

    char* const callp = me + WS_CALL + (size_t)(bid * 4 + wid) * 64;

    const unsigned vsrc = (lane & 15) * KDIM + (lane >> 4) * 16, vdst = lane * 16;

    if (bid < 32) {
        const int w = bid * 4 + wid, pi = w % 7, j = w / 7;
        const unsigned peer = (unsigned)(rank + 1 + pi) & 7u;
        unsigned vzero = 0;
        asm volatile("v_accvgpr_write_b32 a33, 1\n\ts_nop 1" : "+v"(vzero) :: "a0", "a31", "a32", "a33", "a34");
        if (w >= 7 * WPP) return;
        auto srcp = [&](unsigned u) { return p.x + unit_src(u < NUNIT ? u : NUNIT - 1); };
        unsigned tk0 = j, tk1 = j + WPP, prev = NUNIT;
        snd_ticket(callp, vzero);
        snd_load<0>(srcp(tk0), vsrc);
        const unsigned call = snd_wait_ticket<4>() + 1u;
        acc_set<34>(call);
        const unsigned tbase = (call - 1u) * NUNIT - 2u * WPP;
        char* const dst = p.ws[peer] + WS_X + (size_t)((call & 1u) * NR + rank) * SHARD;
        char* const fl = p.ws[peer] + WS_FLAG + rank * 2048;
        char* const ctr = me + WS_TICK + pi * 128;
        char* const dummy = me + WS_TMO + 128 + w * 4;
        auto flagp = [&](unsigned u) { return u < NUNIT ? fl + unit_flag(u) : dummy; };
#define SEND_STEP(BA, BB, NLOAD) { \
            snd_ticket(ctr, vzero); \
            snd_load<BB>(srcp(tk1), vsrc); \
            wait_vm<NLOAD>();                                            \
            snd_store<BA>(dst + tk0 * 4096u, vdst); \
            const unsigned tk2 = snd_wait_ticket<8>() - tbase;           \
            snd_flag<34>(flagp(prev), vzero); \
            prev = tk0; tk0 = tk1; tk1 = tk2; }
        SEND_STEP(0, 16, 5)
        for (;;) {
            SEND_STEP(16, 0, 10) if (tk0 >= NUNIT) break;
            SEND_STEP(0, 16, 10) if (tk0 >= NUNIT) break;
        }
#undef SEND_STEP
        wait_vm<0>();
        snd_flag<34>(flagp(prev), vzero);
        STAMP(1);
        return;
    }

    int cg, rg;
    {
        const int xcd = bid & 7, jj = (bid >> 3) - 4;
        const int i = (xcd & 1) * 28 + jj; rg = i >> 3; cg = (xcd >> 1) * 8 + (i & 7);
    }
    const int q = wid;
    const float scale = p.sx[0] * p.sw[0];
    unsigned call;
    const unsigned ldsw = (unsigned)(size_t)(LAS unsigned char*)lds + q * 32768u;
    {
        const __amdgpu_buffer_rsrc_t rw = __builtin_amdgcn_make_buffer_rsrc((void*)p.w, 0, (unsigned)KDIM * NFULL, 0x00020000);
        const unsigned ve = (lane >> 2) * NFULL + (((lane & 3) ^ ((lane >> 4) & 1)) * 16), vo = ve ^ 32u;
        const unsigned so = (unsigned)(q * 1024) * NFULL + rank * NSH + cg * 64;
        const unsigned ab = ldsw + ((lane >> 4) * 16 + ((lane & 15) >> 1)) * 64 + (lane & 1) * 8, f = (lane >> 3) & 3;
        const unsigned a0 = ab + ((0 ^ f) << 4), a1 = ab + ((1 ^ f) << 4), a2 = ab + ((2 ^ f) << 4), a3 = ab + ((3 ^ f) << 4);
        constexpr unsigned SS = 128u * NFULL;
        unsigned vzero = 0;
        asm volatile("v_accvgpr_write_b32 a33, 1\n\ts_nop 1" : "+v"(vzero) :: "a32", "a33");
        snd_ticket(callp, vzero);
#define WOFF(S) (so + (unsigned)(((S) + ROT) & 7) * SS)
#define WFIRST w_stage(rw, ve, vo, ldsw + 0 * 8192, WOFF(0)); w_stage(rw, ve, vo, ldsw + 1 * 8192, WOFF(1)); \
               w_stage(rw, ve, vo, ldsw + 2 * 8192, WOFF(2)); w_stage(rw, ve, vo, ldsw + 3 * 8192, WOFF(3));
        if (rg >= 3) {
            const unsigned uo = (unsigned)(((rg - 3) * 32 + cg) * 4 + q);
            snd_load<224>(p.x + unit_src(uo), vsrc);
            call = snd_wait_ticket<0>() + 1u;
            acc_set<240>(call);
            snd_store<224>(me + WS_X + (size_t)((call & 1u) * NR + rank) * SHARD + uo * 4096u, vdst);
            WFIRST
            wait_vm<32>();
            snd_flag<240>(me + WS_FLAG + rank * 2048 + unit_flag(uo), vzero);
            wait_vm<24>();
        } else {
            WFIRST
            call = snd_wait_ticket<24>() + 1u;
        }
#undef WFIRST
        w_read<0, 0>(a0, a1, a2, a3); w_stage(rw, ve, vo, ldsw + 0 * 8192, WOFF(4));
        wait_vm<24>(); w_read<1, 1>(a0, a1, a2, a3); w_stage(rw, ve, vo, ldsw + 1 * 8192, WOFF(5));
        wait_vm<24>(); w_read<2, 2>(a0, a1, a2, a3); w_stage(rw, ve, vo, ldsw + 2 * 8192, WOFF(6));
        wait_vm<24>(); w_read<3, 3>(a0, a1, a2, a3); w_stage(rw, ve, vo, ldsw + 3 * 8192, WOFF(7));
#undef WOFF
        wait_vm<24>(); w_read<4, 0>(a0, a1, a2, a3);
        wait_vm<16>(); w_read<5, 1>(a0, a1, a2, a3);
        wait_vm<8>();  w_read<6, 2>(a0, a1, a2, a3);
        wait_vm<0>();  w_read<7, 3>(a0, a1, a2, a3);
    }
    STAMP(1);
    const unsigned peer = (unsigned)(rank + 1 + rg) & 7u;
    constexpr int n = NBLK + 5;
    constexpr unsigned long long OWNPOS = OWNMASK;
    static_assert(__builtin_popcountll(OWNPOS) == 5 && (OWNPOS >> n) == 0, "");
    void* const xall = me + WS_X + (size_t)(call & 1u) * NR * SHARD;
    const unsigned sfl = peer * 2048 + q * 512, sflo = rank * 2048 + q * 512, vfl = lane * 8;
    __builtin_amdgcn_s_barrier();
    const unsigned orow = q * 4 + (lane >> 4), voffO = orow * (NSH * 4) + (cg * 64 + (lane & 15) * 4) * 4;
    const unsigned lwb = q * 4096 + (lane & 15) * 256 + (((lane >> 4) ^ (lane & 15)) << 4);
    LAS unsigned char* const lw0 = (LAS unsigned char*)lds + lwb; LAS unsigned char* const lw1 = (LAS unsigned char*)lds + (lwb ^ 64);
    LAS unsigned char* const lw2 = (LAS unsigned char*)lds + (lwb ^ 128); LAS unsigned char* const lw3 = (LAS unsigned char*)lds + (lwb ^ 192);
    LAS unsigned char* const lr = (LAS unsigned char*)lds + orow * 256 + (((lane & 15) ^ orow) << 4);
    int lim = NBLK;
    f32x4 cinit = {0.f, 0.f, 0.f, 0.f};
    unsigned xszn = NR * SHARD;

    auto count = [&](u32x2 f) { const unsigned long long m = ~__ballot(f.x >= call && f.y >= call); return m ? (int)(__builtin_ctzll(m) >> 1) : 2 * NBLK; };
    auto spin = [&](unsigned soff, int need) {
        const char* const pf = me + WS_FLAG + soff;
        const unsigned long long t0 = __builtin_amdgcn_s_memrealtime();
        asm volatile("" :: "s"(t0));
        for (;;) {
            u32x2 f; unsigned tmo;
            asm volatile("global_load_dwordx2 %0, %2, %3 sc0 sc1\n\tglobal_load_dword %1, %4, %5 sc1\n\ts_waitcnt vmcnt(0)"
                         : "=&v"(f), "=&v"(tmo) : "v"(vfl), "s"(pf), "v"(0u), "s"(me + WS_TMO) : "memory");
            const int c = count(f);
            if (c > need) return c;
            if (__builtin_amdgcn_readfirstlane(tmo) == call || __builtin_amdgcn_s_memrealtime() - t0 > DEADLINE_TICKS) {
                if (lane == 0) __hip_atomic_store((unsigned*)(me + WS_TMO), call, __ATOMIC_RELAXED, __HIP_MEMORY_SCOPE_AGENT);
                lim = 0; xszn = 0; const float qn = __builtin_nanf(""); cinit = (f32x4){qn, qn, qn, qn};
                asm volatile("s_nop 1" : "+v"(cinit));
                return 2 * NBLK;
            }
            __builtin_amdgcn_s_sleep(SPIN_SLEEP);
        }
    };
    unsigned voffR[8];
#pragma unroll
    for (int k = 0; k < 8; ++k) voffR[k] = vdst + q * 16384 + (unsigned)((k + ROT) & 7) * 2048u;
    f32x4 c0 = cinit, c1 = cinit, c2 = cinit, c3 = cinit;
    asm volatile("s_nop 1" : "+v"(cinit), "+v"(c0), "+v"(c1), "+v"(c2), "+v"(c3));
#define XLOAD(X, k) (X).h[k] = __builtin_bit_cast(u32x4, __builtin_amdgcn_raw_buffer_load_b128(rx, voffR[(k) >> 1], XSX + ((k) & 1) * 1024u, XAUX))
#define POLL() __builtin_bit_cast(u32x2, __builtin_amdgcn_raw_buffer_load_b64(__builtin_amdgcn_make_buffer_rsrc(me + WS_FLAG, 0, psz, 0x00020000), vfl, sfl, 17))
    XBlk X0, X1, X2;
    int rdy = 0, rdyo = 0;
    int nr = 0, no = rg, blk, have;
    unsigned sx, sxn, g0, g1, g2, g3, own, oso = 0, osz = 0, psz = 0;
#define FIRST(X, POS, G) { \
        if constexpr ((OWNPOS >> POS) & 1) { if (no >= rdyo) rdyo = spin(sflo, no); G = rank * NBLK + no; no += 7; } \
        else { if (nr >= rdy) rdy = spin(sfl, nr); G = peer * NBLK + nr; nr += 1; } \
        const __amdgpu_buffer_rsrc_t rx = __builtin_amdgcn_make_buffer_rsrc(xall, 0, xszn, 0x00020000); \
        sx = G * 65536u; \
        _Pragma("unroll") for (int k = 0; k < 16; ++k) XLOAD(X, k); }
    FIRST(X2, 0, g0) FIRST(X0, 1, g1)
#define GL g2
#undef FIRST
    if constexpr ((OWNPOS >> DIST) & 1) { own = 1; blk = no; have = rdyo; GL = rank * NBLK + no; no += 7; } else { own = 0; blk = nr; have = rdy; GL = peer * NBLK + nr; nr += 1; }
    sxn = GL * 65536u;
    auto store_rows = [&](const f32x4& r) {
        const __amdgpu_buffer_rsrc_t ro = __builtin_amdgcn_make_buffer_rsrc((void*)p.out, 0, OSZ, 0x00020000);
        __builtin_amdgcn_raw_buffer_store_b128(__builtin_bit_cast(u32x4, r), ro, voffO, oso, OUT_AUX);
    };
    auto store_at = [&](const f32x4& r, unsigned o, unsigned z) {
        __builtin_amdgcn_raw_buffer_store_b128(__builtin_bit_cast(u32x4, r), __builtin_amdgcn_make_buffer_rsrc((void*)p.out, 0, z, 0x00020000), voffO, o, OUT_AUX);
    };
    f32x4 h0 = cinit, h1 = cinit; unsigned ho0 = 0, ho1 = 0, hz0 = 0, hz1 = 0;
#define SB __builtin_amdgcn_sched_barrier(0);
#define XFRAG(X, S) __builtin_shufflevector((X).h[2 * (S)], (X).h[2 * (S) + 1], 0, 1, 2, 3, 4, 5, 6, 7)
#define PIN(v) asm volatile("" : "+s"(v));
#define SLOT(CUR, NXT, PP, PC, PO, PN, J, C) \
        mf<(J) / 4, C>(c##C, XFRAG(CUR, (J) / 4), cinit); SB \
        if constexpr ((J) == 0 && !(ABL & 4)) *(LAS f32x4*)(lw2 + PP * 16384) = c2; \
        if constexpr ((J) == 1 && !(ABL & 4)) *(LAS f32x4*)(lw3 + PP * 16384) = c3; \
        if constexpr ((J) == 2 && !(ABL & 4)) *(LAS f32x4*)(lw0 + PP * 16384) = c0; \
        if constexpr ((J) == 31 && !(ABL & 4)) *(LAS f32x4*)(lw1 + PC * 16384) = c1; \
        if constexpr ((J) == S_POLL && !(ABL & 16)) PN = POLL(); \
        if constexpr (ld_at(J) >= 0) { if constexpr (DIST == 2) XLOAD(NXT, ld_at(J) & 15); else XLOAD(CUR, ld_at(J) & 15); } \
        if constexpr (DIST == 3 && (J) == 31) XLOAD(CUR, 15); \
        if constexpr ((J) == S_BAR) { if (!(ABL & 2)) wait_lgkm0(); if (!(ABL & 1)) __builtin_amdgcn_s_barrier(); asm volatile("" ::: "memory"); } \
        if constexpr ((J) == S_RD) { r0 = *(const LAS f32x4*)(lr + PP * 16384); r1 = *(const LAS f32x4*)(lr + PP * 16384 + 4096); } \
        if constexpr ((J) == S_A1 && !(ABL & 8)) r1 = fadd4(r0, r1); \
        if constexpr ((J) == S_A1 + 1) { r2 = *(const LAS f32x4*)(lr + PP * 16384 + 8192); r3 = *(const LAS f32x4*)(lr + PP * 16384 + 12288); } \
        if constexpr ((J) == S_A2 && !(ABL & 8)) r2 = fadd4(r2, r3); \
        if constexpr ((J) == S_A3 && !(ABL & 8)) r1 = fadd4(r1, r2); \
        if constexpr ((J) == S_MU && !(ABL & 8)) r1 = fmul4(r1, scale); \
        if constexpr ((J) == S_ST) { \
            if constexpr (!BATCH3) store_rows(r1); \
            else if constexpr (PP == 2) { h0 = r1; ho0 = oso; hz0 = osz; } \
            else if constexpr (PP == 0) { h1 = r1; ho1 = oso; hz1 = osz; } \
            else store_rows(r1); } \
        if constexpr (BATCH3 && PP == 1 && (J) == S_ST + 2) store_at(h0, ho0, hz0); \
        if constexpr (BATCH3 && PP == 1 && (J) == S_ST + 4) store_at(h1, ho1, hz1); \
        if constexpr ((J) == S_K1) { PIN(i) own = (unsigned)(OWNPOS >> (i + DIST + 1)) & 1u; PIN(own) } \
        if constexpr ((J) == S_K2) { PIN(own) blk = own ? no : nr; src = own ? rank : peer; int no7 = no + 7, nr1 = nr + 1; PIN(no7) PIN(nr1) no = own ? no7 : no; nr = own ? nr : nr1; PIN(blk) PIN(src) PIN(no) PIN(nr) } \
        if constexpr ((J) == S_K3) { PIN(blk) PIN(src) sxn = (src * NBLK + (blk & 31)) * 65536u; xszn = blk < lim ? NR * SHARD : 0u; PIN(sxn) PIN(xszn) } \
        if constexpr ((J) == S_K4) { PIN(g0) oso = g0 * 131072u; osz = (int)g0 < 0 ? 0u : (unsigned)(NR * MSH) * NSH * 4u; PIN(oso) PIN(osz) } \
        if constexpr ((J) == S_K5) { PIN(sxn) PIN(blk) g0 = g1; g1 = g2; if constexpr (DIST == 3) g2 = g3; GL = (sxn >> 16) | (blk < NBLK ? 0u : 0x80000000u); PIN(GL) } \
        if constexpr ((J) == S_K6 && !(ABL & 16)) {     \
            if (nr > rdy) { asm volatile("" : "+v"(PN)); const int c = count(PN); rdy = c > rdy ? c : rdy; } \
            else if (pszo) { asm volatile("" : "+v"(PO)); const int c = count(PO); rdy = c > rdy ? c : rdy; } \
            pszo = psz; PIN(rdy) PIN(pszo) } \
        if constexpr ((J) == S_K7) { PIN(own) PIN(blk) have = own ? rdyo : rdy; PIN(have) have = blk < NBLK ? have : 2 * NBLK; psz = rdy - nr < POLL_AHEAD ? NR * 2048 : 0u; BSTAMP(i); ++i; PIN(have) PIN(psz) PIN(i) } \
        SB
#define STEP4(CUR, NXT, PP, PC, PO, PN, S) SLOT(CUR, NXT, PP, PC, PO, PN, 4 * S, 1) SLOT(CUR, NXT, PP, PC, PO, PN, 4 * S + 1, 2) SLOT(CUR, NXT, PP, PC, PO, PN, 4 * S + 2, 3) SLOT(CUR, NXT, PP, PC, PO, PN, 4 * S + 3, 0)
#define BODY(CUR, NXT, PP, PC, PO, PN) { \
        if (blk >= have) { const int c = spin(own ? sflo : sfl, blk); rdyo = own ? c : rdyo; rdy = own ? rdy : c; } \
        if (blk >= 28 && blk < 32 && !own) STAMP(40 + blk - 28); \
        sx = sxn; \
        const __amdgpu_buffer_rsrc_t rx = __builtin_amdgcn_make_buffer_rsrc(xall, 0, xszn, 0x00020000); \
        f32x4 r0, r1, r2, r3; unsigned src; \
        STEP4(CUR, NXT, PP, PC, PO, PN, 0) STEP4(CUR, NXT, PP, PC, PO, PN, 1) STEP4(CUR, NXT, PP, PC, PO, PN, 2) STEP4(CUR, NXT, PP, PC, PO, PN, 3) \
        STEP4(CUR, NXT, PP, PC, PO, PN, 4) STEP4(CUR, NXT, PP, PC, PO, PN, 5) STEP4(CUR, NXT, PP, PC, PO, PN, 6) STEP4(CUR, NXT, PP, PC, PO, PN, 7) }

    static_assert((n - 1) % 3 == 0, "");
    int i = 0; u32x2 P0 = {0u, 0u}, P1, P2; unsigned pszo = 0;
    BODY(X2, X1, 1, 2, P0, P2)
    do { BODY(X0, X2, 2, 0, P2, P0) BODY(X1, X0, 0, 1, P0, P1) BODY(X2, X1, 1, 2, P1, P2) } while (i < n);
    {
        asm volatile("s_nop 7\n\ts_nop 7" : "+v"(c0), "+v"(c2), "+v"(c3));
        *(LAS f32x4*)(lw2 + 2 * 16384) = c2; *(LAS f32x4*)(lw3 + 2 * 16384) = c3; *(LAS f32x4*)(lw0 + 2 * 16384) = c0;
        wait_lgkm0(); __builtin_amdgcn_s_barrier(); asm volatile("" ::: "memory");
        store_rows(fmul4(fadd4(fadd4(*(const LAS f32x4*)(lr + 2 * 16384), *(const LAS f32x4*)(lr + 2 * 16384 + 4096)), fadd4(*(const LAS f32x4*)(lr + 2 * 16384 + 8192), *(const LAS f32x4*)(lr + 2 * 16384 + 12288))), scale));
        STAMP(2 + n);
    }
#undef BODY
#undef STEP4
#undef SLOT
#undef XFRAG
#undef SB
#undef XLOAD
#undef POLL
#undef PIN
}
}

extern "C" void kernel_launch(int rank, int nranks, ncclComm_t comm,
                              hipStream_t stream,
                              void* const* d_in, const int* in_sizes, int n_in,
                              void* d_out, int out_size,
                              const int* in_dtypes, int out_dtype,
                              void* const* d_out_all,
                              void* const* d_ws_all, size_t ws_bytes) {
    (void)comm; (void)in_dtypes; (void)out_dtype; (void)d_out_all;
    if (nranks != NR || (unsigned)rank >= (unsigned)NR || n_in != 4 || ws_bytes < WS_END ||
        in_sizes[0] != MSH * KDIM || in_sizes[1] != KDIM * NFULL || out_size != NR * MSH * NSH) {
        fprintf(stderr, "kernel_launch: unexpected problem shape\n");
        return;
    }
    Params p;
    p.x = (const unsigned char*)d_in[0]; p.w = (const unsigned char*)d_in[1];
    p.sx = (const float*)d_in[2]; p.sw = (const float*)d_in[3]; p.out = (float*)d_out;
    for (int i = 0; i < NR; ++i) p.ws[i] = (char*)d_ws_all[i];
    p.rank = rank; p.pad = 0;
    fused_kernel<<<256, 256, 0, stream>>>(p);
    HIP_CHECK(hipGetLastError());
}
```

```cpp
#include <hip/hip_runtime.h>
#include <rccl/rccl.h>
#include <cstdio>

#define HIP_CHECK(expr) do { hipError_t _e = (expr); if (_e != hipSuccess) { \
    fprintf(stderr, "HIP error %s at %s:%d\n", hipGetErrorString(_e), __FILE__, __LINE__); \
    return; } } while (0)

namespace {
typedef unsigned u32x2 __attribute__((ext_vector_type(2)));
typedef unsigned u32x4 __attribute__((ext_vector_type(4)));
typedef unsigned u32x8 __attribute__((ext_vector_type(8)));
typedef float f32x4 __attribute__((ext_vector_type(4)));
#define LAS __attribute__((address_space(3)))
#define DI __device__ __forceinline__

constexpr int NR = 8;
constexpr int KDIM = 4096, NFULL = 16384, NSH = 2048, MSH = 512;
constexpr unsigned SHARD = MSH * KDIM;
constexpr int NBLK = MSH / 16;
constexpr int NUNIT = 512;
constexpr int WPP = 16;
constexpr size_t WS_X = 0;
constexpr size_t WS_FLAG = 2ull * NR * SHARD;
constexpr size_t WS_TICK = WS_FLAG + NR * 2048;
constexpr size_t WS_TMO = WS_TICK + 1024;
constexpr size_t WS_CALL = WS_TMO + 1024;
constexpr size_t WS_DBG = WS_CALL + 1024 * 64;
constexpr size_t WS_END = WS_DBG + 2 * 256 * 64 * 8;
constexpr unsigned long long DEADLINE_TICKS = 30000000ull;

#define OWNMASK 0x2aaull
#define POLL_AHEAD 3
#define SPOLL 16
#define OUT_AUX 2

struct Params {
    const unsigned char* x; const unsigned char* w; const float* sx; const float* sw; float* out;
    char* wsr[NR];
    int rank; int slow;
};

#define ACLOB "a0","a1","a2","a3","a4","a5","a6","a7","a8","a9","a10","a11","a12","a13","a14","a15","a16","a17","a18","a19","a20","a21","a22","a23","a24","a25","a26","a27","a28","a29","a30","a31","a32","a33","a34","a35","a36","a37","a38","a39","a40","a41","a42","a43","a44","a45","a46","a47","a48","a49","a50","a51","a52","a53","a54","a55","a56","a57","a58","a59","a60","a61","a62","a63","a64","a65","a66","a67","a68","a69","a70","a71","a72","a73","a74","a75","a76","a77","a78","a79","a80","a81","a82","a83","a84","a85","a86","a87","a88","a89","a90","a91","a92","a93","a94","a95","a96","a97","a98","a99","a100","a101","a102","a103","a104","a105","a106","a107","a108","a109","a110","a111","a112","a113","a114","a115","a116","a117","a118","a119","a120","a121","a122","a123","a124","a125","a126","a127","a128","a129","a130","a131","a132","a133","a134","a135","a136","a137","a138","a139","a140","a141","a142","a143","a144","a145","a146","a147","a148","a149","a150","a151","a152","a153","a154","a155","a156","a157","a158","a159","a160","a161","a162","a163","a164","a165","a166","a167","a168","a169","a170","a171","a172","a173","a174","a175","a176","a177","a178","a179","a180","a181","a182","a183","a184","a185","a186","a187","a188","a189","a190","a191","a192","a193","a194","a195","a196","a197","a198","a199","a200","a201","a202","a203","a204","a205","a206","a207","a208","a209","a210","a211","a212","a213","a214","a215","a216","a217","a218","a219","a220","a221","a222","a223","a224","a225","a226","a227","a228","a229","a230","a231","a232","a233","a234","a235","a236","a237","a238","a239","a240","a241","a242","a243","a244","a245","a246","a247","a248","a249","a250","a251","a252","a253","a254","a255"

template <int N> DI void wait_vm() { asm volatile("s_waitcnt vmcnt(%0)" :: "n"(N) : "memory"); }
DI void wait_lgkm0() { asm volatile("s_waitcnt lgkmcnt(0)" ::: "memory"); }
template <int B> DI void snd_load(const unsigned char* base, unsigned voff) {
    asm volatile("global_load_dwordx4 a[%c2:%c2+3], %0, %1\n\t"
                 "global_load_dwordx4 a[%c2+4:%c2+7], %0, %1 offset:64\n\t"
                 "global_load_dwordx4 a[%c2+8:%c2+11], %0, %1 offset:128\n\t"
                 "global_load_dwordx4 a[%c2+12:%c2+15], %0, %1 offset:192" :: "v"(voff), "s"(base), "i"(B) : "memory");
}
template <int B> DI void snd_store(char* base, unsigned voff) {
    asm volatile("global_store_dwordx4 %0, a[%c2:%c2+3], %1 sc0 sc1\n\t"
                 "global_store_dwordx4 %0, a[%c2+4:%c2+7], %1 offset:1024 sc0 sc1\n\t"
                 "global_store_dwordx4 %0, a[%c2+8:%c2+11], %1 offset:2048 sc0 sc1\n\t"
                 "global_store_dwordx4 %0, a[%c2+12:%c2+15], %1 offset:3072 sc0 sc1\n\ts_nop 1" :: "v"(voff), "s"(base), "i"(B) : "memory");
}
template <int A> DI void snd_ticket(char* ctr, unsigned vzero) {
    unsigned long long keep;
    asm volatile("s_mov_b64 %0, exec\n\ts_mov_b64 exec, 1\n\tglobal_atomic_add a[%c3], %1, a33, %2 sc0\n\ts_mov_b64 exec, %0"
                 : "=&s"(keep) : "v"(vzero), "s"(ctr), "i"(A) : "memory");
}
DI void code_prefetch(unsigned lane) {
    unsigned long long pc; unsigned t0, t1, t2;
    asm volatile("s_getpc_b64 %0\n"
                 ".Lfk_code_begin:\n\t"
                 "v_min_u32 %1, .Lfk_code_end-.Lfk_code_begin, %4\n\tv_min_u32 %2, .Lfk_code_end-.Lfk_code_begin, %5\n\tv_min_u32 %3, .Lfk_code_end-.Lfk_code_begin, %6\n\t"
                 "global_load_dword a252, %1, %0\n\tglobal_load_dword a253, %2, %0\n\tglobal_load_dword a254, %3, %0"
                 : "=&s"(pc), "=&v"(t0), "=&v"(t1), "=&v"(t2) : "v"(lane * 128u), "v"(lane * 128u + 8192u), "v"(lane * 128u + 16384u) : "memory", "a252", "a253", "a254");
}
template <int A> DI void snd_flag(char* flag, unsigned vzero) {
    unsigned long long keep;
    asm volatile("s_mov_b64 %0, exec\n\ts_mov_b64 exec, 1\n\tglobal_store_dword %1, a[%c3], %2 sc0 sc1\n\ts_mov_b64 exec, %0"
                 : "=&s"(keep) : "v"(vzero), "s"(flag), "i"(A) : "memory");
}
template <int A> DI void acc_set(unsigned v) { asm volatile("s_nop 1\n\tv_accvgpr_write_b32 a[%c1], %0\n\ts_nop 1" :: "s"(v), "i"(A)); }
DI unsigned unit_src(unsigned u) { return (u >> 4) * 65536u + (u & 15u) * 256u; }
DI unsigned unit_flag(unsigned u) { return (((u >> 2) & 3u) * 128u + (u >> 4) * 4u + (u & 3u)) * 4u; }
template <int N, int A = 32> DI unsigned snd_wait_ticket() {
    unsigned v;
    asm volatile("s_waitcnt vmcnt(%1)\n\tv_accvgpr_read_b32 %0, a[%c2]\n\ts_nop 1" : "=v"(v) : "n"(N), "i"(A) : "memory");
    return __builtin_amdgcn_readfirstlane(v);
}

#define STAMP(k) do { } while (0)
#define BSTAMP(i) do { } while (0)

template <int S, int SLOT> DI void w_read(unsigned a0, unsigned a1, unsigned a2, unsigned a3) {
    constexpr int B = S * 32, O = SLOT * 8192;
#define TRR(c, t, A) "ds_read_b64_tr_b8 a[%c4+" #c "*8+" #t "*2:%c4+" #c "*8+" #t "*2+1], " A " offset:%c5+4096*(" #t "/2)+512*(" #t "%%2)\n\t"
    asm volatile(TRR(0, 0, "%0") TRR(0, 1, "%0") TRR(0, 2, "%0") TRR(0, 3, "%0")
                 TRR(1, 0, "%1") TRR(1, 1, "%1") TRR(1, 2, "%1") TRR(1, 3, "%1")
                 TRR(2, 0, "%2") TRR(2, 1, "%2") TRR(2, 2, "%2") TRR(2, 3, "%2")
                 TRR(3, 0, "%3") TRR(3, 1, "%3") TRR(3, 2, "%3") TRR(3, 3, "%3")
                 "s_waitcnt lgkmcnt(0)"
                 :: "v"(a0), "v"(a1), "v"(a2), "v"(a3), "i"(B), "i"(O) : "memory", ACLOB);
#undef TRR
}
DI void w_stage(__amdgpu_buffer_rsrc_t rs, unsigned ve, unsigned vo, unsigned m0base, unsigned soff) {
    unsigned keep, so;
#define WST(V) "s_nop 0\n\tbuffer_load_dwordx4 " V ", %5, %1 offen lds\n\t"
#define WNX "s_add_u32 m0, m0, 0x400\n\ts_add_u32 %1, %1, 0x40000\n\t"
    asm volatile("s_mov_b32 %0, m0\n\ts_mov_b32 m0, %2\n\ts_mov_b32 %1, %6\n\t"
                 WST("%3") WNX WST("%4") WNX WST("%3") WNX WST("%4") WNX WST("%3") WNX WST("%4") WNX WST("%3") WNX WST("%4")
                 "s_mov_b32 m0, %0"
                 : "=&s"(keep), "=&s"(so) : "s"(m0base), "v"(ve), "v"(vo), "s"(rs), "s"(soff) : "scc", "memory");
#undef WST
#undef WNX
    static_assert(16 * NFULL == 0x40000, "");
}
template <int S, int C> DI void mf(f32x4& c, const u32x8& x, const f32x4& cinit) {
    constexpr int B = S * 32 + C * 8;
    if constexpr (S == 0) asm volatile("v_mfma_f32_16x16x128_f8f6f4 %0, a[%c3:%c3+7], %1, %2" : "+v"(c) : "v"(x), "v"(cinit), "i"(B));
    else asm volatile("v_mfma_f32_16x16x128_f8f6f4 %0, a[%c2:%c2+7], %1, %0" : "+v"(c) : "v"(x), "i"(B));
}
DI float fadd(float a, float b) { float r; asm("v_add_f32_e32 %0, %1, %2" : "=v"(r) : "v"(a), "v"(b)); return r; }
DI float fmul(float a, float b) { float r; asm("v_mul_f32_e32 %0, %1, %2" : "=v"(r) : "v"(a), "v"(b)); return r; }
DI f32x4 fadd4(f32x4 a, f32x4 b) { return (f32x4){fadd(a[0], b[0]), fadd(a[1], b[1]), fadd(a[2], b[2]), fadd(a[3], b[3])}; }
DI f32x4 fmul4(f32x4 a, float b) { return (f32x4){fmul(a[0], b), fmul(a[1], b), fmul(a[2], b), fmul(a[3], b)}; }

constexpr int S_POLL = SPOLL, S_BAR = 6, S_RD = 8, S_A1 = 11, S_A2 = 16, S_A3 = 18, S_MU = 20, S_ST = 22, S_K1 = 4, S_K2 = 10, S_K3 = 14, S_K4 = 24, S_K5 = 26, S_K6 = 29, S_K7 = 30;

struct XBlk { u32x4 h[16]; };

__global__ __launch_bounds__(256, 1) void fused_kernel(Params p) {
    __shared__ __attribute__((aligned(16))) unsigned char lds[4 * 32768];
    const int lane = threadIdx.x & 63;
    const int wid = __builtin_amdgcn_readfirstlane(threadIdx.x >> 6);
    const int rank = p.rank;
    char* const me = p.wsr[7];
    const int bid = blockIdx.x;
    code_prefetch(lane);
    const bool sender = bid < 24 || (bid < 32 && (bid & 2));
    const int w = (bid < 24 ? bid : 24 + (bid & 1) + ((bid >> 1) & 2)) * 4 + wid, pi = w % 7;
    char* const pws = p.wsr[pi];
    asm volatile("" :: "s"(p.x), "s"(p.w), "s"(me), "s"(rank), "s"(pws));
    STAMP(0);

    char* const callp = me + WS_CALL + (size_t)(bid * 4 + wid) * 64;

    const unsigned vsrc = (lane & 15) * KDIM + (lane >> 4) * 16, vdst = lane * 16;

    if (__builtin_expect(sender, 1)) {
        const int j = w / 7;
        unsigned vzero = 0;
        asm volatile("v_accvgpr_write_b32 a33, 1" : "+v"(vzero) :: "a0", "a31", "a32", "a33", "a34", "a35");
        auto srcp = [&](unsigned u) { return p.x + unit_src(u < NUNIT ? u : NUNIT - 1); };
        unsigned tk0 = j, tk1 = j + WPP, prev;
        char* const ctr = me + WS_TICK + pi * 128;
        snd_load<0>(srcp(tk0), vsrc);
        snd_ticket<35>(callp, vzero);
        snd_ticket<32>(ctr, vzero);
        snd_load<16>(srcp(tk1), vsrc);
        const unsigned call = snd_wait_ticket<5, 35>() + 1u;
        char* const dst = pws + WS_X + (size_t)((call & 1u) * NR + rank) * SHARD;
        snd_store<0>(dst + tk0 * 4096u, vdst);
        acc_set<34>(call);
        const unsigned tbase = (call - 1u) * NUNIT - 2u * WPP;
        char* const fl = pws + WS_FLAG + rank * 2048;
        prev = tk0; tk0 = tk1; tk1 = snd_wait_ticket<8>() - tbase;
#define SEND_STEP(BA, BB, NLOAD) { \
            snd_ticket<32>(ctr, vzero); \
            snd_load<BB>(srcp(tk1), vsrc); \
            wait_vm<NLOAD>();                                            \
            snd_store<BA>(dst + tk0 * 4096u, vdst); \
            const unsigned tk2 = snd_wait_ticket<8>() - tbase;           \
            snd_flag<34>(fl + unit_flag(prev), vzero); \
            prev = tk0; tk0 = tk1; tk1 = tk2; }
        SEND_STEP(16, 0, 9)
        if (tk0 < NUNIT)
        for (;;) {
            SEND_STEP(0, 16, 10) if (tk0 >= NUNIT) break;
            SEND_STEP(16, 0, 10) if (tk0 >= NUNIT) break;
        }
#undef SEND_STEP
        wait_vm<0>();
        snd_flag<34>(fl + unit_flag(prev), vzero);
        STAMP(1);
        return;
    }

    int cg, rg;
    {
        const int xcd = bid & 7, i = (xcd & 1) * 28 + (bid >> 3) - 4;
        rg = bid < 32 ? 7 : i >> 3;
        cg = bid < 32 ? p.slow + (xcd & 1) + (xcd >> 2) * 16 : (xcd >> 1) * 8 + (i & 7);
    }
    const int q = wid;
    const float scale = p.sx[0] * p.sw[0];
    unsigned call;
    const unsigned ldsw = (unsigned)(size_t)(LAS unsigned char*)lds + q * 32768u;
    {
        const __amdgpu_buffer_rsrc_t rw = __builtin_amdgcn_make_buffer_rsrc((void*)p.w, 0, (unsigned)KDIM * NFULL, 0x00020000);
        const unsigned ve = (lane >> 2) * NFULL + (((lane & 3) ^ ((lane >> 4) & 1)) * 16), vo = ve ^ 32u;
        const unsigned so = (unsigned)(q * 1024) * NFULL + rank * NSH + cg * 64;
        const unsigned ab = ldsw + ((lane >> 4) * 16 + ((lane & 15) >> 1)) * 64 + (lane & 1) * 8, f = (lane >> 3) & 3;
        const unsigned a0 = ab + ((0 ^ f) << 4), a1 = ab + ((1 ^ f) << 4), a2 = ab + ((2 ^ f) << 4), a3 = ab + ((3 ^ f) << 4);
        constexpr unsigned SS = 128u * NFULL;
        unsigned vzero = 0;
        asm volatile("v_accvgpr_write_b32 a33, 1\n\ts_nop 1" : "+v"(vzero) :: "a32", "a33");
        snd_ticket<32>(callp, vzero);
#define WOFF(S) (so + (unsigned)(((S) + rg) & 7) * SS)
#define WFIRST w_stage(rw, ve, vo, ldsw + 0 * 8192, WOFF(0)); w_stage(rw, ve, vo, ldsw + 1 * 8192, WOFF(1)); \
               w_stage(rw, ve, vo, ldsw + 2 * 8192, WOFF(2)); w_stage(rw, ve, vo, ldsw + 3 * 8192, WOFF(3));
        if (rg >= 3 && rg < 7) {
            const unsigned uo = (unsigned)(((rg - 3) * 32 + cg) * 4 + q);
            snd_load<224>(p.x + unit_src(uo), vsrc);
            call = snd_wait_ticket<0>() + 1u;
            acc_set<240>(call);
            snd_store<224>(me + WS_X + (size_t)((call & 1u) * NR + rank) * SHARD + uo * 4096u, vdst);
            WFIRST
            wait_vm<32>();
            snd_flag<240>(me + WS_FLAG + rank * 2048 + unit_flag(uo), vzero);
            wait_vm<24>();
        } else {
            WFIRST
            call = snd_wait_ticket<24>() + 1u;
        }
#undef WFIRST
        w_read<0, 0>(a0, a1, a2, a3); w_stage(rw, ve, vo, ldsw + 0 * 8192, WOFF(4));
        wait_vm<24>(); w_read<1, 1>(a0, a1, a2, a3); w_stage(rw, ve, vo, ldsw + 1 * 8192, WOFF(5));
        wait_vm<24>(); w_read<2, 2>(a0, a1, a2, a3); w_stage(rw, ve, vo, ldsw + 2 * 8192, WOFF(6));
        wait_vm<24>(); w_read<3, 3>(a0, a1, a2, a3); w_stage(rw, ve, vo, ldsw + 3 * 8192, WOFF(7));
#undef WOFF
        wait_vm<24>(); w_read<4, 0>(a0, a1, a2, a3);
        wait_vm<16>(); w_read<5, 1>(a0, a1, a2, a3);
        wait_vm<8>();  w_read<6, 2>(a0, a1, a2, a3);
        wait_vm<0>();  w_read<7, 3>(a0, a1, a2, a3);
    }
    STAMP(1);
    const unsigned peer = (unsigned)(rank + 1 + rg) & 7u;
    static_assert(__builtin_popcountll(OWNMASK) == 5 && (OWNMASK >> 32) == 0, "");
    const bool slowcg = (cg & 14) == p.slow;
    const int n = rg == 7 ? NBLK - 7 : slowcg ? NBLK + 1 : NBLK + 5, ostep = rg == 7 ? 1 : 7;
    const unsigned long long ownpos = (unsigned)__builtin_amdgcn_readfirstlane(rg == 7 ? ~0u : slowcg ? (unsigned)(OWNMASK & -OWNMASK) : (unsigned)OWNMASK);
    void* const xall = me + WS_X + (size_t)(call & 1u) * NR * SHARD;
    const unsigned sfl = peer * 2048 + q * 512, sflo = rank * 2048 + q * 512, vfl = lane * 8;
    __builtin_amdgcn_s_barrier();
    const unsigned orow = q * 4 + (lane >> 4), voffO = orow * (NSH * 4) + (cg * 64 + (lane & 15) * 4) * 4;
    const unsigned lwb = q * 4096 + (lane & 15) * 256 + (((lane >> 4) ^ (lane & 15)) << 4);
    LAS unsigned char* const lw0 = (LAS unsigned char*)lds + lwb; LAS unsigned char* const lw1 = (LAS unsigned char*)lds + (lwb ^ 64);
    LAS unsigned char* const lw2 = (LAS unsigned char*)lds + (lwb ^ 128); LAS unsigned char* const lw3 = (LAS unsigned char*)lds + (lwb ^ 192);
    LAS unsigned char* const lr = (LAS unsigned char*)lds + orow * 256 + (((lane & 15) ^ orow) << 4);
    int lim = NBLK;
    f32x4 cinit = {0.f, 0.f, 0.f, 0.f};
    unsigned xszn = NR * SHARD;

    auto count = [&](u32x2 f) { const unsigned long long m = ~__ballot(f.x >= call && f.y >= call); return m ? (int)(__builtin_ctzll(m) >> 1) : 2 * NBLK; };
    auto spin = [&](unsigned soff, int need) {
        const char* const pf = me + WS_FLAG + soff;
        const unsigned long long t0 = __builtin_amdgcn_s_memrealtime();
        asm volatile("" :: "s"(t0));
        for (;;) {
            u32x2 f; unsigned tmo;
            asm volatile("global_load_dwordx2 %0, %2, %3 sc0 sc1\n\tglobal_load_dword %1, %4, %5 sc1\n\ts_waitcnt vmcnt(0)"
                         : "=&v"(f), "=&v"(tmo) : "v"(vfl), "s"(pf), "v"(0u), "s"(me + WS_TMO) : "memory");
            const int c = count(f);
            if (c > need) return c;
            if (__builtin_amdgcn_readfirstlane(tmo) == call || __builtin_amdgcn_s_memrealtime() - t0 > DEADLINE_TICKS) {
                if (lane == 0) __hip_atomic_store((unsigned*)(me + WS_TMO), call, __ATOMIC_RELAXED, __HIP_MEMORY_SCOPE_AGENT);
                lim = 0; xszn = 0; const float qn = __builtin_nanf(""); cinit = (f32x4){qn, qn, qn, qn};
                asm volatile("s_nop 1" : "+v"(cinit));
                return 2 * NBLK;
            }
            __builtin_amdgcn_s_sleep(1);
        }
    };
    unsigned voffR[8];
#pragma unroll
    for (int k = 0; k < 8; ++k) voffR[k] = vdst + q * 16384 + (unsigned)((k + rg) & 7) * 2048u;
    f32x4 c0 = cinit, c1 = cinit, c2 = cinit, c3 = cinit;
    asm volatile("s_nop 1" : "+v"(cinit), "+v"(c0), "+v"(c1), "+v"(c2), "+v"(c3));
#define XLOAD(X, k) (X).h[k] = __builtin_bit_cast(u32x4, __builtin_amdgcn_raw_buffer_load_b128(rx, voffR[(k) >> 1], sx + ((k) & 1) * 1024u, 17))
#define POLL() __builtin_bit_cast(u32x2, __builtin_amdgcn_raw_buffer_load_b64(__builtin_amdgcn_make_buffer_rsrc(me + WS_FLAG, 0, psz, 0x00020000), vfl, sfl, 17))
    XBlk X0, X1;
    int rdy = rg == 7 ? 2 * NBLK : 0, rdyo = 0;
    int nr = 0, no = rg, blk, have;
    unsigned sx, sxn, g0, g1, own, oso = 0, osz = 0, psz = 0, pszo = 0;
    {
        own = (unsigned)ownpos & 1u; blk = own ? no : nr;
        const int c = spin(own ? sflo : sfl, blk); rdyo = own ? c : rdyo; rdy = own ? rdy : c;
        g0 = (own ? rank : peer) * NBLK + blk; no += own ? ostep : 0; nr += own ? 0 : 1;
        const __amdgpu_buffer_rsrc_t rx = __builtin_amdgcn_make_buffer_rsrc(xall, 0, xszn, 0x00020000);
        sx = g0 * 65536u;
#pragma unroll
        for (int k = 0; k < 16; ++k) XLOAD(X1, k);
        own = (unsigned)(ownpos >> 1) & 1u; blk = own ? no : nr; have = own ? rdyo : rdy;
        g1 = (own ? rank : peer) * NBLK + blk; no += own ? ostep : 0; nr += own ? 0 : 1;
        sxn = g1 * 65536u;
        nr = __builtin_amdgcn_readfirstlane(nr); no = __builtin_amdgcn_readfirstlane(no); g1 = __builtin_amdgcn_readfirstlane(g1); sxn = __builtin_amdgcn_readfirstlane(sxn);
    }
    auto store_rows = [&](const f32x4& r) {
        __builtin_amdgcn_raw_buffer_store_b128(__builtin_bit_cast(u32x4, r), __builtin_amdgcn_make_buffer_rsrc((void*)p.out, 0, osz, 0x00020000), voffO, oso, OUT_AUX);
    };
#define SB __builtin_amdgcn_sched_barrier(0);
#define XFRAG(X, S) __builtin_shufflevector((X).h[2 * (S)], (X).h[2 * (S) + 1], 0, 1, 2, 3, 4, 5, 6, 7)
#define PIN(v) asm volatile("" : "+s"(v));
#define SLOT(CUR, NXT, PP, PO, PN, J, C) \
        mf<(J) / 4, C>(c##C, XFRAG(CUR, (J) / 4), cinit); SB \
        if constexpr ((J) == 0) *(LAS f32x4*)(lw2 + PP * 16384) = c2; \
        if constexpr ((J) == 1) *(LAS f32x4*)(lw3 + PP * 16384) = c3; \
        if constexpr ((J) == 2) *(LAS f32x4*)(lw0 + PP * 16384) = c0; \
        if constexpr ((J) == 31) *(LAS f32x4*)(lw1 + (1 - PP) * 16384) = c1; \
        if constexpr ((J) == S_POLL) PN = POLL(); \
        if constexpr ((J) & 1) XLOAD(NXT, (J) >> 1); \
        if constexpr ((J) == S_BAR) { wait_lgkm0(); __builtin_amdgcn_s_barrier(); asm volatile("" ::: "memory"); } \
        if constexpr ((J) == S_RD) { r0 = *(const LAS f32x4*)(lr + PP * 16384); r1 = *(const LAS f32x4*)(lr + PP * 16384 + 4096); \
                                     r2 = *(const LAS f32x4*)(lr + PP * 16384 + 8192); r3 = *(const LAS f32x4*)(lr + PP * 16384 + 12288); } \
        if constexpr ((J) == S_A1) r1 = fadd4(r0, r1); \
        if constexpr ((J) == S_A2) r2 = fadd4(r2, r3); \
        if constexpr ((J) == S_A3) r1 = fadd4(r1, r2); \
        if constexpr ((J) == S_MU) r1 = fmul4(r1, scale); \
        if constexpr ((J) == S_ST) store_rows(r1); \
        if constexpr ((J) == S_K1) { PIN(i) own = (unsigned)(ownpos >> (i + 2)) & 1u; PIN(own) } \
        if constexpr ((J) == S_K2) { PIN(own) blk = own ? no : nr; src = own ? rank : peer; int no7 = no + ostep, nr1 = nr + 1; PIN(no7) PIN(nr1) no = own ? no7 : no; nr = own ? nr : nr1; PIN(blk) PIN(src) PIN(no) PIN(nr) } \
        if constexpr ((J) == S_K3) { PIN(blk) PIN(src) sxn = (src * NBLK + (blk & 31)) * 65536u; xszn = blk < lim ? NR * SHARD : 0u; PIN(sxn) PIN(xszn) } \
        if constexpr ((J) == S_K4) { PIN(g0) oso = g0 * 131072u; osz = (int)g0 < 0 ? 0u : (unsigned)(NR * MSH) * NSH * 4u; PIN(oso) PIN(osz) } \
        if constexpr ((J) == S_K5) { PIN(sxn) PIN(blk) g0 = g1; g1 = (sxn >> 16) | (blk < NBLK ? 0u : 0x80000000u); PIN(g1) } \
        if constexpr ((J) == S_K6) {     \
            if (nr > rdy) { asm volatile("" : "+v"(PN)); const int c = count(PN); rdy = c > rdy ? c : rdy; } \
            else if (pszo) { asm volatile("" : "+v"(PO)); const int c = count(PO); rdy = c > rdy ? c : rdy; } \
            pszo = psz; PIN(rdy) PIN(pszo) } \
        if constexpr ((J) == S_K7) { PIN(own) PIN(blk) have = own ? rdyo : rdy; PIN(have) have = blk < NBLK ? have : 2 * NBLK; psz = rdy - nr < POLL_AHEAD ? NR * 2048 : 0u; BSTAMP(i); ++i; PIN(have) PIN(psz) PIN(i) } \
        SB
#define STEP4(CUR, NXT, PP, PO, PN, S) SLOT(CUR, NXT, PP, PO, PN, 4 * S, 1) SLOT(CUR, NXT, PP, PO, PN, 4 * S + 1, 2) SLOT(CUR, NXT, PP, PO, PN, 4 * S + 2, 3) SLOT(CUR, NXT, PP, PO, PN, 4 * S + 3, 0)
#define BODY(CUR, NXT, PP, PO, PN) { \
        if (blk >= have) { const int c = spin(own ? sflo : sfl, blk); rdyo = own ? c : rdyo; rdy = own ? rdy : c; } \
        if (blk >= 28 && blk < 32 && !own) STAMP(40 + blk - 28); \
        sx = sxn; \
        const __amdgpu_buffer_rsrc_t rx = __builtin_amdgcn_make_buffer_rsrc(xall, 0, xszn, 0x00020000); \
        f32x4 r0, r1, r2, r3; unsigned src; \
        STEP4(CUR, NXT, PP, PO, PN, 0) STEP4(CUR, NXT, PP, PO, PN, 1) STEP4(CUR, NXT, PP, PO, PN, 2) STEP4(CUR, NXT, PP, PO, PN, 3) \
        STEP4(CUR, NXT, PP, PO, PN, 4) STEP4(CUR, NXT, PP, PO, PN, 5) STEP4(CUR, NXT, PP, PO, PN, 6) STEP4(CUR, NXT, PP, PO, PN, 7) }

    int i = 0; u32x2 P0 = {0u, 0u}, P1;
    BODY(X1, X0, 1, P0, P1)
    do { BODY(X0, X1, 0, P1, P0) BODY(X1, X0, 1, P0, P1) } while (i < n);
    {
        asm volatile("s_nop 7\n\ts_nop 7" : "+v"(c0), "+v"(c2), "+v"(c3));
        *(LAS f32x4*)(lw2 + 0 * 16384) = c2; *(LAS f32x4*)(lw3 + 0 * 16384) = c3; *(LAS f32x4*)(lw0 + 0 * 16384) = c0;
        wait_lgkm0(); __builtin_amdgcn_s_barrier(); asm volatile("" ::: "memory");
        store_rows(fmul4(fadd4(fadd4(*(const LAS f32x4*)(lr + 0 * 16384), *(const LAS f32x4*)(lr + 0 * 16384 + 4096)), fadd4(*(const LAS f32x4*)(lr + 0 * 16384 + 8192), *(const LAS f32x4*)(lr + 0 * 16384 + 12288))), scale));
        STAMP(39);
        asm volatile(".Lfk_code_end:");
    }
#undef BODY
#undef STEP4
#undef SLOT
#undef XFRAG
#undef SB
#undef XLOAD
#undef POLL
#undef PIN
}
}

extern "C" void kernel_launch(int rank, int nranks, ncclComm_t comm,
                              hipStream_t stream,
                              void* const* d_in, const int* in_sizes, int n_in,
                              void* d_out, int out_size,
                              const int* in_dtypes, int out_dtype,
                              void* const* d_out_all,
                              void* const* d_ws_all, size_t ws_bytes) {
    (void)comm; (void)in_dtypes; (void)out_dtype; (void)d_out_all;
    if (nranks != NR || (unsigned)rank >= (unsigned)NR || n_in != 4 || ws_bytes < WS_END ||
        in_sizes[0] != MSH * KDIM || in_sizes[1] != KDIM * NFULL || out_size != NR * MSH * NSH) {
        fprintf(stderr, "kernel_launch: unexpected problem shape\n");
        return;
    }
    Params p;
    p.x = (const unsigned char*)d_in[0]; p.w = (const unsigned char*)d_in[1];
    p.sx = (const float*)d_in[2]; p.sw = (const float*)d_in[3]; p.out = (float*)d_out;
    for (int i = 0; i < NR; ++i) p.wsr[i] = (char*)d_ws_all[(rank + 1 + i) & 7];
    p.rank = rank; p.slow = (int)((3u - (unsigned)((size_t)p.w >> 7)) & 7u) * 2;
    fused_kernel<<<256, 256, 0, stream>>>(p);
    HIP_CHECK(hipGetLastError());
}
```

```cpp
#include <hip/hip_runtime.h>
#include <rccl/rccl.h>
#include <cstdio>

#define HIP_CHECK(expr) do { hipError_t _e = (expr); if (_e != hipSuccess) { \
    fprintf(stderr, "HIP error %s at %s:%d\n", hipGetErrorString(_e), __FILE__, __LINE__); \
    return; } } while (0)

namespace {
typedef unsigned u32x2 __attribute__((ext_vector_type(2)));
typedef unsigned u32x4 __attribute__((ext_vector_type(4)));
typedef unsigned u32x8 __attribute__((ext_vector_type(8)));
typedef float f32x4 __attribute__((ext_vector_type(4)));
#define LAS __attribute__((address_space(3)))
#define DI __device__ __forceinline__

constexpr int NR = 8;
constexpr int KDIM = 4096, NFULL = 16384, NSH = 2048, MSH = 512;
constexpr unsigned SHARD = MSH * KDIM;
constexpr int NBLK = MSH / 16;
constexpr int NUNIT = 512;
constexpr int WPP = 16;
constexpr size_t WS_X = 0;
constexpr size_t WS_FLAG = 2ull * NR * SHARD;
constexpr size_t WS_RTR = WS_FLAG + NR * 2048;
constexpr size_t WS_TICK = WS_RTR + NR * 128;
constexpr size_t WS_TMO = WS_TICK + 1024;
constexpr size_t WS_CALL = WS_TMO + 1024;
constexpr size_t WS_DBG = WS_CALL + 1024 * 64;
constexpr size_t WS_END = WS_DBG + 2 * 256 * 64 * 8;
constexpr unsigned long long DEADLINE_TICKS = 30000000ull;

#define OWNMASK 0x155ull
#define POLL_AHEAD 3
#define NSPEC 8
#define TESTREDO 0
#define NOSTORE 0
#define WTOUCH 1
#define NOSLOW 0
#define SPOLL 16
#define OUT_AUX 2

struct Params {
    const unsigned char* x; const unsigned char* w; const float* sx; const float* sw; float* out;
    char* wsr[NR];
    int rank; int slow;
};

#define ACLOB "a0","a1","a2","a3","a4","a5","a6","a7","a8","a9","a10","a11","a12","a13","a14","a15","a16","a17","a18","a19","a20","a21","a22","a23","a24","a25","a26","a27","a28","a29","a30","a31","a32","a33","a34","a35","a36","a37","a38","a39","a40","a41","a42","a43","a44","a45","a46","a47","a48","a49","a50","a51","a52","a53","a54","a55","a56","a57","a58","a59","a60","a61","a62","a63","a64","a65","a66","a67","a68","a69","a70","a71","a72","a73","a74","a75","a76","a77","a78","a79","a80","a81","a82","a83","a84","a85","a86","a87","a88","a89","a90","a91","a92","a93","a94","a95","a96","a97","a98","a99","a100","a101","a102","a103","a104","a105","a106","a107","a108","a109","a110","a111","a112","a113","a114","a115","a116","a117","a118","a119","a120","a121","a122","a123","a124","a125","a126","a127","a128","a129","a130","a131","a132","a133","a134","a135","a136","a137","a138","a139","a140","a141","a142","a143","a144","a145","a146","a147","a148","a149","a150","a151","a152","a153","a154","a155","a156","a157","a158","a159","a160","a161","a162","a163","a164","a165","a166","a167","a168","a169","a170","a171","a172","a173","a174","a175","a176","a177","a178","a179","a180","a181","a182","a183","a184","a185","a186","a187","a188","a189","a190","a191","a192","a193","a194","a195","a196","a197","a198","a199","a200","a201","a202","a203","a204","a205","a206","a207","a208","a209","a210","a211","a212","a213","a214","a215","a216","a217","a218","a219","a220","a221","a222","a223","a224","a225","a226","a227","a228","a229","a230","a231","a232","a233","a234","a235","a236","a237","a238","a239","a240","a241","a242","a243","a244","a245","a246","a247","a248","a249","a250","a251","a252","a253","a254","a255"

template <int N> DI void wait_vm() { asm volatile("s_waitcnt vmcnt(%0)" :: "n"(N) : "memory"); }
DI void wait_lgkm0() { asm volatile("s_waitcnt lgkmcnt(0)" ::: "memory"); }
template <int B> DI void snd_load(const unsigned char* base, unsigned voff) {
    asm volatile("global_load_dwordx4 a[%c2:%c2+3], %0, %1\n\t"
                 "global_load_dwordx4 a[%c2+4:%c2+7], %0, %1 offset:64\n\t"
                 "global_load_dwordx4 a[%c2+8:%c2+11], %0, %1 offset:128\n\t"
                 "global_load_dwordx4 a[%c2+12:%c2+15], %0, %1 offset:192" :: "v"(voff), "s"(base), "i"(B) : "memory");
}
template <int B> DI void snd_store(char* base, unsigned voff) {
    asm volatile("global_store_dwordx4 %0, a[%c2:%c2+3], %1 sc0 sc1\n\t"
                 "global_store_dwordx4 %0, a[%c2+4:%c2+7], %1 offset:1024 sc0 sc1\n\t"
                 "global_store_dwordx4 %0, a[%c2+8:%c2+11], %1 offset:2048 sc0 sc1\n\t"
                 "global_store_dwordx4 %0, a[%c2+12:%c2+15], %1 offset:3072 sc0 sc1\n\ts_nop 1" :: "v"(voff), "s"(base), "i"(B) : "memory");
}
template <int A> DI void snd_ticket(char* ctr, unsigned vzero) {
    unsigned long long keep;
    asm volatile("s_mov_b64 %0, exec\n\ts_mov_b64 exec, 1\n\tglobal_atomic_add a[%c3], %1, a33, %2 sc0\n\ts_mov_b64 exec, %0"
                 : "=&s"(keep) : "v"(vzero), "s"(ctr), "i"(A) : "memory");
}
DI void code_prefetch(unsigned lane) {
    unsigned long long pc; unsigned t0, t1, t2;
    asm volatile("s_getpc_b64 %0\n"
                 ".Lfk_code_begin:\n\t"
                 "v_min_u32 %1, .Lfk_code_end-.Lfk_code_begin, %4\n\tv_min_u32 %2, .Lfk_code_end-.Lfk_code_begin, %5\n\tv_min_u32 %3, .Lfk_code_end-.Lfk_code_begin, %6\n\t"
                 "global_load_dword a252, %1, %0\n\tglobal_load_dword a253, %2, %0\n\tglobal_load_dword a254, %3, %0"
                 : "=&s"(pc), "=&v"(t0), "=&v"(t1), "=&v"(t2) : "v"(lane * 128u), "v"(lane * 128u + 8192u), "v"(lane * 128u + 16384u) : "memory", "a252", "a253", "a254");
}
template <int A> DI void snd_flag(char* flag, unsigned vzero) {
    unsigned long long keep;
    asm volatile("s_mov_b64 %0, exec\n\ts_mov_b64 exec, 1\n\tglobal_store_dword %1, a[%c3], %2 sc0 sc1\n\ts_mov_b64 exec, %0"
                 : "=&s"(keep) : "v"(vzero), "s"(flag), "i"(A) : "memory");
}
template <int A> DI void acc_set(unsigned v) { asm volatile("s_nop 1\n\tv_accvgpr_write_b32 a[%c1], %0\n\ts_nop 1" :: "s"(v), "i"(A)); }
DI unsigned unit_src(unsigned u) { return (u >> 4) * 65536u + (u & 15u) * 256u; }
DI unsigned unit_flag(unsigned u) { return (((u >> 2) & 3u) * 128u + (u >> 4) * 4u + (u & 3u)) * 4u; }
template <int N, int A = 32> DI unsigned snd_wait_ticket() {
    unsigned v;
    asm volatile("s_waitcnt vmcnt(%1)\n\tv_accvgpr_read_b32 %0, a[%c2]\n\ts_nop 1" : "=v"(v) : "n"(N), "i"(A) : "memory");
    return __builtin_amdgcn_readfirstlane(v);
}

#define STAMP(k) do { } while (0)
#define GOSTAMP
#define BSTAMP(i) do { } while (0)

template <int S, int SLOT> DI void w_read(unsigned a0, unsigned a1, unsigned a2, unsigned a3) {
    constexpr int B = S * 32, O = SLOT * 8192;
#define TRR(c, t, A) "ds_read_b64_tr_b8 a[%c4+" #c "*8+" #t "*2:%c4+" #c "*8+" #t "*2+1], " A " offset:%c5+4096*(" #t "/2)+512*(" #t "%%2)\n\t"
    asm volatile(TRR(0, 0, "%0") TRR(0, 1, "%0") TRR(0, 2, "%0") TRR(0, 3, "%0")
                 TRR(1, 0, "%1") TRR(1, 1, "%1") TRR(1, 2, "%1") TRR(1, 3, "%1")
                 TRR(2, 0, "%2") TRR(2, 1, "%2") TRR(2, 2, "%2") TRR(2, 3, "%2")
                 TRR(3, 0, "%3") TRR(3, 1, "%3") TRR(3, 2, "%3") TRR(3, 3, "%3")
                 "s_waitcnt lgkmcnt(0)"
                 :: "v"(a0), "v"(a1), "v"(a2), "v"(a3), "i"(B), "i"(O) : "memory", ACLOB);
#undef TRR
}
DI void w_stage(__amdgpu_buffer_rsrc_t rs, unsigned ve, unsigned vo, unsigned m0base, unsigned soff) {
    unsigned keep, so;
#define WST(V) "s_nop 0\n\tbuffer_load_dwordx4 " V ", %5, %1 offen lds\n\t"
#define WNX "s_add_u32 m0, m0, 0x400\n\ts_add_u32 %1, %1, 0x40000\n\t"
    asm volatile("s_mov_b32 %0, m0\n\ts_mov_b32 m0, %2\n\ts_mov_b32 %1, %6\n\t"
                 WST("%3") WNX WST("%4") WNX WST("%3") WNX WST("%4") WNX WST("%3") WNX WST("%4") WNX WST("%3") WNX WST("%4")
                 "s_mov_b32 m0, %0"
                 : "=&s"(keep), "=&s"(so) : "s"(m0base), "v"(ve), "v"(vo), "s"(rs), "s"(soff) : "scc", "memory");
#undef WST
#undef WNX
    static_assert(16 * NFULL == 0x40000, "");
}
DI void x_stage(const unsigned char* base, unsigned vdst, unsigned m0base) {
    unsigned keep, v;
#define XST(r) "v_xor_b32 %1, " #r "*16, %3\n\tv_add_u32 %1, " #r "*4096, %1\n\tglobal_load_lds_dwordx4 %1, %4\n\ts_add_u32 m0, m0, 0x400\n\t"
    asm volatile("s_mov_b32 %0, m0\n\ts_mov_b32 m0, %2\n\t"
                 XST(0) XST(1) XST(2) XST(3) XST(4) XST(5) XST(6) XST(7) XST(8) XST(9) XST(10) XST(11) XST(12) XST(13) XST(14) XST(15)
                 "s_mov_b32 m0, %0"
                 : "=&s"(keep), "=&v"(v) : "s"(m0base), "v"(vdst), "s"(base) : "scc", "memory");
#undef XST
}
template <int S, int C> DI void mf(f32x4& c, const u32x8& x, const f32x4& cinit) {
    constexpr int B = S * 32 + C * 8;
    if constexpr (S == 0) asm volatile("v_mfma_f32_16x16x128_f8f6f4 %0, a[%c3:%c3+7], %1, %2" : "+v"(c) : "v"(x), "v"(cinit), "i"(B));
    else asm volatile("v_mfma_f32_16x16x128_f8f6f4 %0, a[%c2:%c2+7], %1, %0" : "+v"(c) : "v"(x), "i"(B));
}
DI float fadd(float a, float b) { float r; asm("v_add_f32_e32 %0, %1, %2" : "=v"(r) : "v"(a), "v"(b)); return r; }
DI float fmul(float a, float b) { float r; asm("v_mul_f32_e32 %0, %1, %2" : "=v"(r) : "v"(a), "v"(b)); return r; }
DI f32x4 fadd4(f32x4 a, f32x4 b) { return (f32x4){fadd(a[0], b[0]), fadd(a[1], b[1]), fadd(a[2], b[2]), fadd(a[3], b[3])}; }
DI f32x4 fmul4(f32x4 a, float b) { return (f32x4){fmul(a[0], b), fmul(a[1], b), fmul(a[2], b), fmul(a[3], b)}; }

constexpr int S_K1 = 4, S_BAR = 6, S_RD = 8, S_K2 = 10, S_A1 = 12, S_A2 = 14, S_POLL = SPOLL, S_A3 = 18, S_MU = 20, S_ST = 22, S_K3 = 24, S_K4 = 26, S_K5 = 28, S_K6 = 30;

struct XBlk { u32x4 h[16]; };

__global__ __launch_bounds__(256, 1) void fused_kernel(Params p) {
    __shared__ __attribute__((aligned(16))) unsigned char lds[4 * 32768 + 32768];
    const int lane = threadIdx.x & 63;
    const int wid = __builtin_amdgcn_readfirstlane(threadIdx.x >> 6);
    const int rank = p.rank;
    char* const me = p.wsr[7];
    const int bid = blockIdx.x;
    code_prefetch(lane);
    const bool sender = bid < 24 || (bid < 32 && (bid & 2));
    const int w = (bid < 24 ? bid : 24 + (bid & 1) + ((bid >> 1) & 2)) * 4 + wid, pi = w % 7;
    char* const pws = p.wsr[pi];
    asm volatile("" :: "s"(p.x), "s"(p.w), "s"(me), "s"(rank), "s"(pws));
    STAMP(0);

    char* const callp = me + WS_CALL + (size_t)(bid * 4 + wid) * 64;

    const unsigned vsrc = (lane & 15) * KDIM + (lane >> 4) * 16, vdst = lane * 16;

    if (__builtin_expect(sender, 1)) {
        const int j = w / 7;
        unsigned vzero = 0;
        asm volatile("v_accvgpr_write_b32 a33, 1" : "+v"(vzero) :: "a0", "a31", "a32", "a33", "a34", "a35", "a36");
        auto srcp = [&](unsigned u) { return p.x + unit_src(u < NUNIT ? u : NUNIT - 1); };
        unsigned tk0 = j, tk1 = j + WPP, prev;
        char* const ctr = me + WS_TICK + pi * 128;
        snd_load<0>(srcp(tk0), vsrc);
        snd_ticket<35>(callp, vzero);
        snd_ticket<32>(ctr, vzero);
        snd_load<16>(srcp(tk1), vsrc);
        const unsigned call = snd_wait_ticket<5, 35>() + 1u;
        char* const dst = pws + WS_X + (size_t)((call & 1u) * NR + rank) * SHARD;
        snd_store<0>(dst + tk0 * 4096u, vdst);
        acc_set<34>(call);
        const unsigned tbase = (call - 1u) * NUNIT - 2u * WPP;
        char* const fl = pws + WS_FLAG + rank * 2048;
        char* const own = me + WS_X + (size_t)((call & 1u) * NR + rank) * SHARD;
        prev = tk0; tk0 = tk1; tk1 = snd_wait_ticket<8>() - tbase;
        const char* const rtr = me + WS_RTR + ((unsigned)(rank + 1 + pi) & 7u) * 128;
        int fill = 0;
#define SEND_STEP(BA, BB, NLOAD) { \
            if (tk0 >= NUNIT - 16 * NSPEC - 64 && fill == 0) { asm volatile("global_load_dword a36, %0, %1 sc0 sc1" :: "v"((lane & 31) * 4), "s"(rtr) : "memory"); fill = 1; } \
            snd_ticket<32>(ctr, vzero); \
            snd_load<BB>(srcp(tk1), vsrc); \
            wait_vm<NLOAD>();                                            \
            if (tk0 >= NUNIT - 16 * NSPEC && (fill & 1)) { \
                const unsigned long long t0 = __builtin_amdgcn_s_memrealtime(); \
                for (fill = 2;;) { \
                    unsigned f, tmo; \
                    asm volatile("global_load_dword %0, %2, %3 sc0 sc1\n\tglobal_load_dword %1, %4, %5 sc1\n\ts_waitcnt vmcnt(0)" \
                                 : "=&v"(f), "=&v"(tmo) : "v"((lane & 31) * 4), "s"(rtr), "v"(vzero), "s"(me + WS_TMO) : "memory"); \
                    if (__all(f >= call)) break; \
                    if (__builtin_amdgcn_readfirstlane(tmo) == call || __builtin_amdgcn_s_memrealtime() - t0 > DEADLINE_TICKS) { snd_flag<34>(me + WS_TMO, vzero); fill = 4; break; } \
                    __builtin_amdgcn_s_sleep(4); \
                } \
            } \
            if (TESTREDO && tk0 >= NUNIT - 16 * NSPEC && (tk0 & 3) == 1) { __builtin_amdgcn_s_sleep(127); __builtin_amdgcn_s_sleep(127); } \
            snd_store<BA>((fill == 4 ? own : dst) + tk0 * 4096u, vdst); \
            const unsigned tk2 = snd_wait_ticket<8>() - tbase;           \
            if (fill == 1) { unsigned f; asm volatile("v_accvgpr_read_b32 %0, a36" : "=v"(f)); fill = __all(f >= call) ? 2 : 3; } \
            snd_flag<34>(fill == 4 ? me + WS_TMO : fl + unit_flag(prev), vzero); \
            prev = tk0; tk0 = tk1; tk1 = tk2; }
        SEND_STEP(16, 0, 9)
        if (tk0 < NUNIT)
        for (;;) {
            SEND_STEP(0, 16, 10) if (tk0 >= NUNIT) break;
            SEND_STEP(16, 0, 10) if (tk0 >= NUNIT) break;
        }
#undef SEND_STEP
        wait_vm<0>();
        snd_flag<34>(fill == 4 ? me + WS_TMO : fl + unit_flag(prev), vzero);
        STAMP(1);
        return;
    }

    int cg, rg;
    {
        const int xcd = bid & 7, i = (xcd & 1) * 28 + (bid >> 3) - 4;
        rg = bid < 32 ? 7 : i >> 3;
        cg = bid < 32 ? p.slow + (xcd & 1) + (xcd >> 2) * 16 : (xcd >> 1) * 8 + (i & 7);
    }
    const int q = wid;
    const float scale = p.sx[0] * p.sw[0];
    unsigned call;
    const unsigned ldsw = (unsigned)(size_t)(LAS unsigned char*)lds + q * 32768u;
    const unsigned peer = (unsigned)(rank + 1 + rg) & 7u;
    const unsigned sfl = peer * 2048 + q * 512, sflo = rank * 2048 + q * 512, vfl = lane * 8;
    XBlk X0, X1;
    u32x2 pf;
    {
        const __amdgpu_buffer_rsrc_t rw = __builtin_amdgcn_make_buffer_rsrc((void*)p.w, 0, (unsigned)KDIM * NFULL, 0x00020000);
        const unsigned ve = (lane >> 2) * NFULL + (((lane & 3) ^ ((lane >> 4) & 1)) * 16), vo = ve ^ 32u;
        const unsigned so = (unsigned)(q * 1024) * NFULL + rank * NSH + cg * 64;
        const unsigned ab = ldsw + ((lane >> 4) * 16 + ((lane & 15) >> 1)) * 64 + (lane & 1) * 8, f = (lane >> 3) & 3;
        const unsigned a0 = ab + ((0 ^ f) << 4), a1 = ab + ((1 ^ f) << 4), a2 = ab + ((2 ^ f) << 4), a3 = ab + ((3 ^ f) << 4);
        constexpr unsigned SS = 128u * NFULL;
        if (rg < 7) {
            const unsigned l0 = (unsigned)(((bid >> 3) - 4) * 4 + q) * 64u + lane, cb = rank * NSH + (cg & 24) * 64;
            const unsigned l2 = l0 + 2 * 7168u;
            asm volatile("buffer_load_dword a249, %0, %3, %4 offen\n\tbuffer_load_dword a250, %1, %3, %4 offen\n\tbuffer_load_dword a251, %2, %3, %4 offen"
                         :: "v"((l0 >> 2) * NFULL + (l0 & 3) * 128u), "v"(((l0 + 7168u) >> 2) * NFULL + (l0 & 3) * 128u),
                            "v"(((l2 < 16384u ? l2 : l0) >> 2) * NFULL + (l0 & 3) * 128u), "s"(rw), "s"(cb) : "memory", "a249", "a250", "a251");
        }
        unsigned vzero = 0;
        asm volatile("v_accvgpr_write_b32 a33, 1\n\ts_nop 1" : "+v"(vzero) :: "a32", "a33");
        snd_ticket<32>(callp, vzero);
#define WOFF(S) (so + (unsigned)(((S) + rg) & 7) * SS)
#define WFIRST w_stage(rw, ve, vo, ldsw + 0 * 8192, WOFF(0)); w_stage(rw, ve, vo, ldsw + 1 * 8192, WOFF(1)); \
               w_stage(rw, ve, vo, ldsw + 2 * 8192, WOFF(2)); w_stage(rw, ve, vo, ldsw + 3 * 8192, WOFF(3));
        if (rg >= 3 && rg < 7) {
            const unsigned uo = (unsigned)(((rg - 3) * 32 + cg) * 4 + q);
            snd_load<224>(p.x + unit_src(uo), vsrc);
            call = snd_wait_ticket<0>() + 1u;
            acc_set<240>(call);
            snd_store<224>(me + WS_X + (size_t)((call & 1u) * NR + rank) * SHARD + uo * 4096u, vdst);
            WFIRST
            wait_vm<32>();
            snd_flag<240>(me + WS_FLAG + rank * 2048 + unit_flag(uo), vzero);
            wait_vm<24>();
        } else {
            WFIRST
            call = snd_wait_ticket<24>() + 1u;
        }
#undef WFIRST
        if (rg < 7) {
            const u32x4 ff = {~0u, ~0u, ~0u, ~0u};
            char* const pz = me + WS_X + (size_t)((call & 1u) * NR + peer) * SHARD + (NBLK - NSPEC) * 65536;
#pragma unroll
            for (int k = 0; k < (NSPEC + 1) / 2; ++k)
                if (k * 128 + cg * 4 + q < NSPEC * 64)
                    asm volatile("global_store_dwordx4 %0, %1, %2 sc0 sc1\n\ts_nop 1" :: "v"(vdst), "v"(ff), "s"(pz + (k * 128 + cg * 4 + q) * 1024) : "memory");
        }
        w_read<0, 0>(a0, a1, a2, a3); w_stage(rw, ve, vo, ldsw + 0 * 8192, WOFF(4));
        wait_vm<24>(); w_read<1, 1>(a0, a1, a2, a3); w_stage(rw, ve, vo, ldsw + 1 * 8192, WOFF(5));
        wait_vm<24>(); w_read<2, 2>(a0, a1, a2, a3); w_stage(rw, ve, vo, ldsw + 2 * 8192, WOFF(6));
        wait_vm<24>(); w_read<3, 3>(a0, a1, a2, a3); w_stage(rw, ve, vo, ldsw + 3 * 8192, WOFF(7));
#undef WOFF
        wait_vm<24>(); w_read<4, 0>(a0, a1, a2, a3);
        wait_vm<16>(); w_read<5, 1>(a0, a1, a2, a3);
        pf = __builtin_bit_cast(u32x2, __builtin_amdgcn_raw_buffer_load_b64(__builtin_amdgcn_make_buffer_rsrc(me + WS_FLAG, 0, NR * 2048, 0x00020000), vfl, sfl, 17));
        x_stage(p.x + rg * 65536 + q * 1024, vdst, ldsw);
        wait_vm<8 + 17>(); w_read<6, 2>(a0, a1, a2, a3);
        wait_vm<17>();     w_read<7, 3>(a0, a1, a2, a3);
        wait_vm<0>();
        __builtin_amdgcn_s_barrier();
        if (q == 0 && rg < 7) asm volatile("s_mov_b64 exec, 1\n\tglobal_store_dword %0, %1, %2 sc0 sc1\n\ts_mov_b64 exec, -1" :: "v"(vzero), "v"(call), "s"(p.wsr[rg] + WS_RTR + rank * 128 + cg * 4) : "memory");
        const unsigned xb = q * 32768 + (lane & 15) * 1024 + (((lane >> 4) ^ (lane & 3)) << 4), xs = (lane & 12) << 4;
#pragma unroll
        for (int k = 0; k < 16; ++k) X1.h[k] = *(const LAS u32x4*)((LAS unsigned char*)lds + xb + (((unsigned)((k + 2 * rg) & 15) << 6) ^ xs));
    }
    STAMP(1);
    static_assert(__builtin_popcountll(OWNMASK) == 5 && (OWNMASK & 3) == 1 && (OWNMASK >> 32) == 0, "");
    const bool slowcg = (cg & 14) == p.slow;
    const int n = rg == 7 ? NBLK - 7 : slowcg ? NBLK + 1 : NBLK + 5, ostep = rg == 7 ? 1 : 7;
    const unsigned ownpos = rg == 7 ? ~0u : slowcg ? 1u : (unsigned)OWNMASK;
    unsigned tabx; int tabr;
    {
        const int before = __builtin_amdgcn_mbcnt_lo(ownpos, 0u);
        const bool isown = lane < 32 && ((ownpos >> (lane & 31)) & 1u);
        const int b = isown ? rg + ostep * before : lane - before;
        tabx = (((isown ? rank : peer) * NBLK + (b & 31)) << 16) | (b < NBLK ? 0u : 1u);
        tabr = isown || b >= NBLK ? -1 : b;
    }
    void* const xall = me + WS_X + (size_t)(call & 1u) * NR * SHARD;
    const unsigned orow = q * 4 + (lane >> 4), voffO = orow * (NSH * 4) + (cg * 64 + (lane & 15) * 4) * 4;
    const unsigned lwb = 131072 + q * 4096 + (lane & 15) * 256 + (((lane >> 4) ^ (lane & 15)) << 4);
    LAS unsigned char* const lw0 = (LAS unsigned char*)lds + lwb; LAS unsigned char* const lw1 = (LAS unsigned char*)lds + (lwb ^ 64);
    LAS unsigned char* const lw2 = (LAS unsigned char*)lds + (lwb ^ 128); LAS unsigned char* const lw3 = (LAS unsigned char*)lds + (lwb ^ 192);
    LAS unsigned char* const lr = (LAS unsigned char*)lds + 131072 + orow * 256 + (((lane & 15) ^ orow) << 4);
    unsigned xlim = NR * SHARD;
    f32x4 cinit = {0.f, 0.f, 0.f, 0.f};
    unsigned xsz = NR * SHARD;

    auto count = [&](u32x2 f) { unsigned r; asm("s_ff1_i32_b64 %0, %1" : "=s"(r) : "s"(__ballot(f.x < call || f.y < call))); return (int)(r >> 1); };
    auto spin = [&](unsigned soff, int need) {
        const char* const pf = me + WS_FLAG + soff;
        const unsigned long long t0 = __builtin_amdgcn_s_memrealtime();
        asm volatile("" :: "s"(t0));
        for (;;) {
            u32x2 f; unsigned tmo;
            asm volatile("global_load_dwordx2 %0, %2, %3 sc0 sc1\n\tglobal_load_dword %1, %4, %5 sc1\n\ts_waitcnt vmcnt(0)"
                         : "=&v"(f), "=&v"(tmo) : "v"(vfl), "s"(pf), "v"(0u), "s"(me + WS_TMO) : "memory");
            const int c = count(f);
            if (c > need) return c;
            if (__builtin_amdgcn_readfirstlane(tmo) == call || __builtin_amdgcn_s_memrealtime() - t0 > DEADLINE_TICKS) {
                if (lane == 0) __hip_atomic_store((unsigned*)(me + WS_TMO), call, __ATOMIC_RELAXED, __HIP_MEMORY_SCOPE_AGENT);
                xlim = 0; xsz = 0; const float qn = __builtin_nanf(""); cinit = (f32x4){qn, qn, qn, qn};
                asm volatile("s_nop 1" : "+v"(cinit));
                return 2 * NBLK;
            }
            __builtin_amdgcn_s_sleep(1);
        }
    };
    unsigned voffR[8];
#pragma unroll
    for (int k = 0; k < 8; ++k) voffR[k] = vdst + q * 16384 + (unsigned)((k + rg) & 7) * 2048u;
    f32x4 c0 = cinit, c1 = cinit, c2 = cinit, c3 = cinit;
    asm volatile("s_nop 1" : "+v"(cinit), "+v"(c0), "+v"(c1), "+v"(c2), "+v"(c3));
#define XLOAD(X, k) (X).h[k] = __builtin_bit_cast(u32x4, __builtin_amdgcn_raw_buffer_load_b128(rx, voffR[(k) >> 1], sx + ((k) & 1) * 1024u, 17))
#define POLL(size, off) __builtin_bit_cast(u32x2, __builtin_amdgcn_raw_buffer_load_b64(__builtin_amdgcn_make_buffer_rsrc(me + WS_FLAG, 0, size, 0x00020000), vfl, off, 17))
    int rdy = rg == 7 ? 2 * NBLK : count(pf);
    unsigned xm = 1u, x0 = __builtin_amdgcn_readlane(tabx, 0), x1 = __builtin_amdgcn_readlane(tabx, 1);
    int r1 = __builtin_amdgcn_readlane(tabr, 1), need = r1 + 1, thr = need;
    int sp0 = NBLK;
    unsigned sx, oso = 0, osz = 0, psz = NR * 2048;
    auto store_rows = [&](const f32x4& r) {
        __builtin_amdgcn_raw_buffer_store_b128(__builtin_bit_cast(u32x4, r), __builtin_amdgcn_make_buffer_rsrc((void*)p.out, 0, NOSTORE ? 0u : osz, 0x00020000), voffO, oso, OUT_AUX);
    };
#define SB __builtin_amdgcn_sched_barrier(0);
#define XFRAG(X, S) __builtin_shufflevector((X).h[2 * (S)], (X).h[2 * (S) + 1], 0, 1, 2, 3, 4, 5, 6, 7)
#define PIN(v) asm volatile("" : "+s"(v));
#define SLOT(FIRST, CUR, NXT, PP, PO, PN, J, C) \
        mf<(J) / 4, C>(c##C, XFRAG(CUR, (J) / 4), cinit); SB \
        if constexpr ((J) == 0) *(LAS f32x4*)(lw2 + PP * 16384) = c2; \
        if constexpr ((J) == 1) *(LAS f32x4*)(lw3 + PP * 16384) = c3; \
        if constexpr ((J) == 2) *(LAS f32x4*)(lw0 + PP * 16384) = c0; \
        if constexpr ((J) == 31) *(LAS f32x4*)(lw1 + (1 - PP) * 16384) = c1; \
        if constexpr ((J) == S_POLL) PN = POLL(psz, sfl); \
        if constexpr (FIRST && (J) == S_POLL - 2) pf = POLL(NR * 2048, sflo); \
        if constexpr ((J) & 1) XLOAD(NXT, (J) >> 1); \
        if constexpr ((J) == S_BAR) { wait_lgkm0(); __builtin_amdgcn_s_barrier(); asm volatile("" ::: "memory"); } \
        if constexpr ((J) == S_RD) { q0 = *(const LAS f32x4*)(lr + PP * 16384); q1 = *(const LAS f32x4*)(lr + PP * 16384 + 4096); \
                                     q2 = *(const LAS f32x4*)(lr + PP * 16384 + 8192); q3 = *(const LAS f32x4*)(lr + PP * 16384 + 12288); } \
        if constexpr ((J) == S_A1) q1 = fadd4(q0, q1); \
        if constexpr ((J) == S_A2) q2 = fadd4(q2, q3); \
        if constexpr ((J) == S_A3) q1 = fadd4(q1, q2); \
        if constexpr ((J) == S_MU) q1 = fmul4(q1, scale); \
        if constexpr ((J) == S_ST) store_rows(q1); \
        if constexpr ((J) == S_K1) { PIN(r1) sp1 = r1 >= rdy ? r1 : NBLK; PIN(sp1) PIN(i) i2 = i + 2; PIN(i2) } \
        if constexpr ((J) == S_K2) { PIN(xm) oso = xm << 1; osz = (xm & 1u) ? 0u : (unsigned)(NR * MSH) * NSH * 4u; PIN(oso) PIN(osz) } \
        if constexpr ((J) == S_K3) { PIN(i2) xm = x0; x0 = x1; x1 = __builtin_amdgcn_readlane(tabx, i2); r1 = __builtin_amdgcn_readlane(tabr, i2); PIN(x1) PIN(r1) } \
        if constexpr ((J) == S_K3 + 1) { PIN(x1) xszn = (x1 & 1u) ? 0u : xlim; PIN(xszn) } \
        if constexpr ((J) == S_K4) { PIN(r1) int r2 = r1 + 1; PIN(r2) need = r1 < NBLK - NSPEC ? r2 : r1; PIN(need) } \
        if constexpr ((J) == S_K4 + 1) { PIN(need) thr = sp0 < NBLK ? -1 : need; BSTAMP(i); ++i; PIN(thr) PIN(i) } \
        if constexpr ((J) == S_K5) { asm volatile("" : "+v"(PO)); const int c = count(PO); rdy = c > rdy ? c : rdy; PIN(rdy) }     \
        if constexpr ((J) == S_K6) { PIN(need) psz = rdy < need + POLL_AHEAD ? NR * 2048 : 0u; PIN(psz) } \
        SB
#define STEP4(FIRST, CUR, NXT, PP, PO, PN, S) SLOT(FIRST, CUR, NXT, PP, PO, PN, 4 * S, 1) SLOT(FIRST, CUR, NXT, PP, PO, PN, 4 * S + 1, 2) SLOT(FIRST, CUR, NXT, PP, PO, PN, 4 * S + 2, 3) SLOT(FIRST, CUR, NXT, PP, PO, PN, 4 * S + 3, 0)
#define RECHECK(PP) \
        if (spm < NBLK) { \
            asm volatile("s_nop 7\n\ts_nop 7" : "+v"(c0)); \
            if (__any(c0[0] != c0[0])) { \
                rdy = spin(sfl, spm); \
                const __amdgpu_buffer_rsrc_t rx = __builtin_amdgcn_make_buffer_rsrc(xall, 0, xlim, 0x00020000); \
                sx = xm; \
                XBlk T; \
                _Pragma("unroll") for (int k = 0; k < 16; ++k) XLOAD(T, k); \
                M4(T, 0) M4(T, 1) M4(T, 2) M4(T, 3) M4(T, 4) M4(T, 5) M4(T, 6) M4(T, 7) \
                asm volatile("s_nop 7\n\ts_nop 7" : "+v"(c0), "+v"(c1), "+v"(c2), "+v"(c3)); \
                *(LAS f32x4*)(lw1 + PP * 16384) = c1; \
                STAMP(48); \
            } \
        }
#define M4(X, S) mf<S, 1>(c1, XFRAG(X, S), cinit); mf<S, 2>(c2, XFRAG(X, S), cinit); mf<S, 3>(c3, XFRAG(X, S), cinit); mf<S, 0>(c0, XFRAG(X, S), cinit);
#define BODY(FIRST, CUR, NXT, PP, PO, PN) { \
        if (__builtin_expect((unsigned)rdy < (unsigned)thr, 0)) { \
            RECHECK(PP) \
            if (rdy < need) { const int c = count(PO); rdy = c > rdy ? c : rdy; }     \
            if (rdy < need) rdy = spin(sfl, need - 1); \
        } \
        GOSTAMP \
        sx = x1; \
        const __amdgpu_buffer_rsrc_t rx = __builtin_amdgcn_make_buffer_rsrc(xall, 0, xsz, 0x00020000); \
        f32x4 q0, q1, q2, q3; unsigned xszn; int sp1, i2; \
        STEP4(FIRST, CUR, NXT, PP, PO, PN, 0) STEP4(FIRST, CUR, NXT, PP, PO, PN, 1) STEP4(FIRST, CUR, NXT, PP, PO, PN, 2) STEP4(FIRST, CUR, NXT, PP, PO, PN, 3) \
        STEP4(FIRST, CUR, NXT, PP, PO, PN, 4) STEP4(FIRST, CUR, NXT, PP, PO, PN, 5) STEP4(FIRST, CUR, NXT, PP, PO, PN, 6) STEP4(FIRST, CUR, NXT, PP, PO, PN, 7) \
        xsz = xszn; spm = sp0; sp0 = sp1; }

    int i = 0, spm = NBLK;
    u32x2 P0 = {0u, 0u}, P1;
    if (rg == 7) spin(sflo, NBLK - 1);
    BODY(true, X1, X0, 1, P0, P1)
    if (ownpos != 1u && count(pf) < NBLK) spin(sflo, NBLK - 1);
    do { BODY(false, X0, X1, 0, P1, P0) BODY(false, X1, X0, 1, P0, P1) } while (i < n);
    RECHECK(0)
    oso = xm << 1; osz = (unsigned)(NR * MSH) * NSH * 4u;
    {
        asm volatile("s_nop 7\n\ts_nop 7" : "+v"(c0), "+v"(c2), "+v"(c3));
        *(LAS f32x4*)(lw2 + 0 * 16384) = c2; *(LAS f32x4*)(lw3 + 0 * 16384) = c3; *(LAS f32x4*)(lw0 + 0 * 16384) = c0;
        wait_lgkm0(); __builtin_amdgcn_s_barrier(); asm volatile("" ::: "memory");
        store_rows(fmul4(fadd4(fadd4(*(const LAS f32x4*)(lr + 0 * 16384), *(const LAS f32x4*)(lr + 0 * 16384 + 4096)), fadd4(*(const LAS f32x4*)(lr + 0 * 16384 + 8192), *(const LAS f32x4*)(lr + 0 * 16384 + 12288))), scale));
        STAMP(39);
        asm volatile(".Lfk_code_end:");
    }
#undef BODY
#undef M4
#undef RECHECK
#undef STEP4
#undef SLOT
#undef XFRAG
#undef SB
#undef XLOAD
#undef POLL
#undef PIN
}
}

extern "C" void kernel_launch(int rank, int nranks, ncclComm_t comm,
                              hipStream_t stream,
                              void* const* d_in, const int* in_sizes, int n_in,
                              void* d_out, int out_size,
                              const int* in_dtypes, int out_dtype,
                              void* const* d_out_all,
                              void* const* d_ws_all, size_t ws_bytes) {
    (void)comm; (void)in_dtypes; (void)out_dtype; (void)d_out_all;
    if (nranks != NR || (unsigned)rank >= (unsigned)NR || n_in != 4 || ws_bytes < WS_END ||
        in_sizes[0] != MSH * KDIM || in_sizes[1] != KDIM * NFULL || out_size != NR * MSH * NSH) {
        fprintf(stderr, "kernel_launch: unexpected problem shape\n");
        return;
    }
    Params p;
    p.x = (const unsigned char*)d_in[0]; p.w = (const unsigned char*)d_in[1];
    p.sx = (const float*)d_in[2]; p.sw = (const float*)d_in[3]; p.out = (float*)d_out;
    for (int i = 0; i < NR; ++i) p.wsr[i] = (char*)d_ws_all[(rank + 1 + i) & 7];
    p.rank = rank; p.slow = (int)((3u - (unsigned)((size_t)p.w >> 7)) & 7u) * 2;
    fused_kernel<<<256, 256, 0, stream>>>(p);
    HIP_CHECK(hipGetLastError());
}
```

```cpp
#include <hip/hip_runtime.h>
#include <rccl/rccl.h>
#include <cstdio>

#define HIP_CHECK(expr) do { hipError_t _e = (expr); if (_e != hipSuccess) { \
    fprintf(stderr, "HIP error %s at %s:%d\n", hipGetErrorString(_e), __FILE__, __LINE__); \
    return; } } while (0)

namespace {
typedef unsigned u32x2 __attribute__((ext_vector_type(2)));
typedef unsigned u32x4 __attribute__((ext_vector_type(4)));
typedef unsigned u32x8 __attribute__((ext_vector_type(8)));
typedef float f32x4 __attribute__((ext_vector_type(4)));
#define LAS __attribute__((address_space(3)))
#define DI __device__ __forceinline__

constexpr int NR = 8;
constexpr int KDIM = 4096, NFULL = 16384, NSH = 2048, MSH = 512;
constexpr unsigned SHARD = MSH * KDIM;
constexpr int NBLK = MSH / 16;
constexpr int NUNIT = 512;
constexpr int WPP = 16;
constexpr size_t WS_X = 0;
constexpr size_t WS_FLAG = 2ull * NR * SHARD;
constexpr size_t WS_RTR = WS_FLAG + NR * 2048;
constexpr size_t WS_TICK = WS_RTR + NR * 128;
constexpr size_t WS_TMO = WS_TICK + 1024;
constexpr size_t WS_CALL = WS_TMO + 1024;
constexpr size_t WS_CNT = WS_CALL + 1024 * 4;
constexpr size_t WS_DBG = WS_CNT + 7 * 128 * 64;
constexpr size_t WS_END = WS_DBG + 2 * 256 * 64 * 8;
constexpr unsigned long long DEADLINE_TICKS = 30000000ull;

#define OWNMASK 0x155ull
#define POLL_AHEAD 3
#define NSPEC 8
#define TESTREDO 0
#define NOSTORE 0
#define WTOUCH 1
#define NOSLOW 0
#define ROT rg
#define WDELAY 0
#define NAGG (26 * 16)
#define TOUCHFIRST 0
#define SPOLL 16
#define OUT_AUX 2

struct Params {
    const unsigned char* x; const unsigned char* w; const float* sx; const float* sw; float* out;
    char* wsr[NR];
    int rank; int slow;
};

#define ACLOB "a0","a1","a2","a3","a4","a5","a6","a7","a8","a9","a10","a11","a12","a13","a14","a15","a16","a17","a18","a19","a20","a21","a22","a23","a24","a25","a26","a27","a28","a29","a30","a31","a32","a33","a34","a35","a36","a37","a38","a39","a40","a41","a42","a43","a44","a45","a46","a47","a48","a49","a50","a51","a52","a53","a54","a55","a56","a57","a58","a59","a60","a61","a62","a63","a64","a65","a66","a67","a68","a69","a70","a71","a72","a73","a74","a75","a76","a77","a78","a79","a80","a81","a82","a83","a84","a85","a86","a87","a88","a89","a90","a91","a92","a93","a94","a95","a96","a97","a98","a99","a100","a101","a102","a103","a104","a105","a106","a107","a108","a109","a110","a111","a112","a113","a114","a115","a116","a117","a118","a119","a120","a121","a122","a123","a124","a125","a126","a127","a128","a129","a130","a131","a132","a133","a134","a135","a136","a137","a138","a139","a140","a141","a142","a143","a144","a145","a146","a147","a148","a149","a150","a151","a152","a153","a154","a155","a156","a157","a158","a159","a160","a161","a162","a163","a164","a165","a166","a167","a168","a169","a170","a171","a172","a173","a174","a175","a176","a177","a178","a179","a180","a181","a182","a183","a184","a185","a186","a187","a188","a189","a190","a191","a192","a193","a194","a195","a196","a197","a198","a199","a200","a201","a202","a203","a204","a205","a206","a207","a208","a209","a210","a211","a212","a213","a214","a215","a216","a217","a218","a219","a220","a221","a222","a223","a224","a225","a226","a227","a228","a229","a230","a231","a232","a233","a234","a235","a236","a237","a238","a239","a240","a241","a242","a243","a244","a245","a246","a247","a248","a249","a250","a251","a252","a253","a254","a255"

template <int N> DI void wait_vm() { asm volatile("s_waitcnt vmcnt(%0)" :: "n"(N) : "memory"); }
DI void wait_lgkm0() { asm volatile("s_waitcnt lgkmcnt(0)" ::: "memory"); }
template <int B> DI void snd_load(const unsigned char* base, unsigned voff) {
    asm volatile("global_load_dwordx4 a[%c2:%c2+3], %0, %1\n\t"
                 "global_load_dwordx4 a[%c2+4:%c2+7], %0, %1 offset:64\n\t"
                 "global_load_dwordx4 a[%c2+8:%c2+11], %0, %1 offset:128\n\t"
                 "global_load_dwordx4 a[%c2+12:%c2+15], %0, %1 offset:192" :: "v"(voff), "s"(base), "i"(B) : "memory");
}
template <int B> DI void snd_store(char* base, unsigned voff) {
    asm volatile("global_store_dwordx4 %0, a[%c2:%c2+3], %1 sc0 sc1\n\t"
                 "global_store_dwordx4 %0, a[%c2+4:%c2+7], %1 offset:1024 sc0 sc1\n\t"
                 "global_store_dwordx4 %0, a[%c2+8:%c2+11], %1 offset:2048 sc0 sc1\n\t"
                 "global_store_dwordx4 %0, a[%c2+12:%c2+15], %1 offset:3072 sc0 sc1\n\ts_nop 1" :: "v"(voff), "s"(base), "i"(B) : "memory");
}
template <int A> DI void snd_ticket(char* ctr, unsigned vzero) {
    unsigned long long keep;
    asm volatile("s_mov_b64 %0, exec\n\ts_mov_b64 exec, 1\n\tglobal_atomic_add a[%c3], %1, a33, %2 sc0\n\ts_mov_b64 exec, %0"
                 : "=&s"(keep) : "v"(vzero), "s"(ctr), "i"(A) : "memory");
}
DI void code_prefetch(unsigned lane) {
    unsigned long long pc; unsigned t0, t1, t2;
    asm volatile("s_getpc_b64 %0\n"
                 ".Lfk_code_begin:\n\t"
                 "v_min_u32 %1, .Lfk_code_end-.Lfk_code_begin, %4\n\tv_min_u32 %2, .Lfk_code_end-.Lfk_code_begin, %5\n\tv_min_u32 %3, .Lfk_code_end-.Lfk_code_begin, %6\n\t"
                 "global_load_dword a252, %1, %0\n\tglobal_load_dword a253, %2, %0\n\tglobal_load_dword a254, %3, %0"
                 : "=&s"(pc), "=&v"(t0), "=&v"(t1), "=&v"(t2) : "v"(lane * 128u), "v"(lane * 128u + 8192u), "v"(lane * 128u + 16384u) : "memory", "a252", "a253", "a254");
}
template <int A> DI void call_load(const char* ctr, unsigned vzero) { asm volatile("global_load_dword a[%c2], %0, %1 sc0 sc1" :: "v"(vzero), "s"(ctr), "i"(A) : "memory"); }
template <int A> DI void snd_flag(char* flag, unsigned vzero) {
    unsigned long long keep;
    asm volatile("s_mov_b64 %0, exec\n\ts_mov_b64 exec, 1\n\tglobal_store_dword %1, a[%c3], %2 sc0 sc1\n\ts_mov_b64 exec, %0"
                 : "=&s"(keep) : "v"(vzero), "s"(flag), "i"(A) : "memory");
}
template <int A> DI void snd_flag4(char* flag, unsigned vzero) {
    unsigned long long keep;
    asm volatile("s_mov_b64 %0, exec\n\ts_mov_b64 exec, 1\n\tglobal_store_dwordx4 %1, a[%c3:%c3+3], %2 sc0 sc1\n\ts_mov_b64 exec, %0"
                 : "=&s"(keep) : "v"(vzero), "s"(flag), "i"(A) : "memory");
}
template <int A> DI void acc_set(unsigned v) { asm volatile("s_nop 1\n\tv_accvgpr_write_b32 a[%c1], %0\n\ts_nop 1" :: "s"(v), "i"(A)); }
DI unsigned unit_src(unsigned u) { return (u >> 4) * 65536u + (u & 15u) * 256u; }
DI unsigned unit_flag(unsigned u) { return (((u >> 2) & 3u) * 128u + (u >> 4) * 4u + (u & 3u)) * 4u; }
template <int N, int A = 32> DI unsigned snd_wait_ticket() {
    unsigned v;
    asm volatile("s_waitcnt vmcnt(%1)\n\tv_accvgpr_read_b32 %0, a[%c2]\n\ts_nop 1" : "=v"(v) : "n"(N), "i"(A) : "memory");
    return __builtin_amdgcn_readfirstlane(v);
}

#define STAMP(k) do { } while (0)
#define GOSTAMP
#define BSTAMP(i) do { } while (0)

template <int S, int SLOT> DI void w_read(unsigned a0, unsigned a1, unsigned a2, unsigned a3) {
    constexpr int B = S * 32, O = SLOT * 8192;
#define TRR(c, t, A) "ds_read_b64_tr_b8 a[%c4+" #c "*8+" #t "*2:%c4+" #c "*8+" #t "*2+1], " A " offset:%c5+4096*(" #t "/2)+512*(" #t "%%2)\n\t"
    asm volatile(TRR(0, 0, "%0") TRR(0, 1, "%0") TRR(0, 2, "%0") TRR(0, 3, "%0")
                 TRR(1, 0, "%1") TRR(1, 1, "%1") TRR(1, 2, "%1") TRR(1, 3, "%1")
                 TRR(2, 0, "%2") TRR(2, 1, "%2") TRR(2, 2, "%2") TRR(2, 3, "%2")
                 TRR(3, 0, "%3") TRR(3, 1, "%3") TRR(3, 2, "%3") TRR(3, 3, "%3")
                 "s_waitcnt lgkmcnt(0)"
                 :: "v"(a0), "v"(a1), "v"(a2), "v"(a3), "i"(B), "i"(O) : "memory", ACLOB);
#undef TRR
}
DI void w_stage(__amdgpu_buffer_rsrc_t rs, unsigned ve, unsigned vo, unsigned m0base, unsigned soff) {
    unsigned keep, so;
#define WST(V) "s_nop 0\n\tbuffer_load_dwordx4 " V ", %5, %1 offen lds\n\t"
#define WNX "s_add_u32 m0, m0, 0x400\n\ts_add_u32 %1, %1, 0x40000\n\t"
    asm volatile("s_mov_b32 %0, m0\n\ts_mov_b32 m0, %2\n\ts_mov_b32 %1, %6\n\t"
                 WST("%3") WNX WST("%4") WNX WST("%3") WNX WST("%4") WNX WST("%3") WNX WST("%4") WNX WST("%3") WNX WST("%4")
                 "s_mov_b32 m0, %0"
                 : "=&s"(keep), "=&s"(so) : "s"(m0base), "v"(ve), "v"(vo), "s"(rs), "s"(soff) : "scc", "memory");
#undef WST
#undef WNX
    static_assert(16 * NFULL == 0x40000, "");
}
DI void x_stage(const unsigned char* base, unsigned vdst, unsigned m0base) {
    unsigned keep, v;
#define XST(r) "v_xor_b32 %1, " #r "*16, %3\n\tv_add_u32 %1, " #r "*4096, %1\n\tglobal_load_lds_dwordx4 %1, %4\n\ts_add_u32 m0, m0, 0x400\n\t"
    asm volatile("s_mov_b32 %0, m0\n\ts_mov_b32 m0, %2\n\t"
                 XST(0) XST(1) XST(2) XST(3) XST(4) XST(5) XST(6) XST(7) XST(8) XST(9) XST(10) XST(11) XST(12) XST(13) XST(14) XST(15)
                 "s_mov_b32 m0, %0"
                 : "=&s"(keep), "=&v"(v) : "s"(m0base), "v"(vdst), "s"(base) : "scc", "memory");
#undef XST
}
template <int S, int C> DI void mf(f32x4& c, const u32x8& x, const f32x4& cinit) {
    constexpr int B = S * 32 + C * 8;
    if constexpr (S == 0) asm volatile("v_mfma_f32_16x16x128_f8f6f4 %0, a[%c3:%c3+7], %1, %2" : "+v"(c) : "v"(x), "v"(cinit), "i"(B));
    else asm volatile("v_mfma_f32_16x16x128_f8f6f4 %0, a[%c2:%c2+7], %1, %0" : "+v"(c) : "v"(x), "i"(B));
}
DI float fadd(float a, float b) { float r; asm("v_add_f32_e32 %0, %1, %2" : "=v"(r) : "v"(a), "v"(b)); return r; }
DI float fmul(float a, float b) { float r; asm("v_mul_f32_e32 %0, %1, %2" : "=v"(r) : "v"(a), "v"(b)); return r; }
DI f32x4 fadd4(f32x4 a, f32x4 b) { return (f32x4){fadd(a[0], b[0]), fadd(a[1], b[1]), fadd(a[2], b[2]), fadd(a[3], b[3])}; }
DI f32x4 fmul4(f32x4 a, float b) { return (f32x4){fmul(a[0], b), fmul(a[1], b), fmul(a[2], b), fmul(a[3], b)}; }

constexpr int S_K1 = 4, S_BAR = 6, S_RD = 8, S_K2 = 10, S_A1 = 12, S_A2 = 14, S_POLL = SPOLL, S_A3 = 18, S_MU = 20, S_ST = 22, S_K3 = 24, S_K4 = 26, S_K5 = 28, S_K6 = 30;

struct XBlk { u32x4 h[16]; };

__global__ __launch_bounds__(256, 1) void fused_kernel(Params p) {
    __shared__ __attribute__((aligned(16))) unsigned char lds[4 * 32768 + 32768];
    const int lane = threadIdx.x & 63;
    const int wid = __builtin_amdgcn_readfirstlane(threadIdx.x >> 6);
    const int rank = p.rank;
    char* const me = p.wsr[7];
    const int bid = blockIdx.x;
    code_prefetch(lane);
    const bool sender = bid < 24 || (bid < 32 && (bid & 2));
    const int w = (bid < 24 ? bid : 24 + (bid & 1) + ((bid >> 1) & 2)) * 4 + wid, pi = w % 7;
    char* const pws = p.wsr[pi];
    asm volatile("" :: "s"(p.x), "s"(p.w), "s"(me), "s"(rank), "s"(pws));
    STAMP(0);

    char* const callp = me + WS_CALL + (bid * 4 + wid) * 4;

    const unsigned vsrc = (lane & 15) * KDIM + (lane >> 4) * 16, vdst = lane * 16;

    if (__builtin_expect(sender, 1)) {
        const int j = w / 7;
        unsigned vzero = 0;
        asm volatile("v_accvgpr_write_b32 a33, 1" : "+v"(vzero) :: "a0", "a31", "a32", "a33", "a35", "a36", "a37", "a38", "a43");
        auto srcp = [&](unsigned u) { return p.x + unit_src(u < NUNIT ? u : NUNIT - 1); };
        unsigned tk0 = j, tk1 = j + WPP, prev;
        char* const ctr = me + WS_TICK + pi * 128;
        call_load<35>(callp, vzero);
        snd_load<0>(srcp(tk0), vsrc);
        snd_ticket<32>(ctr, vzero);
        snd_load<16>(srcp(tk1), vsrc);
        const unsigned call = snd_wait_ticket<5, 35>() + 1u;
        char* const dst = pws + WS_X + (size_t)((call & 1u) * NR + rank) * SHARD;
        STAMP(2);
        snd_store<0>(dst + tk0 * 4096u, vdst);
        acc_set<40>(call); acc_set<41>(call); acc_set<42>(call); acc_set<43>(call);
        const unsigned tbase = (call - 1u) * NUNIT - 2u * WPP;
        char* const fl = pws + WS_FLAG + rank * 2048;
        char* const own = me + WS_X + (size_t)((call & 1u) * NR + rank) * SHARD;
        prev = tk0; tk0 = tk1; tk1 = snd_wait_ticket<8>() - tbase;
        int fill = 0;
        char* const cnt = me + WS_CNT + pi * (128 * 64);
        auto filler = [&]() { asm volatile("global_load_dword a38, %0, %1" :: "v"(vzero), "s"(p.x) : "memory"); };
        unsigned pp = NUNIT;
        auto publish = [&](unsigned u) {
            if (fill == 4) filler(); else if (u < NAGG) snd_ticket<37>(cnt + (u >> 2) * 64, vzero); else snd_flag<40>(fl + unit_flag(u), vzero);
        };
        auto publish4 = [&](unsigned u) {
            unsigned c; asm volatile("v_accvgpr_read_b32 %0, a37\n\ts_nop 1" : "=v"(c));
            if (u < NAGG && fill != 4 && (__builtin_amdgcn_readfirstlane(c) & 3) == 3) snd_flag4<40>(fl + unit_flag(u & ~3u), vzero); else filler();
        };
        const char* const rtr = me + WS_RTR + ((unsigned)(rank + 1 + pi) & 7u) * 128;
#define SEND_STEP(BA, BB, NLOAD) { \
            if (tk0 >= NUNIT - 16 * NSPEC - 64 && fill == 0) { asm volatile("global_load_dword a36, %0, %1 sc0 sc1" :: "v"((lane & 31) * 4), "s"(rtr) : "memory"); fill = 1; } \
            snd_ticket<32>(ctr, vzero); \
            snd_load<BB>(srcp(tk1), vsrc); \
            wait_vm<NLOAD>();                                            \
            if (tk0 >= NUNIT - 16 * NSPEC && (fill & 1)) { \
                const unsigned long long t0 = __builtin_amdgcn_s_memrealtime(); \
                for (fill = 2;;) { \
                    unsigned f, tmo; \
                    asm volatile("global_load_dword %0, %2, %3 sc0 sc1\n\tglobal_load_dword %1, %4, %5 sc1\n\ts_waitcnt vmcnt(0)" \
                                 : "=&v"(f), "=&v"(tmo) : "v"((lane & 31) * 4), "s"(rtr), "v"(vzero), "s"(me + WS_TMO) : "memory"); \
                    if (__all(f >= call)) break; \
                    if (__builtin_amdgcn_readfirstlane(tmo) == call || __builtin_amdgcn_s_memrealtime() - t0 > DEADLINE_TICKS) { snd_flag<40>(me + WS_TMO, vzero); fill = 4; break; } \
                    __builtin_amdgcn_s_sleep(4); \
                } \
            } \
            if (TESTREDO && tk0 >= NUNIT - 16 * NSPEC && (tk0 & 3) == 1) { __builtin_amdgcn_s_sleep(127); __builtin_amdgcn_s_sleep(127); } \
            snd_store<BA>((fill == 4 ? own : dst) + tk0 * 4096u, vdst); \
            const unsigned tk2 = snd_wait_ticket<8>() - tbase;           \
            if (fill == 1) { unsigned f; asm volatile("v_accvgpr_read_b32 %0, a36" : "=v"(f)); fill = __all(f >= call) ? 2 : 3; } \
            publish4(pp); publish(prev); pp = prev; \
            if ((prev & 48) == 0 && prev < 448) STAMP(3 + (prev >> 6)); \
            prev = tk0; tk0 = tk1; tk1 = tk2; }
        SEND_STEP(16, 0, 9)
        if (tk0 < NUNIT)
        for (;;) {
            SEND_STEP(0, 16, 11) if (tk0 >= NUNIT) break;
            SEND_STEP(16, 0, 11) if (tk0 >= NUNIT) break;
        }
#undef SEND_STEP
        wait_vm<0>();
        publish4(pp); publish(prev);
        if (prev < NAGG) { wait_vm<0>(); publish4(prev); }
        snd_flag<40>(callp, vzero);
        STAMP(1);
        return;
    }

    int cg, rg;
    {
        const int xcd = bid & 7, i = (xcd & 1) * 28 + (bid >> 3) - 4;
        rg = bid < 32 ? 7 : i >> 3;
        cg = bid < 32 ? p.slow + (xcd & 1) + (xcd >> 2) * 16 : (xcd >> 1) * 8 + (i & 7);
    }
    const int q = wid;
    for (int d = 0; d < WDELAY; d += 8) __builtin_amdgcn_s_sleep(8);
    const float scale = p.sx[0] * p.sw[0];
    unsigned call;
    const unsigned ldsw = (unsigned)(size_t)(LAS unsigned char*)lds + q * 32768u;
    const unsigned peer = (unsigned)(rank + 1 + rg) & 7u;
    const unsigned sfl = peer * 2048 + q * 512, sflo = rank * 2048 + q * 512, vfl = lane * 8;
    XBlk X0, X1;
    u32x2 pf;
    {
        const __amdgpu_buffer_rsrc_t rw = __builtin_amdgcn_make_buffer_rsrc((void*)p.w, 0, (unsigned)KDIM * NFULL, 0x00020000);
        const unsigned ve = (lane >> 2) * NFULL + (((lane & 3) ^ ((lane >> 4) & 1)) * 16), vo = ve ^ 32u;
        const unsigned so = (unsigned)(q * 1024) * NFULL + rank * NSH + cg * 64;
        const unsigned ab = ldsw + ((lane >> 4) * 16 + ((lane & 15) >> 1)) * 64 + (lane & 1) * 8, f = (lane >> 3) & 3;
        const unsigned a0 = ab + ((0 ^ f) << 4), a1 = ab + ((1 ^ f) << 4), a2 = ab + ((2 ^ f) << 4), a3 = ab + ((3 ^ f) << 4);
        constexpr unsigned SS = 128u * NFULL;
        unsigned vzero = 0;
        asm volatile("" : "+v"(vzero));
        call_load<32>(callp, vzero);
#define WOFF(S) (so + (unsigned)(((S) + ROT) & 7) * SS)
#define WFIRST w_stage(rw, ve, vo, ldsw + 0 * 8192, WOFF(0)); w_stage(rw, ve, vo, ldsw + 1 * 8192, WOFF(1)); \
               w_stage(rw, ve, vo, ldsw + 2 * 8192, WOFF(2)); w_stage(rw, ve, vo, ldsw + 3 * 8192, WOFF(3));
        const bool copier = rg >= 3 && rg < 7;
        const unsigned uo = (unsigned)(((rg - 3) * 32 + cg) * 4 + q);
        if (copier) snd_load<224>(p.x + unit_src(uo), vsrc);
        if (rg < 7) {
            const unsigned l0 = (unsigned)(((bid >> 3) - 4) * 4 + q) * 64u + lane, cb = rank * NSH + (cg & 24) * 64;
            const unsigned l2 = l0 + 2 * 7168u;
            asm volatile("buffer_load_dword a249, %0, %3, %4 offen\n\tbuffer_load_dword a250, %1, %3, %4 offen\n\tbuffer_load_dword a251, %2, %3, %4 offen"
                         :: "v"((l0 >> 2) * NFULL + (l0 & 3) * 128u), "v"(((l0 + 7168u) >> 2) * NFULL + (l0 & 3) * 128u),
                            "v"(((l2 < 16384u ? l2 : l0) >> 2) * NFULL + (l0 & 3) * 128u), "s"(rw), "s"(cb) : "memory", "a249", "a250", "a251");
        }
        if (copier) {
            call = snd_wait_ticket<TOUCHFIRST ? 0 : 3 * WTOUCH>() + 1u;
            acc_set<240>(call);
            snd_store<224>(me + WS_X + (size_t)((call & 1u) * NR + rank) * SHARD + uo * 4096u, vdst);
            WFIRST
            wait_vm<32>();
            snd_flag<240>(me + WS_FLAG + rank * 2048 + unit_flag(uo), vzero);
            wait_vm<24>();
        } else {
            WFIRST
            call = snd_wait_ticket<24>() + 1u;
        }
#undef WFIRST
        if (rg < 7) {
            const u32x4 ff = {~0u, ~0u, ~0u, ~0u};
            char* const pz = me + WS_X + (size_t)((call & 1u) * NR + peer) * SHARD + (NBLK - NSPEC) * 65536;
#pragma unroll
            for (int k = 0; k < (NSPEC + 1) / 2; ++k)
                if (k * 128 + cg * 4 + q < NSPEC * 64)
                    asm volatile("global_store_dwordx4 %0, %1, %2 sc0 sc1\n\ts_nop 1" :: "v"(vdst), "v"(ff), "s"(pz + (k * 128 + cg * 4 + q) * 1024) : "memory");
        }
        w_read<0, 0>(a0, a1, a2, a3); w_stage(rw, ve, vo, ldsw + 0 * 8192, WOFF(4));
        wait_vm<24>(); w_read<1, 1>(a0, a1, a2, a3); w_stage(rw, ve, vo, ldsw + 1 * 8192, WOFF(5));
        wait_vm<24>(); w_read<2, 2>(a0, a1, a2, a3); w_stage(rw, ve, vo, ldsw + 2 * 8192, WOFF(6));
        wait_vm<24>(); w_read<3, 3>(a0, a1, a2, a3); w_stage(rw, ve, vo, ldsw + 3 * 8192, WOFF(7));
#undef WOFF
        wait_vm<24>(); w_read<4, 0>(a0, a1, a2, a3);
        wait_vm<16>(); w_read<5, 1>(a0, a1, a2, a3);
        pf = __builtin_bit_cast(u32x2, __builtin_amdgcn_raw_buffer_load_b64(__builtin_amdgcn_make_buffer_rsrc(me + WS_FLAG, 0, NR * 2048, 0x00020000), vfl, sfl, 17));
        x_stage(p.x + rg * 65536 + q * 1024, vdst, ldsw);
        wait_vm<8 + 17>(); w_read<6, 2>(a0, a1, a2, a3);
        wait_vm<17>();     w_read<7, 3>(a0, a1, a2, a3);
        wait_vm<0>();
        __builtin_amdgcn_s_barrier();
        asm volatile("s_mov_b64 exec, 1\n\tglobal_store_dword %0, %1, %2 sc0 sc1\n\ts_mov_b64 exec, -1" :: "v"(vzero), "v"(call), "s"(callp) : "memory");
        if (q == 0 && rg < 7) asm volatile("s_mov_b64 exec, 1\n\tglobal_store_dword %0, %1, %2 sc0 sc1\n\ts_mov_b64 exec, -1" :: "v"(vzero), "v"(call), "s"(p.wsr[rg] + WS_RTR + rank * 128 + cg * 4) : "memory");
        const unsigned xb = q * 32768 + (lane & 15) * 1024 + (((lane >> 4) ^ (lane & 3)) << 4), xs = (lane & 12) << 4;
#pragma unroll
        for (int k = 0; k < 16; ++k) X1.h[k] = *(const LAS u32x4*)((LAS unsigned char*)lds + xb + (((unsigned)((k + 2 * ROT) & 15) << 6) ^ xs));
    }
    STAMP(1);
    static_assert(__builtin_popcountll(OWNMASK) == 5 && (OWNMASK & 3) == 1 && (OWNMASK >> 32) == 0, "");
    const bool slowcg = (cg & 14) == p.slow;
    const int n = rg == 7 ? NBLK - 7 : slowcg ? NBLK + 1 : NBLK + 5, ostep = rg == 7 ? 1 : 7;
    const unsigned ownpos = rg == 7 ? ~0u : slowcg ? 1u : (unsigned)OWNMASK;
    unsigned tabx; int tabr;
    {
        const int before = __builtin_amdgcn_mbcnt_lo(ownpos, 0u);
        const bool isown = lane < 32 && ((ownpos >> (lane & 31)) & 1u);
        const int b = isown ? rg + ostep * before : lane - before;
        tabx = (((isown ? rank : peer) * NBLK + (b & 31)) << 16) | (b < NBLK ? 0u : 1u);
        tabr = isown || b >= NBLK ? -1 : b;
    }
    void* const xall = me + WS_X + (size_t)(call & 1u) * NR * SHARD;
    const unsigned orow = q * 4 + (lane >> 4), voffO = orow * (NSH * 4) + (cg * 64 + (lane & 15) * 4) * 4;
    const unsigned lwb = 131072 + q * 4096 + (lane & 15) * 256 + (((lane >> 4) ^ (lane & 15)) << 4);
    LAS unsigned char* const lw0 = (LAS unsigned char*)lds + lwb; LAS unsigned char* const lw1 = (LAS unsigned char*)lds + (lwb ^ 64);
    LAS unsigned char* const lw2 = (LAS unsigned char*)lds + (lwb ^ 128); LAS unsigned char* const lw3 = (LAS unsigned char*)lds + (lwb ^ 192);
    LAS unsigned char* const lr = (LAS unsigned char*)lds + 131072 + orow * 256 + (((lane & 15) ^ orow) << 4);
    unsigned xlim = NR * SHARD;
    f32x4 cinit = {0.f, 0.f, 0.f, 0.f};
    unsigned xsz = NR * SHARD;

    auto count = [&](u32x2 f) { unsigned r; asm("s_ff1_i32_b64 %0, %1" : "=s"(r) : "s"(__ballot(f.x < call || f.y < call))); return (int)(r >> 1); };
    auto spin = [&](unsigned soff, int need) {
        const char* const pf = me + WS_FLAG + soff;
        const unsigned long long t0 = __builtin_amdgcn_s_memrealtime();
        asm volatile("" :: "s"(t0));
        for (;;) {
            u32x2 f; unsigned tmo;
            asm volatile("global_load_dwordx2 %0, %2, %3 sc0 sc1\n\tglobal_load_dword %1, %4, %5 sc1\n\ts_waitcnt vmcnt(0)"
                         : "=&v"(f), "=&v"(tmo) : "v"(vfl), "s"(pf), "v"(0u), "s"(me + WS_TMO) : "memory");
            const int c = count(f);
            if (c > need) return c;
            if (__builtin_amdgcn_readfirstlane(tmo) == call || __builtin_amdgcn_s_memrealtime() - t0 > DEADLINE_TICKS) {
                if (lane == 0) __hip_atomic_store((unsigned*)(me + WS_TMO), call, __ATOMIC_RELAXED, __HIP_MEMORY_SCOPE_AGENT);
                xlim = 0; xsz = 0; const float qn = __builtin_nanf(""); cinit = (f32x4){qn, qn, qn, qn};
                asm volatile("s_nop 1" : "+v"(cinit));
                return 2 * NBLK;
            }
            __builtin_amdgcn_s_sleep(1);
        }
    };
    unsigned voffR[8];
#pragma unroll
    for (int k = 0; k < 8; ++k) voffR[k] = vdst + q * 16384 + (unsigned)((k + ROT) & 7) * 2048u;
    f32x4 c0 = cinit, c1 = cinit, c2 = cinit, c3 = cinit;
    asm volatile("s_nop 1" : "+v"(cinit), "+v"(c0), "+v"(c1), "+v"(c2), "+v"(c3));
#define XLOAD(X, k) (X).h[k] = __builtin_bit_cast(u32x4, __builtin_amdgcn_raw_buffer_load_b128(rx, voffR[(k) >> 1], sx + ((k) & 1) * 1024u, 17))
#define POLL(size, off) __builtin_bit_cast(u32x2, __builtin_amdgcn_raw_buffer_load_b64(__builtin_amdgcn_make_buffer_rsrc(me + WS_FLAG, 0, size, 0x00020000), vfl, off, 17))
    int rdy = rg == 7 ? 2 * NBLK : count(pf);
    unsigned xm = 1u, x0 = __builtin_amdgcn_readlane(tabx, 0), x1 = __builtin_amdgcn_readlane(tabx, 1);
    int r1 = __builtin_amdgcn_readlane(tabr, 1), need = r1 + 1, thr = need;
    int sp0 = NBLK;
    unsigned sx, oso = 0, osz = 0, psz = NR * 2048;
    auto store_rows = [&](const f32x4& r) {
        __builtin_amdgcn_raw_buffer_store_b128(__builtin_bit_cast(u32x4, r), __builtin_amdgcn_make_buffer_rsrc((void*)p.out, 0, NOSTORE ? 0u : osz, 0x00020000), voffO, oso, OUT_AUX);
    };
#define SB __builtin_amdgcn_sched_barrier(0);
#define XFRAG(X, S) __builtin_shufflevector((X).h[2 * (S)], (X).h[2 * (S) + 1], 0, 1, 2, 3, 4, 5, 6, 7)
#define PIN(v) asm volatile("" : "+s"(v));
#define SLOT(FIRST, CUR, NXT, PP, PO, PN, J, C) \
        mf<(J) / 4, C>(c##C, XFRAG(CUR, (J) / 4), cinit); SB \
        if constexpr ((J) == 0) *(LAS f32x4*)(lw2 + PP * 16384) = c2; \
        if constexpr ((J) == 1) *(LAS f32x4*)(lw3 + PP * 16384) = c3; \
        if constexpr ((J) == 2) *(LAS f32x4*)(lw0 + PP * 16384) = c0; \
        if constexpr ((J) == 31) *(LAS f32x4*)(lw1 + (1 - PP) * 16384) = c1; \
        if constexpr ((J) == S_POLL) PN = POLL(psz, sfl); \
        if constexpr (FIRST && (J) == S_POLL - 2) pf = POLL(NR * 2048, sflo); \
        if constexpr ((J) & 1) XLOAD(NXT, (J) >> 1); \
        if constexpr ((J) == S_BAR) { wait_lgkm0(); __builtin_amdgcn_s_barrier(); asm volatile("" ::: "memory"); } \
        if constexpr ((J) == S_RD) { q0 = *(const LAS f32x4*)(lr + PP * 16384); q1 = *(const LAS f32x4*)(lr + PP * 16384 + 4096); \
                                     q2 = *(const LAS f32x4*)(lr + PP * 16384 + 8192); q3 = *(const LAS f32x4*)(lr + PP * 16384 + 12288); } \
        if constexpr ((J) == S_A1) q1 = fadd4(q0, q1); \
        if constexpr ((J) == S_A2) q2 = fadd4(q2, q3); \
        if constexpr ((J) == S_A3) q1 = fadd4(q1, q2); \
        if constexpr ((J) == S_MU) q1 = fmul4(q1, scale); \
        if constexpr ((J) == S_ST) store_rows(q1); \
        if constexpr ((J) == S_K1) { PIN(r1) sp1 = r1 >= rdy ? r1 : NBLK; PIN(sp1) PIN(i) i2 = i + 2; PIN(i2) } \
        if constexpr ((J) == S_K2) { PIN(xm) oso = xm << 1; osz = (xm & 1u) ? 0u : (unsigned)(NR * MSH) * NSH * 4u; PIN(oso) PIN(osz) } \
        if constexpr ((J) == S_K3) { PIN(i2) xm = x0; x0 = x1; x1 = __builtin_amdgcn_readlane(tabx, i2); r1 = __builtin_amdgcn_readlane(tabr, i2); PIN(x1) PIN(r1) } \
        if constexpr ((J) == S_K3 + 1) { PIN(x1) xszn = (x1 & 1u) ? 0u : xlim; PIN(xszn) } \
        if constexpr ((J) == S_K4) { PIN(r1) int r2 = r1 + 1; PIN(r2) need = r1 < NBLK - NSPEC ? r2 : r1; PIN(need) } \
        if constexpr ((J) == S_K4 + 1) { PIN(need) thr = sp0 < NBLK ? -1 : need; BSTAMP(i); ++i; PIN(thr) PIN(i) } \
        if constexpr ((J) == S_K5) { asm volatile("" : "+v"(PO)); const int c = count(PO); rdy = c > rdy ? c : rdy; PIN(rdy) }     \
        if constexpr ((J) == S_K6) { PIN(need) psz = rdy < need + POLL_AHEAD ? NR * 2048 : 0u; PIN(psz) } \
        SB
#define STEP4(FIRST, CUR, NXT, PP, PO, PN, S) SLOT(FIRST, CUR, NXT, PP, PO, PN, 4 * S, 1) SLOT(FIRST, CUR, NXT, PP, PO, PN, 4 * S + 1, 2) SLOT(FIRST, CUR, NXT, PP, PO, PN, 4 * S + 2, 3) SLOT(FIRST, CUR, NXT, PP, PO, PN, 4 * S + 3, 0)
#define RECHECK(PP) \
        if (spm < NBLK) { \
            asm volatile("s_nop 7\n\ts_nop 7" : "+v"(c0)); \
            if (__any(c0[0] != c0[0])) { \
                rdy = spin(sfl, spm); \
                const __amdgpu_buffer_rsrc_t rx = __builtin_amdgcn_make_buffer_rsrc(xall, 0, xlim, 0x00020000); \
                sx = xm; \
                XBlk T; \
                _Pragma("unroll") for (int k = 0; k < 16; ++k) XLOAD(T, k); \
                M4(T, 0) M4(T, 1) M4(T, 2) M4(T, 3) M4(T, 4) M4(T, 5) M4(T, 6) M4(T, 7) \
                asm volatile("s_nop 7\n\ts_nop 7" : "+v"(c0), "+v"(c1), "+v"(c2), "+v"(c3)); \
                *(LAS f32x4*)(lw1 + PP * 16384) = c1; \
                STAMP(48); \
            } \
        }
#define M4(X, S) mf<S, 1>(c1, XFRAG(X, S), cinit); mf<S, 2>(c2, XFRAG(X, S), cinit); mf<S, 3>(c3, XFRAG(X, S), cinit); mf<S, 0>(c0, XFRAG(X, S), cinit);
#define BODY(FIRST, CUR, NXT, PP, PO, PN) { \
        if (__builtin_expect((unsigned)rdy < (unsigned)thr, 0)) { \
            RECHECK(PP) \
            if (rdy < need) { const int c = count(PO); rdy = c > rdy ? c : rdy; }     \
            if (rdy < need) rdy = spin(sfl, need - 1); \
        } \
        GOSTAMP \
        sx = x1; \
        const __amdgpu_buffer_rsrc_t rx = __builtin_amdgcn_make_buffer_rsrc(xall, 0, xsz, 0x00020000); \
        f32x4 q0, q1, q2, q3; unsigned xszn; int sp1, i2; \
        STEP4(FIRST, CUR, NXT, PP, PO, PN, 0) STEP4(FIRST, CUR, NXT, PP, PO, PN, 1) STEP4(FIRST, CUR, NXT, PP, PO, PN, 2) STEP4(FIRST, CUR, NXT, PP, PO, PN, 3) \
        STEP4(FIRST, CUR, NXT, PP, PO, PN, 4) STEP4(FIRST, CUR, NXT, PP, PO, PN, 5) STEP4(FIRST, CUR, NXT, PP, PO, PN, 6) STEP4(FIRST, CUR, NXT, PP, PO, PN, 7) \
        xsz = xszn; spm = sp0; sp0 = sp1; }

    int i = 0, spm = NBLK;
    u32x2 P0 = {0u, 0u}, P1;
    if (rg == 7) spin(sflo, NBLK - 1);
    BODY(true, X1, X0, 1, P0, P1)
    if (ownpos != 1u && count(pf) < NBLK) spin(sflo, NBLK - 1);
    do { BODY(false, X0, X1, 0, P1, P0) BODY(false, X1, X0, 1, P0, P1) } while (i < n);
    RECHECK(0)
    oso = xm << 1; osz = (unsigned)(NR * MSH) * NSH * 4u;
    {
        asm volatile("s_nop 7\n\ts_nop 7" : "+v"(c0), "+v"(c2), "+v"(c3));
        *(LAS f32x4*)(lw2 + 0 * 16384) = c2; *(LAS f32x4*)(lw3 + 0 * 16384) = c3; *(LAS f32x4*)(lw0 + 0 * 16384) = c0;
        wait_lgkm0(); __builtin_amdgcn_s_barrier(); asm volatile("" ::: "memory");
        store_rows(fmul4(fadd4(fadd4(*(const LAS f32x4*)(lr + 0 * 16384), *(const LAS f32x4*)(lr + 0 * 16384 + 4096)), fadd4(*(const LAS f32x4*)(lr + 0 * 16384 + 8192), *(const LAS f32x4*)(lr + 0 * 16384 + 12288))), scale));
        STAMP(39);
        asm volatile(".Lfk_code_end:");
    }
#undef BODY
#undef M4
#undef RECHECK
#undef STEP4
#undef SLOT
#undef XFRAG
#undef SB
#undef XLOAD
#undef POLL
#undef PIN
}
}

extern "C" void kernel_launch(int rank, int nranks, ncclComm_t comm,
                              hipStream_t stream,
                              void* const* d_in, const int* in_sizes, int n_in,
                              void* d_out, int out_size,
                              const int* in_dtypes, int out_dtype,
                              void* const* d_out_all,
                              void* const* d_ws_all, size_t ws_bytes) {
    (void)comm; (void)in_dtypes; (void)out_dtype; (void)d_out_all;
    if (nranks != NR || (unsigned)rank >= (unsigned)NR || n_in != 4 || ws_bytes < WS_END ||
        in_sizes[0] != MSH * KDIM || in_sizes[1] != KDIM * NFULL || out_size != NR * MSH * NSH) {
        fprintf(stderr, "kernel_launch: unexpected problem shape\n");
        return;
    }
    Params p;
    p.x = (const unsigned char*)d_in[0]; p.w = (const unsigned char*)d_in[1];
    p.sx = (const float*)d_in[2]; p.sw = (const float*)d_in[3]; p.out = (float*)d_out;
    for (int i = 0; i < NR; ++i) p.wsr[i] = (char*)d_ws_all[(rank + 1 + i) & 7];
    p.rank = rank; p.slow = (int)((3u - (unsigned)((size_t)p.w >> 7)) & 7u) * 2;
    fused_kernel<<<256, 256, 0, stream>>>(p);
    HIP_CHECK(hipGetLastError());
}
```

```cpp
#include <hip/hip_runtime.h>
#include <rccl/rccl.h>
#include <cstdio>

#define HIP_CHECK(expr) do { hipError_t _e = (expr); if (_e != hipSuccess) { \
    fprintf(stderr, "HIP error %s at %s:%d\n", hipGetErrorString(_e), __FILE__, __LINE__); \
    return; } } while (0)

namespace {
typedef unsigned u32x2 __attribute__((ext_vector_type(2)));
typedef unsigned u32x4 __attribute__((ext_vector_type(4)));
typedef unsigned u32x8 __attribute__((ext_vector_type(8)));
typedef float f32x4 __attribute__((ext_vector_type(4)));
#define LAS __attribute__((address_space(3)))
#define DI __device__ __forceinline__

constexpr int NR = 8;
constexpr int KDIM = 4096, NFULL = 16384, NSH = 2048, MSH = 512;
constexpr unsigned SHARD = MSH * KDIM;
constexpr int NBLK = MSH / 16;
constexpr int NUNIT = 512;
constexpr int WPP = 16;
constexpr size_t WS_X = 0;
constexpr size_t WS_FLAG = 2ull * NR * SHARD;
constexpr size_t WS_RTR = WS_FLAG + NR * 2048;
constexpr size_t WS_TICK = WS_RTR + NR * 128;
constexpr size_t WS_TMO = WS_TICK + 1024;
constexpr size_t WS_CALL = WS_TMO + 1024;
constexpr size_t WS_CNT = WS_CALL + 1024 * 4;
constexpr size_t WS_DBG = WS_CNT + 7 * 128 * 64;
constexpr size_t WS_END = WS_DBG + 2 * 256 * 64 * 8;
constexpr unsigned long long DEADLINE_TICKS = 150000000ull;

#define OWNMASK 0x155ull
#define POLL_AHEAD 3
#define NSPEC 8
#define TESTREDO 0
#define NOSTORE 0
#define NOSLOW 0
#define ROT rg
#define WDELAY 0
#define NAGG (26 * 16)
#define TOUCHPOS 0
#define SPIN_SLEEP 1
#define LATEFILL 1
#define SPOLL 16
#define OUT_AUX 2

struct Params {
    const unsigned char* x; const unsigned char* w; const float* sx; const float* sw; float* out;
    char* wsr[NR];
    int rank; int slow;
};

#define ACLOB "a0","a1","a2","a3","a4","a5","a6","a7","a8","a9","a10","a11","a12","a13","a14","a15","a16","a17","a18","a19","a20","a21","a22","a23","a24","a25","a26","a27","a28","a29","a30","a31","a32","a33","a34","a35","a36","a37","a38","a39","a40","a41","a42","a43","a44","a45","a46","a47","a48","a49","a50","a51","a52","a53","a54","a55","a56","a57","a58","a59","a60","a61","a62","a63","a64","a65","a66","a67","a68","a69","a70","a71","a72","a73","a74","a75","a76","a77","a78","a79","a80","a81","a82","a83","a84","a85","a86","a87","a88","a89","a90","a91","a92","a93","a94","a95","a96","a97","a98","a99","a100","a101","a102","a103","a104","a105","a106","a107","a108","a109","a110","a111","a112","a113","a114","a115","a116","a117","a118","a119","a120","a121","a122","a123","a124","a125","a126","a127","a128","a129","a130","a131","a132","a133","a134","a135","a136","a137","a138","a139","a140","a141","a142","a143","a144","a145","a146","a147","a148","a149","a150","a151","a152","a153","a154","a155","a156","a157","a158","a159","a160","a161","a162","a163","a164","a165","a166","a167","a168","a169","a170","a171","a172","a173","a174","a175","a176","a177","a178","a179","a180","a181","a182","a183","a184","a185","a186","a187","a188","a189","a190","a191","a192","a193","a194","a195","a196","a197","a198","a199","a200","a201","a202","a203","a204","a205","a206","a207","a208","a209","a210","a211","a212","a213","a214","a215","a216","a217","a218","a219","a220","a221","a222","a223","a224","a225","a226","a227","a228","a229","a230","a231","a232","a233","a234","a235","a236","a237","a238","a239","a240","a241","a242","a243","a244","a245","a246","a247","a248","a249","a250","a251","a252","a253","a254","a255"

template <int N> DI void wait_vm() { asm volatile("s_waitcnt vmcnt(%0)" :: "n"(N) : "memory"); }
DI void wait_lgkm0() { asm volatile("s_waitcnt lgkmcnt(0)" ::: "memory"); }
template <int B> DI void snd_load(const unsigned char* base, unsigned voff) {
    asm volatile("global_load_dwordx4 a[%c2:%c2+3], %0, %1\n\t"
                 "global_load_dwordx4 a[%c2+4:%c2+7], %0, %1 offset:64\n\t"
                 "global_load_dwordx4 a[%c2+8:%c2+11], %0, %1 offset:128\n\t"
                 "global_load_dwordx4 a[%c2+12:%c2+15], %0, %1 offset:192" :: "v"(voff), "s"(base), "i"(B) : "memory");
}
template <int B> DI void snd_store(char* base, unsigned voff) {
    asm volatile("global_store_dwordx4 %0, a[%c2:%c2+3], %1 sc0 sc1\n\t"
                 "global_store_dwordx4 %0, a[%c2+4:%c2+7], %1 offset:1024 sc0 sc1\n\t"
                 "global_store_dwordx4 %0, a[%c2+8:%c2+11], %1 offset:2048 sc0 sc1\n\t"
                 "global_store_dwordx4 %0, a[%c2+12:%c2+15], %1 offset:3072 sc0 sc1\n\ts_nop 1" :: "v"(voff), "s"(base), "i"(B) : "memory");
}
template <int A> DI void snd_ticket(char* ctr, unsigned vzero) {
    unsigned long long keep;
    asm volatile("s_mov_b64 %0, exec\n\ts_mov_b64 exec, 1\n\tglobal_atomic_add a[%c3], %1, a33, %2 sc0\n\ts_mov_b64 exec, %0"
                 : "=&s"(keep) : "v"(vzero), "s"(ctr), "i"(A) : "memory");
}
DI void code_prefetch(unsigned lane) {
    unsigned long long pc; unsigned t0, t1, t2;
    asm volatile("s_getpc_b64 %0\n"
                 ".Lfk_code_begin:\n\t"
                 "v_min_u32 %1, .Lfk_code_end-.Lfk_code_begin, %4\n\tv_min_u32 %2, .Lfk_code_end-.Lfk_code_begin, %5\n\tv_min_u32 %3, .Lfk_code_end-.Lfk_code_begin, %6\n\t"
                 "global_load_dword a252, %1, %0\n\tglobal_load_dword a253, %2, %0\n\tglobal_load_dword a254, %3, %0"
                 : "=&s"(pc), "=&v"(t0), "=&v"(t1), "=&v"(t2) : "v"(lane * 128u), "v"(lane * 128u + 8192u), "v"(lane * 128u + 16384u) : "memory", "a252", "a253", "a254");
}
template <int B, int H> DI void snd_half(char* base, unsigned voff) {
    asm volatile("global_store_dwordx4 %0, a[%c2:%c2+3], %1 offset:%c3 sc0 sc1\n\t"
                 "global_store_dwordx4 %0, a[%c2+4:%c2+7], %1 offset:%c3+1024 sc0 sc1\n\ts_nop 1" :: "v"(voff), "s"(base), "i"(B + 8 * H), "i"(2048 * H) : "memory");
}
template <int A> DI void call_load(const char* ctr, unsigned vzero) { asm volatile("global_load_dword a[%c2], %0, %1 sc0 sc1" :: "v"(vzero), "s"(ctr), "i"(A) : "memory"); }
template <int A> DI void snd_flag(char* flag, unsigned vzero) {
    unsigned long long keep;
    asm volatile("s_mov_b64 %0, exec\n\ts_mov_b64 exec, 1\n\tglobal_store_dword %1, a[%c3], %2 sc0 sc1\n\ts_mov_b64 exec, %0"
                 : "=&s"(keep) : "v"(vzero), "s"(flag), "i"(A) : "memory");
}
template <int A> DI void snd_flag4(char* flag, unsigned vzero) {
    unsigned long long keep;
    asm volatile("s_mov_b64 %0, exec\n\ts_mov_b64 exec, 1\n\tglobal_store_dwordx4 %1, a[%c3:%c3+3], %2 sc0 sc1\n\ts_mov_b64 exec, %0"
                 : "=&s"(keep) : "v"(vzero), "s"(flag), "i"(A) : "memory");
}
template <int A> DI void acc_set(unsigned v) { asm volatile("s_nop 1\n\tv_accvgpr_write_b32 a[%c1], %0\n\ts_nop 1" :: "s"(v), "i"(A)); }
DI unsigned unit_src(unsigned u) { return (u >> 4) * 65536u + (u & 15u) * 256u; }
DI unsigned unit_flag(unsigned u) { return (((u >> 2) & 3u) * 128u + (u >> 4) * 4u + (u & 3u)) * 4u; }
template <int N, int A = 32> DI unsigned snd_wait_ticket() {
    unsigned v;
    asm volatile("s_waitcnt vmcnt(%1)\n\tv_accvgpr_read_b32 %0, a[%c2]\n\ts_nop 1" : "=v"(v) : "n"(N), "i"(A) : "memory");
    return __builtin_amdgcn_readfirstlane(v);
}

#define STAMP(k) do { } while (0)
#define GOSTAMP
#define BSTAMP(i) do { } while (0)

template <int S, int SLOT> DI void w_read(unsigned a0, unsigned a1, unsigned a2, unsigned a3) {
    constexpr int B = S * 32, O = SLOT * 8192;
#define TRR(c, t, A) "ds_read_b64_tr_b8 a[%c4+" #c "*8+" #t "*2:%c4+" #c "*8+" #t "*2+1], " A " offset:%c5+4096*(" #t "/2)+512*(" #t "%%2)\n\t"
    asm volatile(TRR(0, 0, "%0") TRR(0, 1, "%0") TRR(0, 2, "%0") TRR(0, 3, "%0")
                 TRR(1, 0, "%1") TRR(1, 1, "%1") TRR(1, 2, "%1") TRR(1, 3, "%1")
                 TRR(2, 0, "%2") TRR(2, 1, "%2") TRR(2, 2, "%2") TRR(2, 3, "%2")
                 TRR(3, 0, "%3") TRR(3, 1, "%3") TRR(3, 2, "%3") TRR(3, 3, "%3")
                 "s_waitcnt lgkmcnt(0)"
                 :: "v"(a0), "v"(a1), "v"(a2), "v"(a3), "i"(B), "i"(O) : "memory", ACLOB);
#undef TRR
}
DI void w_stage(__amdgpu_buffer_rsrc_t rs, unsigned ve, unsigned vo, unsigned m0base, unsigned soff) {
    unsigned keep, so;
#define WST(V) "s_nop 0\n\tbuffer_load_dwordx4 " V ", %5, %1 offen lds\n\t"
#define WNX "s_add_u32 m0, m0, 0x400\n\ts_add_u32 %1, %1, 0x40000\n\t"
    asm volatile("s_mov_b32 %0, m0\n\ts_mov_b32 m0, %2\n\ts_mov_b32 %1, %6\n\t"
                 WST("%3") WNX WST("%4") WNX WST("%3") WNX WST("%4") WNX WST("%3") WNX WST("%4") WNX WST("%3") WNX WST("%4")
                 "s_mov_b32 m0, %0"
                 : "=&s"(keep), "=&s"(so) : "s"(m0base), "v"(ve), "v"(vo), "s"(rs), "s"(soff) : "scc", "memory");
#undef WST
#undef WNX
    static_assert(16 * NFULL == 0x40000, "");
}
DI void x_stage(const unsigned char* base, unsigned vdst, unsigned m0base) {
    unsigned keep, v;
#define XST(r) "v_xor_b32 %1, " #r "*16, %3\n\tv_add_u32 %1, " #r "*4096, %1\n\tglobal_load_lds_dwordx4 %1, %4\n\ts_add_u32 m0, m0, 0x400\n\t"
    asm volatile("s_mov_b32 %0, m0\n\ts_mov_b32 m0, %2\n\t"
                 XST(0) XST(1) XST(2) XST(3) XST(4) XST(5) XST(6) XST(7) XST(8) XST(9) XST(10) XST(11) XST(12) XST(13) XST(14) XST(15)
                 "s_mov_b32 m0, %0"
                 : "=&s"(keep), "=&v"(v) : "s"(m0base), "v"(vdst), "s"(base) : "scc", "memory");
#undef XST
}
template <int S, int C> DI void mf(f32x4& c, const u32x8& x, const f32x4& cinit) {
    constexpr int B = S * 32 + C * 8;
    if constexpr (S == 0) asm volatile("v_mfma_f32_16x16x128_f8f6f4 %0, a[%c3:%c3+7], %1, %2" : "+v"(c) : "v"(x), "v"(cinit), "i"(B));
    else asm volatile("v_mfma_f32_16x16x128_f8f6f4 %0, a[%c2:%c2+7], %1, %0" : "+v"(c) : "v"(x), "i"(B));
}
DI float fadd(float a, float b) { float r; asm("v_add_f32_e32 %0, %1, %2" : "=v"(r) : "v"(a), "v"(b)); return r; }
DI float fmul(float a, float b) { float r; asm("v_mul_f32_e32 %0, %1, %2" : "=v"(r) : "v"(a), "v"(b)); return r; }
DI f32x4 fadd4(f32x4 a, f32x4 b) { return (f32x4){fadd(a[0], b[0]), fadd(a[1], b[1]), fadd(a[2], b[2]), fadd(a[3], b[3])}; }
DI f32x4 fmul4(f32x4 a, float b) { return (f32x4){fmul(a[0], b), fmul(a[1], b), fmul(a[2], b), fmul(a[3], b)}; }

constexpr int S_K1 = 4, S_BAR = 6, S_RD = 8, S_K2 = 10, S_A1 = 12, S_A2 = 14, S_POLL = SPOLL, S_A3 = 18, S_MU = 20, S_ST = 22, S_K3 = 24, S_K4 = 26, S_K5 = 28, S_K6 = 30;

struct XBlk { u32x4 h[16]; };

__global__ __launch_bounds__(256, 1) void fused_kernel(Params p) {
    __shared__ __attribute__((aligned(16))) unsigned char lds[4 * 32768 + 32768];
    const int lane = threadIdx.x & 63;
    const int wid = __builtin_amdgcn_readfirstlane(threadIdx.x >> 6);
    const int rank = p.rank;
    char* const me = p.wsr[7];
    const int bid = blockIdx.x;
    code_prefetch(lane);
    const bool sender = bid < 24 || (bid < 32 && (bid & 2));
    const int w = (bid < 24 ? bid : 24 + (bid & 1) + ((bid >> 1) & 2)) * 4 + wid, pi = w % 7;
    char* const pws = p.wsr[pi];
    asm volatile("" :: "s"(p.x), "s"(p.w), "s"(me), "s"(rank), "s"(pws));
    STAMP(0);

    char* const callp = me + WS_CALL + (bid * 4 + wid) * 4;

    const unsigned vsrc = (lane & 15) * KDIM + (lane >> 4) * 16, vdst = lane * 16;

    if (__builtin_expect(sender, 1)) {
        const int j = w / 7;
        unsigned vzero = 0;
        asm volatile("v_accvgpr_write_b32 a33, 1" : "+v"(vzero) :: "a32", "a33", "a35", "a36", "a37", "a38", "a43", "a111");
        auto srcp = [&](unsigned u) { return p.x + unit_src(u < NUNIT ? u : NUNIT - 1); };
        unsigned u0 = j, u1 = j + WPP, u2 = j + 2 * WPP, prev, pp = NUNIT;
        char* const ctr = me + WS_TICK + pi * 128;
        call_load<35>(callp, vzero);
        snd_load<64>(srcp(u0), vsrc);
        snd_load<80>(srcp(u1), vsrc);
        const unsigned call = snd_wait_ticket<4, 35>() + 1u;
        char* const dst = pws + WS_X + (size_t)((call & 1u) * NR + rank) * SHARD;
        STAMP(2);
        snd_half<64, 0>(dst + u0 * 4096u, vdst);
        snd_ticket<32>(ctr, vzero);
        snd_load<96>(srcp(u2), vsrc);
        snd_half<64, 1>(dst + u0 * 4096u, vdst);
        acc_set<40>(call); acc_set<41>(call); acc_set<42>(call); acc_set<43>(call);
        const unsigned tbase = (call - 1u) * NUNIT - 3u * WPP;
        char* const fl = pws + WS_FLAG + rank * 2048;
        char* const own = me + WS_X + (size_t)((call & 1u) * NR + rank) * SHARD;
        prev = u0; u0 = u1; u1 = u2;
        wait_vm<7>();
        int fill = 0;
        char* const cnt = me + WS_CNT + pi * (128 * 64);
        auto filler = [&]() { asm volatile("global_load_dword a38, %0, %1" :: "v"(vzero), "s"(p.x) : "memory"); };
        auto publish = [&](unsigned u) {
            if (fill == 4) filler(); else if (u < NAGG) snd_ticket<37>(cnt + (u >> 2) * 64, vzero); else snd_flag<40>(fl + unit_flag(u), vzero);
        };
        auto publish4 = [&](unsigned u) {
            unsigned c; asm volatile("v_accvgpr_read_b32 %0, a37\n\ts_nop 1" : "=v"(c));
            if (u < NAGG && fill != 4 && (__builtin_amdgcn_readfirstlane(c) & 3) == 3) snd_flag4<40>(fl + unit_flag(u & ~3u), vzero); else filler();
        };
        const char* const rtr = me + WS_RTR + ((unsigned)(rank + 1 + pi) & 7u) * 128;
#define SEND_STEP(BS, BL, NA) { \
            if (u0 >= NUNIT - 16 * NSPEC - 64 && fill == 0) { asm volatile("global_load_dword a36, %0, %1 sc0 sc1" :: "v"((lane & 31) * 4), "s"(rtr) : "memory"); fill = 1; } \
            if (u0 >= NUNIT - 16 * NSPEC && (fill & 1)) { \
                const unsigned long long t0 = __builtin_amdgcn_s_memrealtime(); \
                for (fill = 2;;) { \
                    unsigned f, tmo; \
                    asm volatile("global_load_dword %0, %2, %3 sc0 sc1\n\tglobal_load_dword %1, %4, %5 sc1\n\ts_waitcnt vmcnt(0)" \
                                 : "=&v"(f), "=&v"(tmo) : "v"((lane & 31) * 4), "s"(rtr), "v"(vzero), "s"(me + WS_TMO) : "memory"); \
                    if (__all(f >= call)) break; \
                    if (__builtin_amdgcn_readfirstlane(tmo) == call || __builtin_amdgcn_s_memrealtime() - t0 > DEADLINE_TICKS) { snd_flag<40>(me + WS_TMO, vzero); fill = 4; break; } \
                    __builtin_amdgcn_s_sleep(4); \
                } \
            } \
            if (TESTREDO && u0 >= NUNIT - 16 * NSPEC && (u0 & 3) == 1) { __builtin_amdgcn_s_sleep(127); __builtin_amdgcn_s_sleep(127); } \
            char* const d = (fill == 4 ? own : dst) + u0 * 4096u; \
            snd_half<BS, 0>(d, vdst); \
            u2 = snd_wait_ticket<NA>() - tbase; \
            snd_ticket<32>(ctr, vzero); \
            snd_load<BL>(srcp(u2), vsrc); \
            wait_vm<7>();                                                \
            publish4(pp); publish(prev); \
            snd_half<BS, 1>(d, vdst); \
            if ((prev & 48) == 0 && prev < 448) STAMP(3 + (prev >> 6)); \
            pp = prev; prev = u0; u0 = u1; u1 = u2; \
            wait_vm<9>();                                                \
            if (fill == 1) { unsigned f; asm volatile("v_accvgpr_read_b32 %0, a36" : "=v"(f)); fill = __all(f >= call) ? 2 : 3; } }
        SEND_STEP(80, 64, 8)
        for (;;) {
            SEND_STEP(96, 80, 10) if (u0 >= NUNIT) break;
            SEND_STEP(64, 96, 10) if (u0 >= NUNIT) break;
            SEND_STEP(80, 64, 10) if (u0 >= NUNIT) break;
        }
#undef SEND_STEP
        wait_vm<0>();
        publish4(pp); publish(prev);
        if (prev < NAGG) { wait_vm<0>(); publish4(prev); }
        snd_flag<40>(callp, vzero);
        STAMP(1);
        return;
    }

    int cg, rg;
    {
        const int xcd = bid & 7, i = (xcd & 1) * 28 + (bid >> 3) - 4;
        rg = bid < 32 ? 7 : i >> 3;
        cg = bid < 32 ? p.slow + (xcd & 1) + (xcd >> 2) * 16 : (xcd >> 1) * 8 + (i & 7);
    }
    const int q = wid;
    for (int d = 0; d < WDELAY; d += 8) __builtin_amdgcn_s_sleep(8);
    const float scale = p.sx[0] * p.sw[0];
    unsigned call;
    const unsigned ldsw = (unsigned)(size_t)(LAS unsigned char*)lds + q * 32768u;
    const unsigned peer = (unsigned)(rank + 1 + rg) & 7u;
    const unsigned sfl = peer * 2048 + q * 512, sflo = rank * 2048 + q * 512, vfl = lane * 8;
    XBlk X0, X1;
    u32x2 pf;
    {
        const __amdgpu_buffer_rsrc_t rw = __builtin_amdgcn_make_buffer_rsrc((void*)p.w, 0, (unsigned)KDIM * NFULL, 0x00020000);
        const unsigned ve = (lane >> 2) * NFULL + (((lane & 3) ^ ((lane >> 4) & 1)) * 16), vo = ve ^ 32u;
        const unsigned so = (unsigned)(q * 1024) * NFULL + rank * NSH + cg * 64;
        const unsigned ab = ldsw + ((lane >> 4) * 16 + ((lane & 15) >> 1)) * 64 + (lane & 1) * 8, f = (lane >> 3) & 3;
        const unsigned a0 = ab + ((0 ^ f) << 4), a1 = ab + ((1 ^ f) << 4), a2 = ab + ((2 ^ f) << 4), a3 = ab + ((3 ^ f) << 4);
        constexpr unsigned SS = 128u * NFULL;
        unsigned vzero = 0;
        asm volatile("" : "+v"(vzero));
        call_load<32>(callp, vzero);
#define WOFF(S) (so + (unsigned)(((S) + ROT) & 7) * SS)
#define WFIRST w_stage(rw, ve, vo, ldsw + 0 * 8192, WOFF(0)); if (TOUCHPOS == 1) touch(); w_stage(rw, ve, vo, ldsw + 1 * 8192, WOFF(1)); if (TOUCHPOS == 2) touch(); \
               w_stage(rw, ve, vo, ldsw + 2 * 8192, WOFF(2)); w_stage(rw, ve, vo, ldsw + 3 * 8192, WOFF(3));
        const bool copier = rg >= 3 && rg < 7;
        const unsigned uo = (unsigned)(((rg - 3) * 32 + cg) * 4 + q);
        if (copier) snd_load<224>(p.x + unit_src(uo), vsrc);
        auto touch = [&]() {
        if (rg < 7) {
            const unsigned l0 = (unsigned)(((bid >> 3) - 4) * 4 + q) * 64u + lane, cb = rank * NSH + (cg & 24) * 64;
            const unsigned l2 = l0 + 2 * 7168u;
            asm volatile("buffer_load_dword a249, %0, %3, %4 offen\n\tbuffer_load_dword a250, %1, %3, %4 offen\n\tbuffer_load_dword a251, %2, %3, %4 offen"
                         :: "v"((l0 >> 2) * NFULL + (l0 & 3) * 128u), "v"(((l0 + 7168u) >> 2) * NFULL + (l0 & 3) * 128u),
                            "v"(((l2 < 16384u ? l2 : l0) >> 2) * NFULL + (l0 & 3) * 128u), "s"(rw), "s"(cb) : "memory", "a249", "a250", "a251");
        }
        };
        if (TOUCHPOS == 0) touch();
        if (copier) {
            call = snd_wait_ticket<TOUCHPOS == 0 ? 3 : 0>() + 1u;
            acc_set<240>(call);
            snd_store<224>(me + WS_X + (size_t)((call & 1u) * NR + rank) * SHARD + uo * 4096u, vdst);
            WFIRST
            wait_vm<32>();
            snd_flag<240>(me + WS_FLAG + rank * 2048 + unit_flag(uo), vzero);
            wait_vm<24>();
        } else {
            WFIRST
            call = snd_wait_ticket<24>() + 1u;
        }
#undef WFIRST
        w_read<0, 0>(a0, a1, a2, a3); w_stage(rw, ve, vo, ldsw + 0 * 8192, WOFF(4));
        wait_vm<24>(); w_read<1, 1>(a0, a1, a2, a3); w_stage(rw, ve, vo, ldsw + 1 * 8192, WOFF(5));
        wait_vm<24>(); w_read<2, 2>(a0, a1, a2, a3); w_stage(rw, ve, vo, ldsw + 2 * 8192, WOFF(6));
        wait_vm<24>(); w_read<3, 3>(a0, a1, a2, a3); w_stage(rw, ve, vo, ldsw + 3 * 8192, WOFF(7));
#undef WOFF
        wait_vm<24>(); w_read<4, 0>(a0, a1, a2, a3);
        wait_vm<16>(); w_read<5, 1>(a0, a1, a2, a3);
        pf = __builtin_bit_cast(u32x2, __builtin_amdgcn_raw_buffer_load_b64(__builtin_amdgcn_make_buffer_rsrc(me + WS_FLAG, 0, NR * 2048, 0x00020000), vfl, sfl, 17));
        x_stage(p.x + rg * 65536 + q * 1024, vdst, ldsw);
        wait_vm<8 + 17>(); w_read<6, 2>(a0, a1, a2, a3);
        wait_vm<17>();     w_read<7, 3>(a0, a1, a2, a3);
        wait_vm<0>();
        asm volatile("s_mov_b64 exec, 1\n\tglobal_store_dword %0, %1, %2 sc0 sc1\n\ts_mov_b64 exec, -1" :: "v"(vzero), "v"(call), "s"(callp) : "memory");
        const unsigned xb = q * 32768 + (lane & 15) * 1024 + (((lane >> 4) ^ (lane & 3)) << 4), xs = (lane & 12) << 4;
#pragma unroll
        for (int k = 0; k < 16; ++k) X1.h[k] = *(const LAS u32x4*)((LAS unsigned char*)lds + xb + (((unsigned)((k + 2 * ROT) & 15) << 6) ^ xs));
    }
    STAMP(1);
    static_assert(__builtin_popcountll(OWNMASK) == 5 && (OWNMASK & 3) == 1 && (OWNMASK >> 32) == 0, "");
    const bool slowcg = (cg & 14) == p.slow;
    const int n = rg == 7 ? NBLK - 7 : slowcg ? NBLK + 1 : NBLK + 5, ostep = rg == 7 ? 1 : 7;
    const unsigned ownpos = rg == 7 ? ~0u : slowcg ? 1u : (unsigned)OWNMASK;
    unsigned tabx; int tabr;
    {
        const int before = __builtin_amdgcn_mbcnt_lo(ownpos, 0u);
        const bool isown = lane < 32 && ((ownpos >> (lane & 31)) & 1u);
        const int b = isown ? rg + ostep * before : lane - before;
        tabx = (((isown ? rank : peer) * NBLK + (b & 31)) << 16) | (b < NBLK ? 0u : 1u);
        tabr = isown || b >= NBLK ? -1 : b;
    }
    void* const xall = me + WS_X + (size_t)(call & 1u) * NR * SHARD;
    const unsigned orow = q * 4 + (lane >> 4), voffO = orow * (NSH * 4) + (cg * 64 + (lane & 15) * 4) * 4;
    const unsigned lwb = 131072 + q * 4096 + (lane & 15) * 256 + (((lane >> 4) ^ (lane & 15)) << 4);
    LAS unsigned char* const lw0 = (LAS unsigned char*)lds + lwb; LAS unsigned char* const lw1 = (LAS unsigned char*)lds + (lwb ^ 64);
    LAS unsigned char* const lw2 = (LAS unsigned char*)lds + (lwb ^ 128); LAS unsigned char* const lw3 = (LAS unsigned char*)lds + (lwb ^ 192);
    LAS unsigned char* const lr = (LAS unsigned char*)lds + 131072 + orow * 256 + (((lane & 15) ^ orow) << 4);
    unsigned xlim = NR * SHARD;
    f32x4 cinit = {0.f, 0.f, 0.f, 0.f};
    unsigned xsz = NR * SHARD;

    auto count = [&](u32x2 f) { unsigned r; asm("s_ff1_i32_b64 %0, %1" : "=s"(r) : "s"(__ballot(f.x < call || f.y < call))); return (int)(r >> 1); };
    auto spin = [&](unsigned soff, int need) {
        const char* const pf = me + WS_FLAG + soff;
        const unsigned long long t0 = __builtin_amdgcn_s_memrealtime();
        asm volatile("" :: "s"(t0));
        for (;;) {
            u32x2 f; unsigned tmo;
            asm volatile("global_load_dwordx2 %0, %2, %3 sc0 sc1\n\tglobal_load_dword %1, %4, %5 sc1\n\ts_waitcnt vmcnt(0)"
                         : "=&v"(f), "=&v"(tmo) : "v"(vfl), "s"(pf), "v"(0u), "s"(me + WS_TMO) : "memory");
            const int c = count(f);
            if (c > need) return c;
            if (__builtin_amdgcn_readfirstlane(tmo) == call || __builtin_amdgcn_s_memrealtime() - t0 > DEADLINE_TICKS) {
                if (lane == 0) __hip_atomic_store((unsigned*)(me + WS_TMO), call, __ATOMIC_RELAXED, __HIP_MEMORY_SCOPE_AGENT);
                xlim = 0; xsz = 0; const float qn = __builtin_nanf(""); cinit = (f32x4){qn, qn, qn, qn};
                asm volatile("s_nop 1" : "+v"(cinit));
                return 2 * NBLK;
            }
            __builtin_amdgcn_s_sleep(SPIN_SLEEP);
        }
    };
    unsigned voffR[8];
#pragma unroll
    for (int k = 0; k < 8; ++k) voffR[k] = vdst + q * 16384 + (unsigned)((k + ROT) & 7) * 2048u;
    f32x4 c0 = cinit, c1 = cinit, c2 = cinit, c3 = cinit;
    asm volatile("s_nop 1" : "+v"(cinit), "+v"(c0), "+v"(c1), "+v"(c2), "+v"(c3));
#define XLOAD(X, k) (X).h[k] = __builtin_bit_cast(u32x4, __builtin_amdgcn_raw_buffer_load_b128(rx, voffR[(k) >> 1], sx + ((k) & 1) * 1024u, 17))
#define POLL(size, off) __builtin_bit_cast(u32x2, __builtin_amdgcn_raw_buffer_load_b64(__builtin_amdgcn_make_buffer_rsrc(me + WS_FLAG, 0, size, 0x00020000), vfl, off, 17))
    int rdy = rg == 7 ? 2 * NBLK : count(pf);
    unsigned xm = 1u, x0 = __builtin_amdgcn_readlane(tabx, 0), x1 = __builtin_amdgcn_readlane(tabx, 1);
    int r1 = __builtin_amdgcn_readlane(tabr, 1), need = r1 + 1, thr = need;
    int sp0 = NBLK;
    unsigned sx, oso = 0, osz = 0, psz = NR * 2048;
    auto store_rows = [&](const f32x4& r) {
        __builtin_amdgcn_raw_buffer_store_b128(__builtin_bit_cast(u32x4, r), __builtin_amdgcn_make_buffer_rsrc((void*)p.out, 0, NOSTORE ? 0u : osz, 0x00020000), voffO, oso, OUT_AUX);
    };
    static_assert(NSPEC % 2 == 0 && NSPEC <= 14, "");
    char* const pz = (char*)xall + (rg < 7 ? (size_t)peer * SHARD + (NBLK - NSPEC) * 65536 + (cg * 4 + q) * 1024 : (size_t)rank * SHARD + (bid & 7) * 4096 + q * 1024);
    const unsigned pzs = rg < 7 ? 131072u : 0u;
    auto fill_piece = [&](int k) {
        const u32x4 ff = {~0u, ~0u, ~0u, ~0u};
        asm volatile("global_store_dwordx4 %0, %1, %2 sc0 sc1\n\ts_nop 1" :: "v"(vdst), "v"(ff), "s"(pz + k * pzs) : "memory");
    };
#define SB __builtin_amdgcn_sched_barrier(0);
#define XFRAG(X, S) __builtin_shufflevector((X).h[2 * (S)], (X).h[2 * (S) + 1], 0, 1, 2, 3, 4, 5, 6, 7)
#define PIN(v) asm volatile("" : "+s"(v));
#define SLOT(FIRST, CUR, NXT, PP, PO, PN, J, C) \
        mf<(J) / 4, C>(c##C, XFRAG(CUR, (J) / 4), cinit); SB \
        if constexpr ((J) == 0) *(LAS f32x4*)(lw2 + PP * 16384) = c2; \
        if constexpr ((J) == 1) *(LAS f32x4*)(lw3 + PP * 16384) = c3; \
        if constexpr ((J) == 2) *(LAS f32x4*)(lw0 + PP * 16384) = c0; \
        if constexpr ((J) == 31) *(LAS f32x4*)(lw1 + (1 - PP) * 16384) = c1; \
        if constexpr ((J) == S_POLL) PN = POLL(psz, sfl); \
        if constexpr (FIRST && (J) == S_POLL - 2) pf = POLL(NR * 2048, sflo); \
        if constexpr (LATEFILL && FIRST && ((J) & 3) == 2 && (J) / 4 < NSPEC / 2) fill_piece((J) / 4); \
        if constexpr ((J) & 1) XLOAD(NXT, (J) >> 1); \
        if constexpr ((J) == S_BAR) { wait_lgkm0(); __builtin_amdgcn_s_barrier(); asm volatile("" ::: "memory"); } \
        if constexpr ((J) == S_RD) { q0 = *(const LAS f32x4*)(lr + PP * 16384); q1 = *(const LAS f32x4*)(lr + PP * 16384 + 4096); \
                                     q2 = *(const LAS f32x4*)(lr + PP * 16384 + 8192); q3 = *(const LAS f32x4*)(lr + PP * 16384 + 12288); } \
        if constexpr ((J) == S_A1) q1 = fadd4(q0, q1); \
        if constexpr ((J) == S_A2) q2 = fadd4(q2, q3); \
        if constexpr ((J) == S_A3) q1 = fadd4(q1, q2); \
        if constexpr ((J) == S_MU) q1 = fmul4(q1, scale); \
        if constexpr ((J) == S_ST) store_rows(q1); \
        if constexpr ((J) == S_K1) { PIN(r1) sp1 = r1 >= rdy ? r1 : NBLK; PIN(sp1) PIN(i) i2 = i + 2; PIN(i2) } \
        if constexpr ((J) == S_K2) { PIN(xm) oso = xm << 1; osz = (xm & 1u) ? 0u : (unsigned)(NR * MSH) * NSH * 4u; PIN(oso) PIN(osz) } \
        if constexpr ((J) == S_K3) { PIN(i2) xm = x0; x0 = x1; x1 = __builtin_amdgcn_readlane(tabx, i2); r1 = __builtin_amdgcn_readlane(tabr, i2); PIN(x1) PIN(r1) } \
        if constexpr ((J) == S_K3 + 1) { PIN(x1) xszn = (x1 & 1u) ? 0u : xlim; PIN(xszn) } \
        if constexpr ((J) == S_K4) { PIN(r1) int r2 = r1 + 1; PIN(r2) need = r1 < NBLK - NSPEC ? r2 : r1; PIN(need) } \
        if constexpr ((J) == S_K4 + 1) { PIN(need) thr = sp0 < NBLK ? -1 : need; BSTAMP(i); ++i; PIN(thr) PIN(i) } \
        if constexpr ((J) == S_K5) { asm volatile("" : "+v"(PO)); const int c = count(PO); rdy = c > rdy ? c : rdy; PIN(rdy) }     \
        if constexpr ((J) == S_K6) { PIN(need) psz = rdy < need + POLL_AHEAD ? NR * 2048 : 0u; PIN(psz) } \
        SB
#define STEP4(FIRST, CUR, NXT, PP, PO, PN, S) SLOT(FIRST, CUR, NXT, PP, PO, PN, 4 * S, 1) SLOT(FIRST, CUR, NXT, PP, PO, PN, 4 * S + 1, 2) SLOT(FIRST, CUR, NXT, PP, PO, PN, 4 * S + 2, 3) SLOT(FIRST, CUR, NXT, PP, PO, PN, 4 * S + 3, 0)
#define RECHECK(PP) \
        if (spm < NBLK) { \
            asm volatile("s_nop 7\n\ts_nop 7" : "+v"(c0)); \
            if (__any(c0[0] != c0[0])) { \
                rdy = spin(sfl, spm); \
                const __amdgpu_buffer_rsrc_t rx = __builtin_amdgcn_make_buffer_rsrc(xall, 0, xlim, 0x00020000); \
                sx = xm; \
                XBlk T; \
                _Pragma("unroll") for (int k = 0; k < 16; ++k) XLOAD(T, k); \
                M4(T, 0) M4(T, 1) M4(T, 2) M4(T, 3) M4(T, 4) M4(T, 5) M4(T, 6) M4(T, 7) \
                asm volatile("s_nop 7\n\ts_nop 7" : "+v"(c0), "+v"(c1), "+v"(c2), "+v"(c3)); \
                *(LAS f32x4*)(lw1 + PP * 16384) = c1; \
                STAMP(48); \
            } \
        }
#define M4(X, S) mf<S, 1>(c1, XFRAG(X, S), cinit); mf<S, 2>(c2, XFRAG(X, S), cinit); mf<S, 3>(c3, XFRAG(X, S), cinit); mf<S, 0>(c0, XFRAG(X, S), cinit);
#define BODY(FIRST, CUR, NXT, PP, PO, PN) { \
        if (__builtin_expect((unsigned)rdy < (unsigned)thr, 0)) { \
            RECHECK(PP) \
            if (rdy < need) { const int c = count(PO); rdy = c > rdy ? c : rdy; }     \
            if (rdy < need) rdy = spin(sfl, need - 1); \
        } \
        GOSTAMP \
        sx = x1; \
        const __amdgpu_buffer_rsrc_t rx = __builtin_amdgcn_make_buffer_rsrc(xall, 0, xsz, 0x00020000); \
        f32x4 q0, q1, q2, q3; unsigned xszn; int sp1, i2; \
        STEP4(FIRST, CUR, NXT, PP, PO, PN, 0) STEP4(FIRST, CUR, NXT, PP, PO, PN, 1) STEP4(FIRST, CUR, NXT, PP, PO, PN, 2) STEP4(FIRST, CUR, NXT, PP, PO, PN, 3) \
        STEP4(FIRST, CUR, NXT, PP, PO, PN, 4) STEP4(FIRST, CUR, NXT, PP, PO, PN, 5) STEP4(FIRST, CUR, NXT, PP, PO, PN, 6) STEP4(FIRST, CUR, NXT, PP, PO, PN, 7) \
        xsz = xszn; spm = sp0; sp0 = sp1; }

    int i = 0, spm = NBLK;
    u32x2 P0 = {0u, 0u}, P1;
    if (rg == 7) spin(sflo, NBLK - 1);
    BODY(true, X1, X0, 1, P0, P1)
    if (ownpos != 1u && count(pf) < NBLK) spin(sflo, NBLK - 1);
    do {
        BODY(false, X0, X1, 0, P1, P0) BODY(false, X1, X0, 1, P0, P1)
        if (LATEFILL && __builtin_expect(i == 3 && q == 0 && rg < 7, 0))
            asm volatile("s_mov_b64 exec, 1\n\tglobal_store_dword %0, %1, %2 sc0 sc1\n\ts_mov_b64 exec, -1" :: "v"(0u), "v"(call), "s"(p.wsr[rg] + WS_RTR + rank * 128 + cg * 4) : "memory");
    } while (i < n);
    RECHECK(0)
    oso = xm << 1; osz = (unsigned)(NR * MSH) * NSH * 4u;
    {
        asm volatile("s_nop 7\n\ts_nop 7" : "+v"(c0), "+v"(c2), "+v"(c3));
        *(LAS f32x4*)(lw2 + 0 * 16384) = c2; *(LAS f32x4*)(lw3 + 0 * 16384) = c3; *(LAS f32x4*)(lw0 + 0 * 16384) = c0;
        wait_lgkm0(); __builtin_amdgcn_s_barrier(); asm volatile("" ::: "memory");
        store_rows(fmul4(fadd4(fadd4(*(const LAS f32x4*)(lr + 0 * 16384), *(const LAS f32x4*)(lr + 0 * 16384 + 4096)), fadd4(*(const LAS f32x4*)(lr + 0 * 16384 + 8192), *(const LAS f32x4*)(lr + 0 * 16384 + 12288))), scale));
        STAMP(39);
        asm volatile(".Lfk_code_end:");
    }
#undef BODY
#undef M4
#undef RECHECK
#undef STEP4
#undef SLOT
#undef XFRAG
#undef SB
#undef XLOAD
#undef POLL
#undef PIN
}
}

extern "C" void kernel_launch(int rank, int nranks, ncclComm_t comm,
                              hipStream_t stream,
                              void* const* d_in, const int* in_sizes, int n_in,
                              void* d_out, int out_size,
                              const int* in_dtypes, int out_dtype,
                              void* const* d_out_all,
                              void* const* d_ws_all, size_t ws_bytes) {
    (void)comm; (void)in_dtypes; (void)out_dtype; (void)d_out_all;
    if (nranks != NR || (unsigned)rank >= (unsigned)NR || n_in != 4 || ws_bytes < WS_END ||
        in_sizes[0] != MSH * KDIM || in_sizes[1] != KDIM * NFULL || out_size != NR * MSH * NSH) {
        fprintf(stderr, "kernel_launch: unexpected problem shape\n");
        return;
    }
    Params p;
    p.x = (const unsigned char*)d_in[0]; p.w = (const unsigned char*)d_in[1];
    p.sx = (const float*)d_in[2]; p.sw = (const float*)d_in[3]; p.out = (float*)d_out;
    for (int i = 0; i < NR; ++i) p.wsr[i] = (char*)d_ws_all[(rank + 1 + i) & 7];
    p.rank = rank; p.slow = (int)((3u - (unsigned)((size_t)p.w >> 7)) & 7u) * 2;
    fused_kernel<<<256, 256, 0, stream>>>(p);
    HIP_CHECK(hipGetLastError());
}
```

```cpp
#include <hip/hip_runtime.h>
#include <rccl/rccl.h>
#include <cstdio>

#define HIP_CHECK(expr) do { hipError_t _e = (expr); if (_e != hipSuccess) { \
    fprintf(stderr, "HIP error %s at %s:%d\n", hipGetErrorString(_e), __FILE__, __LINE__); \
    return; } } while (0)

namespace {
typedef unsigned u32x2 __attribute__((ext_vector_type(2)));
typedef unsigned u32x4 __attribute__((ext_vector_type(4)));
typedef unsigned u32x8 __attribute__((ext_vector_type(8)));
typedef float f32x4 __attribute__((ext_vector_type(4)));
#define LAS __attribute__((address_space(3)))
#define DI __device__ __forceinline__

constexpr int NR = 8;
constexpr int KDIM = 4096, NFULL = 16384, NSH = 2048, MSH = 512;
constexpr unsigned SHARD = MSH * KDIM;
constexpr int NBLK = MSH / 16;
constexpr int NUNIT = 512;
constexpr int WPP = 16;
constexpr size_t WS_X = 0;
constexpr size_t WS_FLAG = 2ull * NR * SHARD;
constexpr size_t WS_RTR = WS_FLAG + NR * 2048;
constexpr size_t WS_TICK = WS_RTR + NR * 128;
constexpr size_t WS_TMO = WS_TICK + 1024;
constexpr size_t WS_CALL = WS_TMO + 1024;
constexpr size_t WS_CNT = WS_CALL + 1024 * 4;
constexpr size_t WS_END = WS_CNT + 7 * 128 * 64;
constexpr unsigned long long DEADLINE_TICKS = 180000000ull;

constexpr unsigned OWNMASK = 0x155u;
constexpr int POLL_AHEAD = 3;
constexpr int NSPEC = 8;
constexpr unsigned NAGG = 26 * 16;
constexpr int OUT_AUX = 2;

struct Params {
    const unsigned char* x; const unsigned char* w; const float* sx; const float* sw; float* out;
    char* wsr[NR];
    int rank; int slow;
};

#define ACLOB "a0","a1","a2","a3","a4","a5","a6","a7","a8","a9","a10","a11","a12","a13","a14","a15","a16","a17","a18","a19","a20","a21","a22","a23","a24","a25","a26","a27","a28","a29","a30","a31","a32","a33","a34","a35","a36","a37","a38","a39","a40","a41","a42","a43","a44","a45","a46","a47","a48","a49","a50","a51","a52","a53","a54","a55","a56","a57","a58","a59","a60","a61","a62","a63","a64","a65","a66","a67","a68","a69","a70","a71","a72","a73","a74","a75","a76","a77","a78","a79","a80","a81","a82","a83","a84","a85","a86","a87","a88","a89","a90","a91","a92","a93","a94","a95","a96","a97","a98","a99","a100","a101","a102","a103","a104","a105","a106","a107","a108","a109","a110","a111","a112","a113","a114","a115","a116","a117","a118","a119","a120","a121","a122","a123","a124","a125","a126","a127","a128","a129","a130","a131","a132","a133","a134","a135","a136","a137","a138","a139","a140","a141","a142","a143","a144","a145","a146","a147","a148","a149","a150","a151","a152","a153","a154","a155","a156","a157","a158","a159","a160","a161","a162","a163","a164","a165","a166","a167","a168","a169","a170","a171","a172","a173","a174","a175","a176","a177","a178","a179","a180","a181","a182","a183","a184","a185","a186","a187","a188","a189","a190","a191","a192","a193","a194","a195","a196","a197","a198","a199","a200","a201","a202","a203","a204","a205","a206","a207","a208","a209","a210","a211","a212","a213","a214","a215","a216","a217","a218","a219","a220","a221","a222","a223","a224","a225","a226","a227","a228","a229","a230","a231","a232","a233","a234","a235","a236","a237","a238","a239","a240","a241","a242","a243","a244","a245","a246","a247","a248","a249","a250","a251","a252","a253","a254","a255"

template <int N> DI void wait_vm() { asm volatile("s_waitcnt vmcnt(%0)" :: "n"(N) : "memory"); }
DI void wait_lgkm0() { asm volatile("s_waitcnt lgkmcnt(0)" ::: "memory"); }
template <int B> DI void snd_load(const unsigned char* base, unsigned voff) {
    asm volatile("global_load_dwordx4 a[%c2:%c2+3], %0, %1\n\t"
                 "global_load_dwordx4 a[%c2+4:%c2+7], %0, %1 offset:64\n\t"
                 "global_load_dwordx4 a[%c2+8:%c2+11], %0, %1 offset:128\n\t"
                 "global_load_dwordx4 a[%c2+12:%c2+15], %0, %1 offset:192" :: "v"(voff), "s"(base), "i"(B) : "memory");
}
template <int B> DI void snd_store(char* base, unsigned voff) {
    asm volatile("global_store_dwordx4 %0, a[%c2:%c2+3], %1 sc0 sc1\n\t"
                 "global_store_dwordx4 %0, a[%c2+4:%c2+7], %1 offset:1024 sc0 sc1\n\t"
                 "global_store_dwordx4 %0, a[%c2+8:%c2+11], %1 offset:2048 sc0 sc1\n\t"
                 "global_store_dwordx4 %0, a[%c2+12:%c2+15], %1 offset:3072 sc0 sc1\n\ts_nop 1" :: "v"(voff), "s"(base), "i"(B) : "memory");
}
template <int A> DI void snd_ticket(char* ctr, unsigned vzero) {
    unsigned long long keep;
    asm volatile("s_mov_b64 %0, exec\n\ts_mov_b64 exec, 1\n\tglobal_atomic_add a[%c3], %1, a33, %2 sc0\n\ts_mov_b64 exec, %0"
                 : "=&s"(keep) : "v"(vzero), "s"(ctr), "i"(A) : "memory");
}
DI void code_prefetch(unsigned lane) {
    unsigned long long pc; unsigned t0, t1, t2;
    asm volatile("s_getpc_b64 %0\n"
                 ".Lfk_code_begin:\n\t"
                 "v_min_u32 %1, .Lfk_code_end-.Lfk_code_begin, %4\n\tv_min_u32 %2, .Lfk_code_end-.Lfk_code_begin, %5\n\tv_min_u32 %3, .Lfk_code_end-.Lfk_code_begin, %6\n\t"
                 "global_load_dword a252, %1, %0\n\tglobal_load_dword a253, %2, %0\n\tglobal_load_dword a254, %3, %0"
                 : "=&s"(pc), "=&v"(t0), "=&v"(t1), "=&v"(t2) : "v"(lane * 128u), "v"(lane * 128u + 8192u), "v"(lane * 128u + 16384u) : "memory", "a252", "a253", "a254");
}
template <int B, int H> DI void snd_half(char* base, unsigned voff) {
    asm volatile("global_store_dwordx4 %0, a[%c2:%c2+3], %1 offset:%c3 sc0 sc1\n\t"
                 "global_store_dwordx4 %0, a[%c2+4:%c2+7], %1 offset:%c3+1024 sc0 sc1\n\ts_nop 1" :: "v"(voff), "s"(base), "i"(B + 8 * H), "i"(2048 * H) : "memory");
}
template <int A> DI void call_load(const char* ctr, unsigned vzero) { asm volatile("global_load_dword a[%c2], %0, %1 sc0 sc1" :: "v"(vzero), "s"(ctr), "i"(A) : "memory"); }
template <int A> DI void snd_flag(char* flag, unsigned vzero) {
    unsigned long long keep;
    asm volatile("s_mov_b64 %0, exec\n\ts_mov_b64 exec, 1\n\tglobal_store_dword %1, a[%c3], %2 sc0 sc1\n\ts_mov_b64 exec, %0"
                 : "=&s"(keep) : "v"(vzero), "s"(flag), "i"(A) : "memory");
}
template <int A> DI void snd_flag4(char* flag, unsigned vzero) {
    unsigned long long keep;
    asm volatile("s_mov_b64 %0, exec\n\ts_mov_b64 exec, 1\n\tglobal_store_dwordx4 %1, a[%c3:%c3+3], %2 sc0 sc1\n\ts_mov_b64 exec, %0"
                 : "=&s"(keep) : "v"(vzero), "s"(flag), "i"(A) : "memory");
}
template <int A> DI void acc_set(unsigned v) { asm volatile("s_nop 1\n\tv_accvgpr_write_b32 a[%c1], %0\n\ts_nop 1" :: "s"(v), "i"(A)); }
DI unsigned unit_src(unsigned u) { return (u >> 4) * 65536u + (u & 15u) * 256u; }
DI unsigned unit_flag(unsigned u) { return (((u >> 2) & 3u) * 128u + (u >> 4) * 4u + (u & 3u)) * 4u; }
template <int N, int A = 32> DI unsigned snd_wait_ticket() {
    unsigned v;
    asm volatile("s_waitcnt vmcnt(%1)\n\tv_accvgpr_read_b32 %0, a[%c2]\n\ts_nop 1" : "=v"(v) : "n"(N), "i"(A) : "memory");
    return __builtin_amdgcn_readfirstlane(v);
}


template <int S, int SLOT> DI void w_read(unsigned a0, unsigned a1, unsigned a2, unsigned a3) {
    constexpr int B = S * 32, O = SLOT * 8192;
#define TRR(c, t, A) "ds_read_b64_tr_b8 a[%c4+" #c "*8+" #t "*2:%c4+" #c "*8+" #t "*2+1], " A " offset:%c5+4096*(" #t "/2)+512*(" #t "%%2)\n\t"
    asm volatile(TRR(0, 0, "%0") TRR(0, 1, "%0") TRR(0, 2, "%0") TRR(0, 3, "%0")
                 TRR(1, 0, "%1") TRR(1, 1, "%1") TRR(1, 2, "%1") TRR(1, 3, "%1")
                 TRR(2, 0, "%2") TRR(2, 1, "%2") TRR(2, 2, "%2") TRR(2, 3, "%2")
                 TRR(3, 0, "%3") TRR(3, 1, "%3") TRR(3, 2, "%3") TRR(3, 3, "%3")
                 "s_waitcnt lgkmcnt(0)"
                 :: "v"(a0), "v"(a1), "v"(a2), "v"(a3), "i"(B), "i"(O) : "memory", ACLOB);
#undef TRR
}
DI void w_stage(__amdgpu_buffer_rsrc_t rs, unsigned ve, unsigned vo, unsigned m0base, unsigned soff) {
    unsigned keep, so;
#define WST(V) "s_nop 0\n\tbuffer_load_dwordx4 " V ", %5, %1 offen lds\n\t"
#define WNX "s_add_u32 m0, m0, 0x400\n\ts_add_u32 %1, %1, 0x40000\n\t"
    asm volatile("s_mov_b32 %0, m0\n\ts_mov_b32 m0, %2\n\ts_mov_b32 %1, %6\n\t"
                 WST("%3") WNX WST("%4") WNX WST("%3") WNX WST("%4") WNX WST("%3") WNX WST("%4") WNX WST("%3") WNX WST("%4")
                 "s_mov_b32 m0, %0"
                 : "=&s"(keep), "=&s"(so) : "s"(m0base), "v"(ve), "v"(vo), "s"(rs), "s"(soff) : "scc", "memory");
#undef WST
#undef WNX
    static_assert(16 * NFULL == 0x40000, "");
}
DI void x_stage(const unsigned char* base, unsigned vdst, unsigned m0base) {
    unsigned keep, v;
#define XST(r) "v_xor_b32 %1, " #r "*16, %3\n\tv_add_u32 %1, " #r "*4096, %1\n\tglobal_load_lds_dwordx4 %1, %4\n\ts_add_u32 m0, m0, 0x400\n\t"
    asm volatile("s_mov_b32 %0, m0\n\ts_mov_b32 m0, %2\n\t"
                 XST(0) XST(1) XST(2) XST(3) XST(4) XST(5) XST(6) XST(7) XST(8) XST(9) XST(10) XST(11) XST(12) XST(13) XST(14) XST(15)
                 "s_mov_b32 m0, %0"
                 : "=&s"(keep), "=&v"(v) : "s"(m0base), "v"(vdst), "s"(base) : "scc", "memory");
#undef XST
}
template <int S, int C> DI void mf(f32x4& c, const u32x8& x, const f32x4& cinit) {
    constexpr int B = S * 32 + C * 8;
    if constexpr (S == 0) asm volatile("v_mfma_f32_16x16x128_f8f6f4 %0, a[%c3:%c3+7], %1, %2" : "+v"(c) : "v"(x), "v"(cinit), "i"(B));
    else asm volatile("v_mfma_f32_16x16x128_f8f6f4 %0, a[%c2:%c2+7], %1, %0" : "+v"(c) : "v"(x), "i"(B));
}
DI float fadd(float a, float b) { float r; asm("v_add_f32_e32 %0, %1, %2" : "=v"(r) : "v"(a), "v"(b)); return r; }
DI float fmul(float a, float b) { float r; asm("v_mul_f32_e32 %0, %1, %2" : "=v"(r) : "v"(a), "v"(b)); return r; }
DI f32x4 fadd4(f32x4 a, f32x4 b) { return (f32x4){fadd(a[0], b[0]), fadd(a[1], b[1]), fadd(a[2], b[2]), fadd(a[3], b[3])}; }
DI f32x4 fmul4(f32x4 a, float b) { return (f32x4){fmul(a[0], b), fmul(a[1], b), fmul(a[2], b), fmul(a[3], b)}; }

constexpr int S_K1 = 4, S_BAR = 6, S_RD = 8, S_K2 = 10, S_A1 = 12, S_A2 = 14, S_POLL = 16, S_A3 = 18, S_MU = 20, S_ST = 22, S_K3 = 24, S_K4 = 26, S_K5 = 28, S_K6 = 30;

struct XBlk { u32x4 h[16]; };

__global__ __launch_bounds__(256, 1) void fused_kernel(Params p) {
    __shared__ __attribute__((aligned(16))) unsigned char lds[4 * 32768 + 32768];
    const int lane = threadIdx.x & 63;
    const int wid = __builtin_amdgcn_readfirstlane(threadIdx.x >> 6);
    const int rank = p.rank;
    char* const me = p.wsr[7];
    const int bid = blockIdx.x;
    code_prefetch(lane);
    const bool sender = bid < 24 || (bid < 32 && (bid & 2));
    const int w = (bid < 24 ? bid : 24 + (bid & 1) + ((bid >> 1) & 2)) * 4 + wid, pi = w % 7;
    char* const pws = p.wsr[pi];
    asm volatile("" :: "s"(p.x), "s"(p.w), "s"(me), "s"(rank), "s"(pws));

    char* const callp = me + WS_CALL + (bid * 4 + wid) * 4;

    const unsigned vsrc = (lane & 15) * KDIM + (lane >> 4) * 16, vdst = lane * 16;

    if (__builtin_expect(sender, 1)) {
        const int j = w / 7;
        unsigned vzero = 0;
        asm volatile("v_accvgpr_write_b32 a33, 1" : "+v"(vzero) :: "a32", "a33", "a35", "a36", "a37", "a38", "a43", "a111");
        auto srcp = [&](unsigned u) { return p.x + unit_src(u < NUNIT ? u : NUNIT - 1); };
        unsigned u0 = j, u1 = j + WPP, u2 = j + 2 * WPP, prev, pp = NUNIT;
        char* const ctr = me + WS_TICK + pi * 128;
        call_load<35>(callp, vzero);
        snd_load<64>(srcp(u0), vsrc);
        snd_load<80>(srcp(u1), vsrc);
        const unsigned call = snd_wait_ticket<4, 35>() + 1u;
        char* const dst = pws + WS_X + (size_t)((call & 1u) * NR + rank) * SHARD;
        snd_half<64, 0>(dst + u0 * 4096u, vdst);
        snd_ticket<32>(ctr, vzero);
        snd_load<96>(srcp(u2), vsrc);
        snd_half<64, 1>(dst + u0 * 4096u, vdst);
        acc_set<40>(call); acc_set<41>(call); acc_set<42>(call); acc_set<43>(call);
        const unsigned tbase = (call - 1u) * NUNIT - 3u * WPP;
        char* const fl = pws + WS_FLAG + rank * 2048;
        char* const own = me + WS_X + (size_t)((call & 1u) * NR + rank) * SHARD;
        prev = u0; u0 = u1; u1 = u2;
        wait_vm<7>();
        int fill = 0;
        char* const cnt = me + WS_CNT + pi * (128 * 64);
        auto filler = [&]() { asm volatile("global_load_dword a38, %0, %1" :: "v"(vzero), "s"(p.x) : "memory"); };
        auto publish = [&](unsigned u) {
            if (fill == 4 || u >= NUNIT) filler(); else if (u < NAGG) snd_ticket<37>(cnt + (u >> 2) * 64, vzero); else snd_flag<40>(fl + unit_flag(u), vzero);
        };
        auto publish4 = [&](unsigned u) {
            unsigned c; asm volatile("v_accvgpr_read_b32 %0, a37\n\ts_nop 1" : "=v"(c));
            if (u < NAGG && fill != 4 && (__builtin_amdgcn_readfirstlane(c) & 3) == 3) snd_flag4<40>(fl + unit_flag(u & ~3u), vzero); else filler();
        };
        const char* const rtr = me + WS_RTR + ((unsigned)(rank + 1 + pi) & 7u) * 128;
#define SEND_STEP(BS, BL, NA) { \
            if (u0 >= NUNIT - 16 * NSPEC - 64 && fill == 0) { asm volatile("global_load_dword a36, %0, %1 sc0 sc1" :: "v"((lane & 31) * 4), "s"(rtr) : "memory"); fill = 1; } \
            if (u0 >= NUNIT - 16 * NSPEC && (fill & 1)) { \
                  \
                wait_vm<0>(); publish4(pp); publish(prev); wait_vm<0>(); publish4(prev); pp = prev = NUNIT; \
                const unsigned long long t0 = __builtin_amdgcn_s_memrealtime(); \
                for (fill = 2;;) { \
                    unsigned f, tmo; \
                    asm volatile("global_load_dword %0, %2, %3 sc0 sc1\n\tglobal_load_dword %1, %4, %5 sc1\n\ts_waitcnt vmcnt(0)" \
                                 : "=&v"(f), "=&v"(tmo) : "v"((lane & 31) * 4), "s"(rtr), "v"(vzero), "s"(me + WS_TMO) : "memory"); \
                    if (__all(f >= call)) break; \
                    if (__builtin_amdgcn_readfirstlane(tmo) == call || __builtin_amdgcn_s_memrealtime() - t0 > DEADLINE_TICKS) { snd_flag<40>(me + WS_TMO, vzero); fill = 4; break; } \
                    __builtin_amdgcn_s_sleep(4); \
                } \
            } \
            char* const d = (fill == 4 ? own : dst) + u0 * 4096u; \
            snd_half<BS, 0>(d, vdst); \
            u2 = snd_wait_ticket<NA>() - tbase; \
            snd_ticket<32>(ctr, vzero); \
            snd_load<BL>(srcp(u2), vsrc); \
            wait_vm<7>();                                                \
            publish4(pp); publish(prev); \
            snd_half<BS, 1>(d, vdst); \
            pp = prev; prev = u0; u0 = u1; u1 = u2; \
            wait_vm<9>();                                                \
            if (fill == 1) { unsigned f; asm volatile("v_accvgpr_read_b32 %0, a36" : "=v"(f)); fill = __all(f >= call) ? 2 : 3; } }
        SEND_STEP(80, 64, 8)
        for (;;) {
            SEND_STEP(96, 80, 10) if (u0 >= NUNIT) break;
            SEND_STEP(64, 96, 10) if (u0 >= NUNIT) break;
            SEND_STEP(80, 64, 10) if (u0 >= NUNIT) break;
        }
#undef SEND_STEP
        wait_vm<0>();
        publish4(pp); publish(prev);
        if (prev < NAGG) { wait_vm<0>(); publish4(prev); }
        snd_flag<40>(callp, vzero);
        return;
    }

    int cg, rg;
    {
        const int xcd = bid & 7, i = (xcd & 1) * 28 + (bid >> 3) - 4;
        rg = bid < 32 ? 7 : i >> 3;
        cg = bid < 32 ? p.slow + (xcd & 1) + (xcd >> 2) * 16 : (xcd >> 1) * 8 + (i & 7);
    }
    const int q = wid;
    const float scale = p.sx[0] * p.sw[0];
    unsigned call;
    const unsigned ldsw = (unsigned)(size_t)(LAS unsigned char*)lds + q * 32768u;
    const unsigned peer = (unsigned)(rank + 1 + rg) & 7u;
    const unsigned sfl = peer * 2048 + q * 512, sflo = rank * 2048 + q * 512, vfl = lane * 8;
    XBlk X0, X1;
    u32x2 pf;
    {
        const __amdgpu_buffer_rsrc_t rw = __builtin_amdgcn_make_buffer_rsrc((void*)p.w, 0, (unsigned)KDIM * NFULL, 0x00020000);
        const unsigned ve = (lane >> 2) * NFULL + (((lane & 3) ^ ((lane >> 4) & 1)) * 16), vo = ve ^ 32u;
        const unsigned so = (unsigned)(q * 1024) * NFULL + rank * NSH + cg * 64;
        const unsigned ab = ldsw + ((lane >> 4) * 16 + ((lane & 15) >> 1)) * 64 + (lane & 1) * 8, f = (lane >> 3) & 3;
        const unsigned a0 = ab + ((0 ^ f) << 4), a1 = ab + ((1 ^ f) << 4), a2 = ab + ((2 ^ f) << 4), a3 = ab + ((3 ^ f) << 4);
        constexpr unsigned SS = 128u * NFULL;
        unsigned vzero = 0;
        asm volatile("" : "+v"(vzero));
        call_load<32>(callp, vzero);
#define WOFF(S) (so + (unsigned)(((S) + rg) & 7) * SS)
#define WFIRST w_stage(rw, ve, vo, ldsw + 0 * 8192, WOFF(0)); w_stage(rw, ve, vo, ldsw + 1 * 8192, WOFF(1)); \
               w_stage(rw, ve, vo, ldsw + 2 * 8192, WOFF(2)); w_stage(rw, ve, vo, ldsw + 3 * 8192, WOFF(3));
        const bool copier = rg >= 3 && rg < 7;
        const unsigned uo = (unsigned)(((rg - 3) * 32 + cg) * 4 + q);
        if (copier) snd_load<224>(p.x + unit_src(uo), vsrc);
        if (rg < 7) {
            const unsigned l0 = (unsigned)(((bid >> 3) - 4) * 4 + q) * 64u + lane, cb = rank * NSH + (cg & 24) * 64;
            const unsigned l2 = l0 + 2 * 7168u;
            asm volatile("buffer_load_dword a249, %0, %3, %4 offen\n\tbuffer_load_dword a250, %1, %3, %4 offen\n\tbuffer_load_dword a251, %2, %3, %4 offen"
                         :: "v"((l0 >> 2) * NFULL + (l0 & 3) * 128u), "v"(((l0 + 7168u) >> 2) * NFULL + (l0 & 3) * 128u),
                            "v"(((l2 < 16384u ? l2 : l0) >> 2) * NFULL + (l0 & 3) * 128u), "s"(rw), "s"(cb) : "memory", "a249", "a250", "a251");
        }
        if (copier) {
            call = snd_wait_ticket<3>() + 1u;
            acc_set<240>(call);
            snd_store<224>(me + WS_X + (size_t)((call & 1u) * NR + rank) * SHARD + uo * 4096u, vdst);
            WFIRST
            wait_vm<32>();
            snd_flag<240>(me + WS_FLAG + rank * 2048 + unit_flag(uo), vzero);
            wait_vm<24>();
        } else {
            WFIRST
            call = snd_wait_ticket<24>() + 1u;
        }
#undef WFIRST
        w_read<0, 0>(a0, a1, a2, a3); w_stage(rw, ve, vo, ldsw + 0 * 8192, WOFF(4));
        wait_vm<24>(); w_read<1, 1>(a0, a1, a2, a3); w_stage(rw, ve, vo, ldsw + 1 * 8192, WOFF(5));
        wait_vm<24>(); w_read<2, 2>(a0, a1, a2, a3); w_stage(rw, ve, vo, ldsw + 2 * 8192, WOFF(6));
        wait_vm<24>(); w_read<3, 3>(a0, a1, a2, a3); w_stage(rw, ve, vo, ldsw + 3 * 8192, WOFF(7));
#undef WOFF
        wait_vm<24>(); w_read<4, 0>(a0, a1, a2, a3);
        wait_vm<16>(); w_read<5, 1>(a0, a1, a2, a3);
        pf = __builtin_bit_cast(u32x2, __builtin_amdgcn_raw_buffer_load_b64(__builtin_amdgcn_make_buffer_rsrc(me + WS_FLAG, 0, NR * 2048, 0x00020000), vfl, sfl, 17));
        x_stage(p.x + rg * 65536 + q * 1024, vdst, ldsw);
        wait_vm<8 + 17>(); w_read<6, 2>(a0, a1, a2, a3);
        wait_vm<17>();     w_read<7, 3>(a0, a1, a2, a3);
        wait_vm<0>();
        asm volatile("s_mov_b64 exec, 1\n\tglobal_store_dword %0, %1, %2 sc0 sc1\n\ts_mov_b64 exec, -1" :: "v"(vzero), "v"(call), "s"(callp) : "memory");
        const unsigned xb = q * 32768 + (lane & 15) * 1024 + (((lane >> 4) ^ (lane & 3)) << 4), xs = (lane & 12) << 4;
#pragma unroll
        for (int k = 0; k < 16; ++k) X1.h[k] = *(const LAS u32x4*)((LAS unsigned char*)lds + xb + (((unsigned)((k + 2 * rg) & 15) << 6) ^ xs));
    }
    static_assert(__builtin_popcount(OWNMASK) == 5 && (OWNMASK & 3) == 1, "");
    const bool slowcg = (cg & 14) == p.slow;
    const int n = rg == 7 ? NBLK - 7 : slowcg ? NBLK + 1 : NBLK + 5, ostep = rg == 7 ? 1 : 7;
    const unsigned ownpos = rg == 7 ? ~0u : slowcg ? 1u : OWNMASK;
    unsigned tabx; int tabr;
    {
        const int before = __builtin_amdgcn_mbcnt_lo(ownpos, 0u);
        const bool isown = lane < 32 && ((ownpos >> (lane & 31)) & 1u);
        const int b = isown ? rg + ostep * before : lane - before;
        tabx = (((isown ? rank : peer) * NBLK + (b & 31)) << 16) | (b < NBLK ? 0u : 1u);
        tabr = isown || b >= NBLK ? -1 : b;
    }
    void* const xall = me + WS_X + (size_t)(call & 1u) * NR * SHARD;
    const unsigned orow = q * 4 + (lane >> 4), voffO = orow * (NSH * 4) + (cg * 64 + (lane & 15) * 4) * 4;
    const unsigned lwb = 131072 + q * 4096 + (lane & 15) * 256 + (((lane >> 4) ^ (lane & 15)) << 4);
    LAS unsigned char* const lw0 = (LAS unsigned char*)lds + lwb; LAS unsigned char* const lw1 = (LAS unsigned char*)lds + (lwb ^ 64);
    LAS unsigned char* const lw2 = (LAS unsigned char*)lds + (lwb ^ 128); LAS unsigned char* const lw3 = (LAS unsigned char*)lds + (lwb ^ 192);
    LAS unsigned char* const lr = (LAS unsigned char*)lds + 131072 + orow * 256 + (((lane & 15) ^ orow) << 4);
    unsigned xlim = NR * SHARD;
    f32x4 cinit = {0.f, 0.f, 0.f, 0.f};
    unsigned xsz = NR * SHARD;

    auto count = [&](u32x2 f) { unsigned r; asm("s_ff1_i32_b64 %0, %1" : "=s"(r) : "s"(__ballot(f.x < call || f.y < call))); return (int)(r >> 1); };
    auto spin = [&](unsigned soff, int need) {
        const char* const pf = me + WS_FLAG + soff;
        const unsigned long long t0 = __builtin_amdgcn_s_memrealtime();
        asm volatile("" :: "s"(t0));
        for (;;) {
            u32x2 f; unsigned tmo;
            asm volatile("global_load_dwordx2 %0, %2, %3 sc0 sc1\n\tglobal_load_dword %1, %4, %5 sc1\n\ts_waitcnt vmcnt(0)"
                         : "=&v"(f), "=&v"(tmo) : "v"(vfl), "s"(pf), "v"(0u), "s"(me + WS_TMO) : "memory");
            const int c = count(f);
            if (c > need) return c;
            if (__builtin_amdgcn_readfirstlane(tmo) == call || __builtin_amdgcn_s_memrealtime() - t0 > DEADLINE_TICKS) {
                if (lane == 0) __hip_atomic_store((unsigned*)(me + WS_TMO), call, __ATOMIC_RELAXED, __HIP_MEMORY_SCOPE_AGENT);
                xlim = 0; xsz = 0; const float qn = __builtin_nanf(""); cinit = (f32x4){qn, qn, qn, qn};
                asm volatile("s_nop 1" : "+v"(cinit));
                return 2 * NBLK;
            }
            __builtin_amdgcn_s_sleep(1);
        }
    };
    unsigned voffR[8];
#pragma unroll
    for (int k = 0; k < 8; ++k) voffR[k] = vdst + q * 16384 + (unsigned)((k + rg) & 7) * 2048u;
    f32x4 c0 = cinit, c1 = cinit, c2 = cinit, c3 = cinit;
    asm volatile("s_nop 1" : "+v"(cinit), "+v"(c0), "+v"(c1), "+v"(c2), "+v"(c3));
#define XLOAD(X, k) (X).h[k] = __builtin_bit_cast(u32x4, __builtin_amdgcn_raw_buffer_load_b128(rx, voffR[(k) >> 1], sx + ((k) & 1) * 1024u, 17))
#define POLL(size, off) __builtin_bit_cast(u32x2, __builtin_amdgcn_raw_buffer_load_b64(__builtin_amdgcn_make_buffer_rsrc(me + WS_FLAG, 0, size, 0x00020000), vfl, off, 17))
    int rdy = rg == 7 ? 2 * NBLK : count(pf);
    unsigned xm = 1u, x0 = __builtin_amdgcn_readlane(tabx, 0), x1 = __builtin_amdgcn_readlane(tabx, 1);
    int r1 = __builtin_amdgcn_readlane(tabr, 1), need = r1 + 1, thr = need;
    int sp0 = NBLK;
    unsigned sx, oso = 0, osz = 0, psz = NR * 2048;
    auto store_rows = [&](const f32x4& r) {
        __builtin_amdgcn_raw_buffer_store_b128(__builtin_bit_cast(u32x4, r), __builtin_amdgcn_make_buffer_rsrc((void*)p.out, 0, osz, 0x00020000), voffO, oso, OUT_AUX);
    };
    static_assert(NSPEC % 2 == 0 && NSPEC <= 14, "");
    char* const pz = (char*)xall + (rg < 7 ? (size_t)peer * SHARD + (NBLK - NSPEC) * 65536 + (cg * 4 + q) * 1024 : (size_t)rank * SHARD + (bid & 7) * 4096 + q * 1024);
    const unsigned pzs = rg < 7 ? 131072u : 0u;
    auto fill_piece = [&](int k) {
        const u32x4 ff = {~0u, ~0u, ~0u, ~0u};
        asm volatile("global_store_dwordx4 %0, %1, %2 sc0 sc1\n\ts_nop 1" :: "v"(vdst), "v"(ff), "s"(pz + k * pzs) : "memory");
    };
#define SB __builtin_amdgcn_sched_barrier(0);
#define XFRAG(X, S) __builtin_shufflevector((X).h[2 * (S)], (X).h[2 * (S) + 1], 0, 1, 2, 3, 4, 5, 6, 7)
#define PIN(v) asm volatile("" : "+s"(v));
#define SLOT(FIRST, CUR, NXT, PP, PO, PN, J, C) \
        mf<(J) / 4, C>(c##C, XFRAG(CUR, (J) / 4), cinit); SB \
        if constexpr ((J) == 0) *(LAS f32x4*)(lw2 + PP * 16384) = c2; \
        if constexpr ((J) == 1) *(LAS f32x4*)(lw3 + PP * 16384) = c3; \
        if constexpr ((J) == 2) *(LAS f32x4*)(lw0 + PP * 16384) = c0; \
        if constexpr ((J) == 31) *(LAS f32x4*)(lw1 + (1 - PP) * 16384) = c1; \
        if constexpr ((J) == S_POLL) PN = POLL(psz, sfl); \
        if constexpr (FIRST && (J) == S_POLL - 2) pf = POLL(NR * 2048, sflo); \
        if constexpr (FIRST && ((J) & 3) == 2 && (J) / 4 < NSPEC / 2) fill_piece((J) / 4); \
        if constexpr ((J) & 1) XLOAD(NXT, (J) >> 1); \
        if constexpr ((J) == S_BAR) { wait_lgkm0(); __builtin_amdgcn_s_barrier(); asm volatile("" ::: "memory"); } \
        if constexpr ((J) == S_RD) { q0 = *(const LAS f32x4*)(lr + PP * 16384); q1 = *(const LAS f32x4*)(lr + PP * 16384 + 4096); \
                                     q2 = *(const LAS f32x4*)(lr + PP * 16384 + 8192); q3 = *(const LAS f32x4*)(lr + PP * 16384 + 12288); } \
        if constexpr ((J) == S_A1) q1 = fadd4(q0, q1); \
        if constexpr ((J) == S_A2) q2 = fadd4(q2, q3); \
        if constexpr ((J) == S_A3) q1 = fadd4(q1, q2); \
        if constexpr ((J) == S_MU) q1 = fmul4(q1, scale); \
        if constexpr ((J) == S_ST) store_rows(q1); \
        if constexpr ((J) == S_K1) { PIN(r1) sp1 = r1 >= rdy ? r1 : NBLK; PIN(sp1) PIN(i) i2 = i + 2; PIN(i2) } \
        if constexpr ((J) == S_K2) { PIN(xm) oso = xm << 1; osz = (xm & 1u) ? 0u : (unsigned)(NR * MSH) * NSH * 4u; PIN(oso) PIN(osz) } \
        if constexpr ((J) == S_K3) { PIN(i2) xm = x0; x0 = x1; x1 = __builtin_amdgcn_readlane(tabx, i2); r1 = __builtin_amdgcn_readlane(tabr, i2); PIN(x1) PIN(r1) } \
        if constexpr ((J) == S_K3 + 1) { PIN(x1) xszn = (x1 & 1u) ? 0u : xlim; PIN(xszn) } \
        if constexpr ((J) == S_K4) { PIN(r1) int r2 = r1 + 1; PIN(r2) need = r1 < NBLK - NSPEC ? r2 : r1; PIN(need) } \
        if constexpr ((J) == S_K4 + 1) { PIN(need) thr = sp0 < NBLK ? -1 : need; ++i; PIN(thr) PIN(i) } \
        if constexpr ((J) == S_K5) { asm volatile("" : "+v"(PO)); const int c = count(PO); rdy = c > rdy ? c : rdy; PIN(rdy) }     \
        if constexpr ((J) == S_K6) { PIN(need) psz = rdy < need + POLL_AHEAD ? NR * 2048 : 0u; PIN(psz) } \
        SB
#define STEP4(FIRST, CUR, NXT, PP, PO, PN, S) SLOT(FIRST, CUR, NXT, PP, PO, PN, 4 * S, 1) SLOT(FIRST, CUR, NXT, PP, PO, PN, 4 * S + 1, 2) SLOT(FIRST, CUR, NXT, PP, PO, PN, 4 * S + 2, 3) SLOT(FIRST, CUR, NXT, PP, PO, PN, 4 * S + 3, 0)
#define RECHECK(PP) \
        if (spm < NBLK) { \
            asm volatile("s_nop 7\n\ts_nop 7" : "+v"(c0)); \
            if (__any(c0[0] != c0[0])) { \
                rdy = spin(sfl, spm); \
                const __amdgpu_buffer_rsrc_t rx = __builtin_amdgcn_make_buffer_rsrc(xall, 0, xlim, 0x00020000); \
                sx = xm; \
                XBlk T; \
                _Pragma("unroll") for (int k = 0; k < 16; ++k) XLOAD(T, k); \
                M4(T, 0) M4(T, 1) M4(T, 2) M4(T, 3) M4(T, 4) M4(T, 5) M4(T, 6) M4(T, 7) \
                asm volatile("s_nop 7\n\ts_nop 7" : "+v"(c0), "+v"(c1), "+v"(c2), "+v"(c3)); \
                *(LAS f32x4*)(lw1 + PP * 16384) = c1; \
            } \
        }
#define M4(X, S) mf<S, 1>(c1, XFRAG(X, S), cinit); mf<S, 2>(c2, XFRAG(X, S), cinit); mf<S, 3>(c3, XFRAG(X, S), cinit); mf<S, 0>(c0, XFRAG(X, S), cinit);
#define BODY(FIRST, CUR, NXT, PP, PO, PN) { \
        if (__builtin_expect((unsigned)rdy < (unsigned)thr, 0)) { \
            RECHECK(PP) \
            if (rdy < need) { const int c = count(PO); rdy = c > rdy ? c : rdy; }     \
            if (rdy < need) rdy = spin(sfl, need - 1); \
        } \
        sx = x1; \
        const __amdgpu_buffer_rsrc_t rx = __builtin_amdgcn_make_buffer_rsrc(xall, 0, xsz, 0x00020000); \
        f32x4 q0, q1, q2, q3; unsigned xszn; int sp1, i2; \
        STEP4(FIRST, CUR, NXT, PP, PO, PN, 0) STEP4(FIRST, CUR, NXT, PP, PO, PN, 1) STEP4(FIRST, CUR, NXT, PP, PO, PN, 2) STEP4(FIRST, CUR, NXT, PP, PO, PN, 3) \
        STEP4(FIRST, CUR, NXT, PP, PO, PN, 4) STEP4(FIRST, CUR, NXT, PP, PO, PN, 5) STEP4(FIRST, CUR, NXT, PP, PO, PN, 6) STEP4(FIRST, CUR, NXT, PP, PO, PN, 7) \
        xsz = xszn; spm = sp0; sp0 = sp1; }

    int i = 0, spm = NBLK;
    u32x2 P0 = {0u, 0u}, P1;
    if (rg == 7) spin(sflo, NBLK - 1);
    BODY(true, X1, X0, 1, P0, P1)
    if (ownpos != 1u && count(pf) < NBLK) spin(sflo, NBLK - 1);
    do {
        BODY(false, X0, X1, 0, P1, P0) BODY(false, X1, X0, 1, P0, P1)
        if (__builtin_expect(i == 3 && q == 0 && rg < 7, 0))
            asm volatile("s_mov_b64 exec, 1\n\tglobal_store_dword %0, %1, %2 sc0 sc1\n\ts_mov_b64 exec, -1" :: "v"(0u), "v"(call), "s"(p.wsr[rg] + WS_RTR + rank * 128 + cg * 4) : "memory");
    } while (i < n);
    RECHECK(0)
    oso = xm << 1; osz = (unsigned)(NR * MSH) * NSH * 4u;
    {
        asm volatile("s_nop 7\n\ts_nop 7" : "+v"(c0), "+v"(c2), "+v"(c3));
        *(LAS f32x4*)(lw2 + 0 * 16384) = c2; *(LAS f32x4*)(lw3 + 0 * 16384) = c3; *(LAS f32x4*)(lw0 + 0 * 16384) = c0;
        wait_lgkm0(); __builtin_amdgcn_s_barrier(); asm volatile("" ::: "memory");
        store_rows(fmul4(fadd4(fadd4(*(const LAS f32x4*)(lr + 0 * 16384), *(const LAS f32x4*)(lr + 0 * 16384 + 4096)), fadd4(*(const LAS f32x4*)(lr + 0 * 16384 + 8192), *(const LAS f32x4*)(lr + 0 * 16384 + 12288))), scale));
        asm volatile(".Lfk_code_end:");
    }
#undef BODY
#undef M4
#undef RECHECK
#undef STEP4
#undef SLOT
#undef XFRAG
#undef SB
#undef XLOAD
#undef POLL
#undef PIN
}
}

extern "C" void kernel_launch(int rank, int nranks, ncclComm_t comm,
                              hipStream_t stream,
                              void* const* d_in, const int* in_sizes, int n_in,
                              void* d_out, int out_size,
                              const int* in_dtypes, int out_dtype,
                              void* const* d_out_all,
                              void* const* d_ws_all, size_t ws_bytes) {
    (void)comm; (void)in_dtypes; (void)out_dtype; (void)d_out_all;
    if (nranks != NR || (unsigned)rank >= (unsigned)NR || n_in != 4 || ws_bytes < WS_END ||
        in_sizes[0] != MSH * KDIM || in_sizes[1] != KDIM * NFULL || out_size != NR * MSH * NSH) {
        fprintf(stderr, "kernel_launch: unexpected problem shape\n");
        return;
    }
    Params p;
    p.x = (const unsigned char*)d_in[0]; p.w = (const unsigned char*)d_in[1];
    p.sx = (const float*)d_in[2]; p.sw = (const float*)d_in[3]; p.out = (float*)d_out;
    for (int i = 0; i < NR; ++i) p.wsr[i] = (char*)d_ws_all[(rank + 1 + i) & 7];
    p.rank = rank; p.slow = (int)((3u - (unsigned)((size_t)p.w >> 7)) & 7u) * 2;
    fused_kernel<<<256, 256, 0, stream>>>(p);
    HIP_CHECK(hipGetLastError());
}
```

```cpp
#include <hip/hip_runtime.h>
#include <rccl/rccl.h>
#include <cstdio>

#define HIP_CHECK(expr) do { hipError_t _e = (expr); if (_e != hipSuccess) { \
    fprintf(stderr, "HIP error %s at %s:%d\n", hipGetErrorString(_e), __FILE__, __LINE__); \
    return; } } while (0)

namespace {
typedef unsigned u32x2 __attribute__((ext_vector_type(2)));
typedef unsigned u32x4 __attribute__((ext_vector_type(4)));
typedef unsigned u32x8 __attribute__((ext_vector_type(8)));
typedef float f32x4 __attribute__((ext_vector_type(4)));
#define LAS __attribute__((address_space(3)))
#define DI __device__ __forceinline__

constexpr int NR = 8;
constexpr int KDIM = 4096, NFULL = 16384, NSH = 2048, MSH = 512;
constexpr unsigned SHARD = MSH * KDIM;
constexpr int NBLK = MSH / 16;
constexpr int NUNIT = 512;
constexpr int WPP = 16;
constexpr size_t WS_X = 0;
constexpr size_t WS_FLAG = 2ull * NR * SHARD;
constexpr size_t WS_RTR = WS_FLAG + NR * 2048;
constexpr size_t WS_TICK = WS_RTR + NR * 128;
constexpr size_t WS_TMO = WS_TICK + 1024;
constexpr size_t WS_CALL = WS_TMO + 1024;
constexpr size_t WS_CNT = WS_CALL + 1024 * 4;
constexpr size_t WS_END = WS_CNT + 7 * 128 * 64;
constexpr unsigned long long DEADLINE_TICKS = 180000000ull;

constexpr unsigned OWNMASK = 0x155u;
constexpr int POLL_AHEAD = 3;
constexpr int NSPEC = 8;
constexpr unsigned NAGG = 26 * 16;
constexpr int OUT_AUX = 2;

struct Params {
    const unsigned char* x; const unsigned char* w; const float* sx; const float* sw; float* out;
    char* wsr[NR];
    int rank; int slow;
};

#define ACLOB "a0","a1","a2","a3","a4","a5","a6","a7","a8","a9","a10","a11","a12","a13","a14","a15","a16","a17","a18","a19","a20","a21","a22","a23","a24","a25","a26","a27","a28","a29","a30","a31","a32","a33","a34","a35","a36","a37","a38","a39","a40","a41","a42","a43","a44","a45","a46","a47","a48","a49","a50","a51","a52","a53","a54","a55","a56","a57","a58","a59","a60","a61","a62","a63","a64","a65","a66","a67","a68","a69","a70","a71","a72","a73","a74","a75","a76","a77","a78","a79","a80","a81","a82","a83","a84","a85","a86","a87","a88","a89","a90","a91","a92","a93","a94","a95","a96","a97","a98","a99","a100","a101","a102","a103","a104","a105","a106","a107","a108","a109","a110","a111","a112","a113","a114","a115","a116","a117","a118","a119","a120","a121","a122","a123","a124","a125","a126","a127","a128","a129","a130","a131","a132","a133","a134","a135","a136","a137","a138","a139","a140","a141","a142","a143","a144","a145","a146","a147","a148","a149","a150","a151","a152","a153","a154","a155","a156","a157","a158","a159","a160","a161","a162","a163","a164","a165","a166","a167","a168","a169","a170","a171","a172","a173","a174","a175","a176","a177","a178","a179","a180","a181","a182","a183","a184","a185","a186","a187","a188","a189","a190","a191","a192","a193","a194","a195","a196","a197","a198","a199","a200","a201","a202","a203","a204","a205","a206","a207","a208","a209","a210","a211","a212","a213","a214","a215","a216","a217","a218","a219","a220","a221","a222","a223","a224","a225","a226","a227","a228","a229","a230","a231","a232","a233","a234","a235","a236","a237","a238","a239","a240","a241","a242","a243","a244","a245","a246","a247","a248","a249","a250","a251","a252","a253","a254","a255"

template <int N> DI void wait_vm() { asm volatile("s_waitcnt vmcnt(%0)" :: "n"(N) : "memory"); }
DI void wait_lgkm0() { asm volatile("s_waitcnt lgkmcnt(0)" ::: "memory"); }
template <int B> DI void snd_load(const unsigned char* base, unsigned voff) {
    asm volatile("global_load_dwordx4 a[%c2:%c2+3], %0, %1\n\t"
                 "global_load_dwordx4 a[%c2+4:%c2+7], %0, %1 offset:64\n\t"
                 "global_load_dwordx4 a[%c2+8:%c2+11], %0, %1 offset:128\n\t"
                 "global_load_dwordx4 a[%c2+12:%c2+15], %0, %1 offset:192" :: "v"(voff), "s"(base), "i"(B) : "memory");
}
template <int B> DI void snd_store(char* base, unsigned voff) {
    asm volatile("global_store_dwordx4 %0, a[%c2:%c2+3], %1 sc0 sc1\n\t"
                 "global_store_dwordx4 %0, a[%c2+4:%c2+7], %1 offset:1024 sc0 sc1\n\t"
                 "global_store_dwordx4 %0, a[%c2+8:%c2+11], %1 offset:2048 sc0 sc1\n\t"
                 "global_store_dwordx4 %0, a[%c2+12:%c2+15], %1 offset:3072 sc0 sc1\n\ts_nop 1" :: "v"(voff), "s"(base), "i"(B) : "memory");
}
template <int A> DI void snd_ticket(char* ctr, unsigned vzero) {
    unsigned long long keep;
    asm volatile("s_mov_b64 %0, exec\n\ts_mov_b64 exec, 1\n\tglobal_atomic_add a[%c3], %1, a33, %2 sc0\n\ts_mov_b64 exec, %0"
                 : "=&s"(keep) : "v"(vzero), "s"(ctr), "i"(A) : "memory");
}
DI void code_prefetch(unsigned lane) {
    unsigned long long pc; unsigned t0, t1, t2;
    asm volatile("s_getpc_b64 %0\n"
                 ".Lfk_code_begin:\n\t"
                 "v_min_u32 %1, .Lfk_code_end-.Lfk_code_begin, %4\n\tv_min_u32 %2, .Lfk_code_end-.Lfk_code_begin, %5\n\tv_min_u32 %3, .Lfk_code_end-.Lfk_code_begin, %6\n\t"
                 "global_load_dword a252, %1, %0\n\tglobal_load_dword a253, %2, %0\n\tglobal_load_dword a254, %3, %0"
                 : "=&s"(pc), "=&v"(t0), "=&v"(t1), "=&v"(t2) : "v"(lane * 128u), "v"(lane * 128u + 8192u), "v"(lane * 128u + 16384u) : "memory", "a252", "a253", "a254");
}
template <int B, int H> DI void snd_half(char* base, unsigned voff) {
    asm volatile("global_store_dwordx4 %0, a[%c2:%c2+3], %1 offset:%c3 sc0 sc1\n\t"
                 "global_store_dwordx4 %0, a[%c2+4:%c2+7], %1 offset:%c3+1024 sc0 sc1\n\ts_nop 1" :: "v"(voff), "s"(base), "i"(B + 8 * H), "i"(2048 * H) : "memory");
}
template <int A> DI void call_load(const char* ctr, unsigned vzero) { asm volatile("global_load_dword a[%c2], %0, %1 sc0 sc1" :: "v"(vzero), "s"(ctr), "i"(A) : "memory"); }
template <int A> DI void snd_flag(char* flag, unsigned vzero) {
    unsigned long long keep;
    asm volatile("s_mov_b64 %0, exec\n\ts_mov_b64 exec, 1\n\tglobal_store_dword %1, a[%c3], %2 sc0 sc1\n\ts_mov_b64 exec, %0"
                 : "=&s"(keep) : "v"(vzero), "s"(flag), "i"(A) : "memory");
}
template <int A> DI void snd_flag4(char* flag, unsigned vzero) {
    unsigned long long keep;
    asm volatile("s_mov_b64 %0, exec\n\ts_mov_b64 exec, 1\n\tglobal_store_dwordx4 %1, a[%c3:%c3+3], %2 sc0 sc1\n\ts_mov_b64 exec, %0"
                 : "=&s"(keep) : "v"(vzero), "s"(flag), "i"(A) : "memory");
}
template <int A> DI void acc_set(unsigned v) { asm volatile("s_nop 1\n\tv_accvgpr_write_b32 a[%c1], %0\n\ts_nop 1" :: "s"(v), "i"(A)); }
DI unsigned unit_src(unsigned u) { return (u >> 4) * 65536u + (u & 15u) * 256u; }
DI unsigned unit_flag(unsigned u) { return (((u >> 2) & 3u) * 128u + (u >> 4) * 4u + (u & 3u)) * 4u; }
template <int N, int A = 32> DI unsigned snd_wait_ticket() {
    unsigned v;
    asm volatile("s_waitcnt vmcnt(%1)\n\tv_accvgpr_read_b32 %0, a[%c2]\n\ts_nop 1" : "=v"(v) : "n"(N), "i"(A) : "memory");
    return __builtin_amdgcn_readfirstlane(v);
}


template <int S, int SLOT> DI void w_read(unsigned a0, unsigned a1, unsigned a2, unsigned a3) {
    constexpr int B = S * 32, O = SLOT * 8192;
#define TRR(c, t, A) "ds_read_b64_tr_b8 a[%c4+" #c "*8+" #t "*2:%c4+" #c "*8+" #t "*2+1], " A " offset:%c5+4096*(" #t "/2)+512*(" #t "%%2)\n\t"
    asm volatile(TRR(0, 0, "%0") TRR(0, 1, "%0") TRR(0, 2, "%0") TRR(0, 3, "%0")
                 TRR(1, 0, "%1") TRR(1, 1, "%1") TRR(1, 2, "%1") TRR(1, 3, "%1")
                 TRR(2, 0, "%2") TRR(2, 1, "%2") TRR(2, 2, "%2") TRR(2, 3, "%2")
                 TRR(3, 0, "%3") TRR(3, 1, "%3") TRR(3, 2, "%3") TRR(3, 3, "%3")
                 "s_waitcnt lgkmcnt(0)"
                 :: "v"(a0), "v"(a1), "v"(a2), "v"(a3), "i"(B), "i"(O) : "memory", ACLOB);
#undef TRR
}
DI void w_stage(__amdgpu_buffer_rsrc_t rs, unsigned ve, unsigned vo, unsigned m0base, unsigned soff) {
    unsigned keep, so;
#define WST(V) "s_nop 0\n\tbuffer_load_dwordx4 " V ", %5, %1 offen lds\n\t"
#define WNX "s_add_u32 m0, m0, 0x400\n\ts_add_u32 %1, %1, 0x40000\n\t"
    asm volatile("s_mov_b32 %0, m0\n\ts_mov_b32 m0, %2\n\ts_mov_b32 %1, %6\n\t"
                 WST("%3") WNX WST("%4") WNX WST("%3") WNX WST("%4") WNX WST("%3") WNX WST("%4") WNX WST("%3") WNX WST("%4")
                 "s_mov_b32 m0, %0"
                 : "=&s"(keep), "=&s"(so) : "s"(m0base), "v"(ve), "v"(vo), "s"(rs), "s"(soff) : "scc", "memory");
#undef WST
#undef WNX
    static_assert(16 * NFULL == 0x40000, "");
}
DI void x_stage(const unsigned char* base, unsigned vdst, unsigned m0base) {
    unsigned keep, v;
#define XST(r) "v_xor_b32 %1, " #r "*16, %3\n\tv_add_u32 %1, " #r "*4096, %1\n\tglobal_load_lds_dwordx4 %1, %4\n\ts_add_u32 m0, m0, 0x400\n\t"
    asm volatile("s_mov_b32 %0, m0\n\ts_mov_b32 m0, %2\n\t"
                 XST(0) XST(1) XST(2) XST(3) XST(4) XST(5) XST(6) XST(7) XST(8) XST(9) XST(10) XST(11) XST(12) XST(13) XST(14) XST(15)
                 "s_mov_b32 m0, %0"
                 : "=&s"(keep), "=&v"(v) : "s"(m0base), "v"(vdst), "s"(base) : "scc", "memory");
#undef XST
}
template <int S, int C> DI void mf(f32x4& c, const u32x8& x, const f32x4& cinit) {
    constexpr int B = S * 32 + C * 8;
    if constexpr (S == 0) asm volatile("v_mfma_f32_16x16x128_f8f6f4 %0, a[%c3:%c3+7], %1, %2" : "+v"(c) : "v"(x), "v"(cinit), "i"(B));
    else asm volatile("v_mfma_f32_16x16x128_f8f6f4 %0, a[%c2:%c2+7], %1, %0" : "+v"(c) : "v"(x), "i"(B));
}
DI float fadd(float a, float b) { float r; asm("v_add_f32_e32 %0, %1, %2" : "=v"(r) : "v"(a), "v"(b)); return r; }
DI float fmul(float a, float b) { float r; asm("v_mul_f32_e32 %0, %1, %2" : "=v"(r) : "v"(a), "v"(b)); return r; }
DI f32x4 fadd4(f32x4 a, f32x4 b) { return (f32x4){fadd(a[0], b[0]), fadd(a[1], b[1]), fadd(a[2], b[2]), fadd(a[3], b[3])}; }
DI f32x4 fmul4(f32x4 a, float b) { return (f32x4){fmul(a[0], b), fmul(a[1], b), fmul(a[2], b), fmul(a[3], b)}; }

constexpr int S_K1 = 4, S_BAR = 6, S_RD = 8, S_K2 = 10, S_A1 = 12, S_A2 = 14, S_POLL = 16, S_A3 = 18, S_MU = 20, S_ST = 22, S_K3 = 24, S_K4 = 26, S_K5 = 28, S_K6 = 30;

struct XBlk { u32x4 h[16]; };

__global__ __launch_bounds__(256, 1) void fused_kernel(Params p) {
    __shared__ __attribute__((aligned(16))) unsigned char lds[4 * 32768 + 32768];
    const int lane = threadIdx.x & 63;
    const int wid = __builtin_amdgcn_readfirstlane(threadIdx.x >> 6);
    const int rank = p.rank;
    char* const me = p.wsr[7];
    const int bid = blockIdx.x;
    code_prefetch(lane);
    const bool sender = bid < 24 || (bid < 32 && (bid & 2));
    const int w = (bid < 24 ? bid : 24 + (bid & 1) + ((bid >> 1) & 2)) * 4 + wid, pi = w % 7;
    char* const pws = p.wsr[pi];
    asm volatile("" :: "s"(p.x), "s"(p.w), "s"(me), "s"(rank), "s"(pws));

    char* const callp = me + WS_CALL + (bid * 4 + wid) * 4;

    const unsigned vsrc = (lane & 15) * KDIM + (lane >> 4) * 16, vdst = lane * 16;

    if (__builtin_expect(sender, 1)) {
        const int j = w / 7;
        unsigned vzero = 0;
        asm volatile("v_accvgpr_write_b32 a33, 1" : "+v"(vzero) :: "a32", "a33", "a35", "a36", "a37", "a38", "a43", "a111");
        auto srcp = [&](unsigned u) { return p.x + unit_src(u < NUNIT ? u : NUNIT - 1); };
        unsigned u0 = j, u1 = j + WPP, u2 = j + 2 * WPP, prev, pp = NUNIT;
        char* const ctr = me + WS_TICK + pi * 128;
        asm volatile("s_load_dword s96, %0, 0x0\n\ts_load_dword s97, %1, 0x0\n\ts_load_dword s98, %2, 0x0"
                     :: "s"(pws + WS_X + (size_t)rank * SHARD), "s"(pws + WS_X + (size_t)(NR + rank) * SHARD), "s"(pws + WS_FLAG + rank * 2048) : "memory", "s96", "s97", "s98");
        call_load<35>(callp, vzero);
        snd_load<64>(srcp(u0), vsrc);
        snd_load<80>(srcp(u1), vsrc);
        const unsigned call = snd_wait_ticket<4, 35>() + 1u;
        char* const dst = pws + WS_X + (size_t)((call & 1u) * NR + rank) * SHARD;
        snd_half<64, 0>(dst + u0 * 4096u, vdst);
        snd_ticket<32>(ctr, vzero);
        snd_load<96>(srcp(u2), vsrc);
        snd_half<64, 1>(dst + u0 * 4096u, vdst);
        acc_set<40>(call); acc_set<41>(call); acc_set<42>(call); acc_set<43>(call);
        const unsigned tbase = (call - 1u) * NUNIT - 3u * WPP;
        char* const fl = pws + WS_FLAG + rank * 2048;
        char* const own = me + WS_X + (size_t)((call & 1u) * NR + rank) * SHARD;
        prev = u0; u0 = u1; u1 = u2;
        wait_vm<7>();
        wait_lgkm0();
        int fill = 0;
        char* const cnt = me + WS_CNT + pi * (128 * 64);
        auto filler = [&]() { asm volatile("global_load_dword a38, %0, %1" :: "v"(vzero), "s"(p.x) : "memory"); };
        auto publish = [&](unsigned u) {
            if (fill == 4 || u >= NUNIT) filler(); else if (u < NAGG) snd_ticket<37>(cnt + (u >> 2) * 64, vzero); else snd_flag<40>(fl + unit_flag(u), vzero);
        };
        auto publish4 = [&](unsigned u) {
            unsigned c; asm volatile("v_accvgpr_read_b32 %0, a37\n\ts_nop 1" : "=v"(c));
            if (u < NAGG && fill != 4 && (__builtin_amdgcn_readfirstlane(c) & 3) == 3) snd_flag4<40>(fl + unit_flag(u & ~3u), vzero); else filler();
        };
        const char* const rtr = pws + WS_RTR + rank * 128;
#define SEND_STEP(BS, BL, NA) { \
            if (u0 >= NUNIT - 16 * NSPEC - 64 && fill == 0) { asm volatile("global_load_dword a36, %0, %1 sc0 sc1" :: "v"((lane & 31) * 4), "s"(rtr) : "memory"); fill = 1; } \
            if (u0 >= NUNIT - 16 * NSPEC && (fill & 1)) { \
                  \
                wait_vm<0>(); publish4(pp); publish(prev); wait_vm<0>(); publish4(prev); pp = prev = NUNIT; \
                const unsigned long long t0 = __builtin_amdgcn_s_memrealtime(); \
                for (fill = 2;;) { \
                    unsigned f, tmo; \
                    asm volatile("global_load_dword %0, %2, %3 sc0 sc1\n\tglobal_load_dword %1, %4, %5 sc1\n\ts_waitcnt vmcnt(0)" \
                                 : "=&v"(f), "=&v"(tmo) : "v"((lane & 31) * 4), "s"(rtr), "v"(vzero), "s"(me + WS_TMO) : "memory"); \
                    if (__all(f >= call)) break; \
                    if (__builtin_amdgcn_readfirstlane(tmo) == call || __builtin_amdgcn_s_memrealtime() - t0 > DEADLINE_TICKS) { snd_flag<40>(me + WS_TMO, vzero); fill = 4; break; } \
                    __builtin_amdgcn_s_sleep(4); \
                } \
            } \
            char* const d = (fill == 4 ? own : dst) + u0 * 4096u; \
            snd_half<BS, 0>(d, vdst); \
            u2 = snd_wait_ticket<NA>() - tbase; \
            snd_ticket<32>(ctr, vzero); \
            snd_load<BL>(srcp(u2), vsrc); \
            wait_vm<7>();                                                \
            publish4(pp); publish(prev); \
            snd_half<BS, 1>(d, vdst); \
            pp = prev; prev = u0; u0 = u1; u1 = u2; \
            wait_vm<9>();                                                \
            if (fill == 1) { unsigned f; asm volatile("v_accvgpr_read_b32 %0, a36" : "=v"(f)); fill = __all(f >= call) ? 2 : 3; } }
        SEND_STEP(80, 64, 8)
        for (;;) {
            SEND_STEP(96, 80, 10) if (u0 >= NUNIT) break;
            SEND_STEP(64, 96, 10) if (u0 >= NUNIT) break;
            SEND_STEP(80, 64, 10) if (u0 >= NUNIT) break;
        }
#undef SEND_STEP
        wait_vm<0>();
        publish4(pp); publish(prev);
        if (prev < NAGG) { wait_vm<0>(); publish4(prev); }
        snd_flag<40>(callp, vzero);
        return;
    }

    int cg, rg;
    {
        const int xcd = bid & 7, i = (xcd & 1) * 28 + (bid >> 3) - 4;
        rg = bid < 32 ? 7 : i >> 3;
        cg = bid < 32 ? p.slow + (xcd & 1) + (xcd >> 2) * 16 : (xcd >> 1) * 8 + (i & 7);
    }
    const int q = wid;
    const float scale = p.sx[0] * p.sw[0];
    unsigned call;
    const unsigned ldsw = (unsigned)(size_t)(LAS unsigned char*)lds + q * 32768u;
    const unsigned peer = (unsigned)(rank + 1 + rg) & 7u;
    const unsigned sfl = peer * 2048 + q * 512, sflo = rank * 2048 + q * 512, vfl = lane * 8;
    XBlk X0, X1;
    u32x2 pf;
    {
        const __amdgpu_buffer_rsrc_t rw = __builtin_amdgcn_make_buffer_rsrc((void*)p.w, 0, (unsigned)KDIM * NFULL, 0x00020000);
        const unsigned ve = (lane >> 2) * NFULL + (((lane & 3) ^ ((lane >> 4) & 1)) * 16), vo = ve ^ 32u;
        const unsigned so = (unsigned)(q * 1024) * NFULL + rank * NSH + cg * 64;
        const unsigned ab = ldsw + ((lane >> 4) * 16 + ((lane & 15) >> 1)) * 64 + (lane & 1) * 8, f = (lane >> 3) & 3;
        const unsigned a0 = ab + ((0 ^ f) << 4), a1 = ab + ((1 ^ f) << 4), a2 = ab + ((2 ^ f) << 4), a3 = ab + ((3 ^ f) << 4);
        constexpr unsigned SS = 128u * NFULL;
        unsigned vzero = 0;
        asm volatile("" : "+v"(vzero));
        call_load<32>(callp, vzero);
#define WOFF(S) (so + (unsigned)(((S) + rg) & 7) * SS)
#define WFIRST w_stage(rw, ve, vo, ldsw + 0 * 8192, WOFF(0)); w_stage(rw, ve, vo, ldsw + 1 * 8192, WOFF(1)); \
               w_stage(rw, ve, vo, ldsw + 2 * 8192, WOFF(2)); w_stage(rw, ve, vo, ldsw + 3 * 8192, WOFF(3));
        const bool copier = rg >= 3 && rg < 7;
        const unsigned uo = (unsigned)(((rg - 3) * 32 + cg) * 4 + q);
        if (copier) snd_load<224>(p.x + unit_src(uo), vsrc);
        if (rg < 7) {
            const unsigned l0 = (unsigned)(((bid >> 3) - 4) * 4 + q) * 64u + lane, cb = rank * NSH + (cg & 24) * 64;
            const unsigned l2 = l0 + 2 * 7168u;
            asm volatile("buffer_load_dword a249, %0, %3, %4 offen\n\tbuffer_load_dword a250, %1, %3, %4 offen\n\tbuffer_load_dword a251, %2, %3, %4 offen"
                         :: "v"((l0 >> 2) * NFULL + (l0 & 3) * 128u), "v"(((l0 + 7168u) >> 2) * NFULL + (l0 & 3) * 128u),
                            "v"(((l2 < 16384u ? l2 : l0) >> 2) * NFULL + (l0 & 3) * 128u), "s"(rw), "s"(cb) : "memory", "a249", "a250", "a251");
        }
        if (copier) {
            call = snd_wait_ticket<3>() + 1u;
            acc_set<240>(call);
            snd_store<224>(me + WS_X + (size_t)((call & 1u) * NR + rank) * SHARD + uo * 4096u, vdst);
            WFIRST
            wait_vm<32>();
            snd_flag<240>(me + WS_FLAG + rank * 2048 + unit_flag(uo), vzero);
            wait_vm<24>();
        } else {
            WFIRST
            call = snd_wait_ticket<24>() + 1u;
        }
#undef WFIRST
        w_read<0, 0>(a0, a1, a2, a3); w_stage(rw, ve, vo, ldsw + 0 * 8192, WOFF(4));
        wait_vm<24>(); w_read<1, 1>(a0, a1, a2, a3); w_stage(rw, ve, vo, ldsw + 1 * 8192, WOFF(5));
        wait_vm<24>(); w_read<2, 2>(a0, a1, a2, a3); w_stage(rw, ve, vo, ldsw + 2 * 8192, WOFF(6));
        wait_vm<24>(); w_read<3, 3>(a0, a1, a2, a3); w_stage(rw, ve, vo, ldsw + 3 * 8192, WOFF(7));
#undef WOFF
        wait_vm<24>(); w_read<4, 0>(a0, a1, a2, a3);
        wait_vm<16>(); w_read<5, 1>(a0, a1, a2, a3);
        pf = __builtin_bit_cast(u32x2, __builtin_amdgcn_raw_buffer_load_b64(__builtin_amdgcn_make_buffer_rsrc(me + WS_FLAG, 0, NR * 2048, 0x00020000), vfl, sfl, 17));
        x_stage(p.x + rg * 65536 + q * 1024, vdst, ldsw);
        wait_vm<8 + 17>(); w_read<6, 2>(a0, a1, a2, a3);
        wait_vm<17>();     w_read<7, 3>(a0, a1, a2, a3);
        wait_vm<0>();
        asm volatile("s_mov_b64 exec, 1\n\tglobal_store_dword %0, %1, %2 sc0 sc1\n\ts_mov_b64 exec, -1" :: "v"(vzero), "v"(call), "s"(callp) : "memory");
        const unsigned xb = q * 32768 + (lane & 15) * 1024 + (((lane >> 4) ^ (lane & 3)) << 4), xs = (lane & 12) << 4;
#pragma unroll
        for (int k = 0; k < 16; ++k) X1.h[k] = *(const LAS u32x4*)((LAS unsigned char*)lds + xb + (((unsigned)((k + 2 * rg) & 15) << 6) ^ xs));
    }
    static_assert(__builtin_popcount(OWNMASK) == 5 && (OWNMASK & 3) == 1, "");
    const bool slowcg = (cg & 14) == p.slow;
    const int n = rg == 7 ? NBLK - 7 : slowcg ? NBLK + 1 : NBLK + 5, ostep = rg == 7 ? 1 : 7;
    const unsigned ownpos = rg == 7 ? ~0u : slowcg ? 1u : OWNMASK;
    unsigned tabx; int tabr;
    {
        const int before = __builtin_amdgcn_mbcnt_lo(ownpos, 0u);
        const bool isown = lane < 32 && ((ownpos >> (lane & 31)) & 1u);
        const int b = isown ? rg + ostep * before : lane - before;
        tabx = (((isown ? rank : peer) * NBLK + (b & 31)) << 16) | (b < NBLK ? 0u : 1u);
        tabr = isown || b >= NBLK ? -1 : b;
    }
    void* const xall = me + WS_X + (size_t)(call & 1u) * NR * SHARD;
    const unsigned orow = q * 4 + (lane >> 4), voffO = orow * (NSH * 4) + (cg * 64 + (lane & 15) * 4) * 4;
    const unsigned lwb = 131072 + q * 4096 + (lane & 15) * 256 + (((lane >> 4) ^ (lane & 15)) << 4);
    LAS unsigned char* const lw0 = (LAS unsigned char*)lds + lwb; LAS unsigned char* const lw1 = (LAS unsigned char*)lds + (lwb ^ 64);
    LAS unsigned char* const lw2 = (LAS unsigned char*)lds + (lwb ^ 128); LAS unsigned char* const lw3 = (LAS unsigned char*)lds + (lwb ^ 192);
    LAS unsigned char* const lr = (LAS unsigned char*)lds + 131072 + orow * 256 + (((lane & 15) ^ orow) << 4);
    unsigned xlim = NR * SHARD;
    f32x4 cinit = {0.f, 0.f, 0.f, 0.f};
    unsigned xsz = NR * SHARD;

    auto count = [&](u32x2 f) { unsigned r; asm("s_ff1_i32_b64 %0, %1" : "=s"(r) : "s"(__ballot(f.x < call || f.y < call))); return (int)(r >> 1); };
    auto spin = [&](unsigned soff, int need) {
        const char* const pf = me + WS_FLAG + soff;
        const unsigned long long t0 = __builtin_amdgcn_s_memrealtime();
        asm volatile("" :: "s"(t0));
        for (;;) {
            u32x2 f; unsigned tmo;
            asm volatile("global_load_dwordx2 %0, %2, %3 sc0 sc1\n\tglobal_load_dword %1, %4, %5 sc1\n\ts_waitcnt vmcnt(0)"
                         : "=&v"(f), "=&v"(tmo) : "v"(vfl), "s"(pf), "v"(0u), "s"(me + WS_TMO) : "memory");
            const int c = count(f);
            if (c > need) return c;
            if (__builtin_amdgcn_readfirstlane(tmo) == call || __builtin_amdgcn_s_memrealtime() - t0 > DEADLINE_TICKS) {
                if (lane == 0) __hip_atomic_store((unsigned*)(me + WS_TMO), call, __ATOMIC_RELAXED, __HIP_MEMORY_SCOPE_AGENT);
                xlim = 0; xsz = 0; const float qn = __builtin_nanf(""); cinit = (f32x4){qn, qn, qn, qn};
                asm volatile("s_nop 1" : "+v"(cinit));
                return 2 * NBLK;
            }
            __builtin_amdgcn_s_sleep(1);
        }
    };
    unsigned voffR[8];
#pragma unroll
    for (int k = 0; k < 8; ++k) voffR[k] = vdst + q * 16384 + (unsigned)((k + rg) & 7) * 2048u;
    f32x4 c0 = cinit, c1 = cinit, c2 = cinit, c3 = cinit;
    asm volatile("s_nop 1" : "+v"(cinit), "+v"(c0), "+v"(c1), "+v"(c2), "+v"(c3));
#define XLOAD(X, k) (X).h[k] = __builtin_bit_cast(u32x4, __builtin_amdgcn_raw_buffer_load_b128(rx, voffR[(k) >> 1], sx + ((k) & 1) * 1024u, 17))
#define POLL(size, off) __builtin_bit_cast(u32x2, __builtin_amdgcn_raw_buffer_load_b64(__builtin_amdgcn_make_buffer_rsrc(me + WS_FLAG, 0, size, 0x00020000), vfl, off, 17))
    int rdy = rg == 7 ? 2 * NBLK : count(pf);
    unsigned xm = 1u, x0 = __builtin_amdgcn_readlane(tabx, 0), x1 = __builtin_amdgcn_readlane(tabx, 1);
    int r1 = __builtin_amdgcn_readlane(tabr, 1), need = r1 + 1, thr = need;
    int sp0 = NBLK;
    unsigned sx, oso = 0, osz = 0, psz = NR * 2048;
    auto store_rows = [&](const f32x4& r) {
        __builtin_amdgcn_raw_buffer_store_b128(__builtin_bit_cast(u32x4, r), __builtin_amdgcn_make_buffer_rsrc((void*)p.out, 0, osz, 0x00020000), voffO, oso, OUT_AUX);
    };
    static_assert(NSPEC % 2 == 0 && NSPEC <= 14, "");
    char* const pz = (char*)xall + (rg < 7 ? (size_t)peer * SHARD + (NBLK - NSPEC) * 65536 + cg * 512 : (size_t)rank * SHARD) + q * 16384;
    const unsigned pzs = rg < 7 ? 131072u : 0u, vpz = rg < 7 ? (lane >> 5) * 65536u + (lane & 31) * 16u : vdst;
    auto fill_piece = [&](int k) {
        const u32x4 ff = {~0u, ~0u, ~0u, ~0u};
        asm volatile("global_store_dwordx4 %0, %1, %2 sc0 sc1\n\ts_nop 1" :: "v"(vpz), "v"(ff), "s"(pz + k * pzs) : "memory");
    };
#define SB __builtin_amdgcn_sched_barrier(0);
#define XFRAG(X, S) __builtin_shufflevector((X).h[2 * (S)], (X).h[2 * (S) + 1], 0, 1, 2, 3, 4, 5, 6, 7)
#define PIN(v) asm volatile("" : "+s"(v));
#define SLOT(FIRST, CUR, NXT, PP, PO, PN, J, C) \
        mf<(J) / 4, C>(c##C, XFRAG(CUR, (J) / 4), cinit); SB \
        if constexpr ((J) == 0) *(LAS f32x4*)(lw2 + PP * 16384) = c2; \
        if constexpr ((J) == 1) *(LAS f32x4*)(lw3 + PP * 16384) = c3; \
        if constexpr ((J) == 2) *(LAS f32x4*)(lw0 + PP * 16384) = c0; \
        if constexpr ((J) == 31) *(LAS f32x4*)(lw1 + (1 - PP) * 16384) = c1; \
        if constexpr ((J) == S_POLL) PN = POLL(psz, sfl); \
        if constexpr (FIRST && (J) == S_POLL - 2) pf = POLL(NR * 2048, sflo); \
        if constexpr (FIRST && ((J) & 3) == 2 && (J) / 4 < NSPEC / 2) fill_piece((J) / 4); \
        if constexpr ((J) & 1) XLOAD(NXT, (J) >> 1); \
        if constexpr ((J) == S_BAR) { wait_lgkm0(); __builtin_amdgcn_s_barrier(); asm volatile("" ::: "memory"); } \
        if constexpr ((J) == S_RD) { q0 = *(const LAS f32x4*)(lr + PP * 16384); q1 = *(const LAS f32x4*)(lr + PP * 16384 + 4096); \
                                     q2 = *(const LAS f32x4*)(lr + PP * 16384 + 8192); q3 = *(const LAS f32x4*)(lr + PP * 16384 + 12288); } \
        if constexpr ((J) == S_A1) q1 = fadd4(q0, q1); \
        if constexpr ((J) == S_A2) q2 = fadd4(q2, q3); \
        if constexpr ((J) == S_A3) q1 = fadd4(q1, q2); \
        if constexpr ((J) == S_MU) q1 = fmul4(q1, scale); \
        if constexpr ((J) == S_ST) store_rows(q1); \
        if constexpr ((J) == S_K1) { PIN(r1) sp1 = r1 >= rdy ? r1 : NBLK; PIN(sp1) PIN(i) i2 = i + 2; PIN(i2) } \
        if constexpr ((J) == S_K2) { PIN(xm) oso = xm << 1; osz = (xm & 1u) ? 0u : (unsigned)(NR * MSH) * NSH * 4u; PIN(oso) PIN(osz) } \
        if constexpr ((J) == S_K3) { PIN(i2) xm = x0; x0 = x1; x1 = __builtin_amdgcn_readlane(tabx, i2); r1 = __builtin_amdgcn_readlane(tabr, i2); PIN(x1) PIN(r1) } \
        if constexpr ((J) == S_K3 + 1) { PIN(x1) xszn = (x1 & 1u) ? 0u : xlim; PIN(xszn) } \
        if constexpr ((J) == S_K4) { PIN(r1) int r2 = r1 + 1; PIN(r2) need = r1 < NBLK - NSPEC ? r2 : r1; PIN(need) } \
        if constexpr ((J) == S_K4 + 1) { PIN(need) thr = sp0 < NBLK ? -1 : need; ++i; PIN(thr) PIN(i) } \
        if constexpr ((J) == S_K5) { asm volatile("" : "+v"(PO)); const int c = count(PO); rdy = c > rdy ? c : rdy; PIN(rdy) }     \
        if constexpr ((J) == S_K6) { PIN(need) psz = rdy < need + POLL_AHEAD ? NR * 2048 : 0u; PIN(psz) } \
        SB
#define STEP4(FIRST, CUR, NXT, PP, PO, PN, S) SLOT(FIRST, CUR, NXT, PP, PO, PN, 4 * S, 1) SLOT(FIRST, CUR, NXT, PP, PO, PN, 4 * S + 1, 2) SLOT(FIRST, CUR, NXT, PP, PO, PN, 4 * S + 2, 3) SLOT(FIRST, CUR, NXT, PP, PO, PN, 4 * S + 3, 0)
#define RECHECK(PP) \
        if (spm < NBLK) { \
            asm volatile("s_nop 7\n\ts_nop 7" : "+v"(c0)); \
            if (__any(c0[0] != c0[0])) { \
                rdy = spin(sfl, spm); \
                const __amdgpu_buffer_rsrc_t rx = __builtin_amdgcn_make_buffer_rsrc(xall, 0, xlim, 0x00020000); \
                sx = xm; \
                XBlk T; \
                _Pragma("unroll") for (int k = 0; k < 16; ++k) XLOAD(T, k); \
                M4(T, 0) M4(T, 1) M4(T, 2) M4(T, 3) M4(T, 4) M4(T, 5) M4(T, 6) M4(T, 7) \
                asm volatile("s_nop 7\n\ts_nop 7" : "+v"(c0), "+v"(c1), "+v"(c2), "+v"(c3)); \
                *(LAS f32x4*)(lw1 + PP * 16384) = c1; \
            } \
        }
#define M4(X, S) mf<S, 1>(c1, XFRAG(X, S), cinit); mf<S, 2>(c2, XFRAG(X, S), cinit); mf<S, 3>(c3, XFRAG(X, S), cinit); mf<S, 0>(c0, XFRAG(X, S), cinit);
#define BODY(FIRST, CUR, NXT, PP, PO, PN) { \
        if (__builtin_expect((unsigned)rdy < (unsigned)thr, 0)) { \
            RECHECK(PP) \
            if (rdy < need) { const int c = count(PO); rdy = c > rdy ? c : rdy; }     \
            if (rdy < need) rdy = spin(sfl, need - 1); \
        } \
        sx = x1; \
        const __amdgpu_buffer_rsrc_t rx = __builtin_amdgcn_make_buffer_rsrc(xall, 0, xsz, 0x00020000); \
        f32x4 q0, q1, q2, q3; unsigned xszn; int sp1, i2; \
        STEP4(FIRST, CUR, NXT, PP, PO, PN, 0) STEP4(FIRST, CUR, NXT, PP, PO, PN, 1) STEP4(FIRST, CUR, NXT, PP, PO, PN, 2) STEP4(FIRST, CUR, NXT, PP, PO, PN, 3) \
        STEP4(FIRST, CUR, NXT, PP, PO, PN, 4) STEP4(FIRST, CUR, NXT, PP, PO, PN, 5) STEP4(FIRST, CUR, NXT, PP, PO, PN, 6) STEP4(FIRST, CUR, NXT, PP, PO, PN, 7) \
        xsz = xszn; spm = sp0; sp0 = sp1; }

    int i = 0, spm = NBLK;
    u32x2 P0 = {0u, 0u}, P1;
    if (rg == 7) spin(sflo, NBLK - 1);
    BODY(true, X1, X0, 1, P0, P1)
    if (ownpos != 1u && count(pf) < NBLK) spin(sflo, NBLK - 1);
    do {
        BODY(false, X0, X1, 0, P1, P0) BODY(false, X1, X0, 1, P0, P1)
        if (__builtin_expect(i == 3 && q == 0 && rg < 7, 0))
            asm volatile("s_mov_b64 exec, 1\n\tglobal_store_dword %0, %1, %2 sc0 sc1\n\ts_mov_b64 exec, -1" :: "v"(0u), "v"(call), "s"(me + WS_RTR + peer * 128 + cg * 4) : "memory");
    } while (i < n);
    RECHECK(0)
    oso = xm << 1; osz = (unsigned)(NR * MSH) * NSH * 4u;
    {
        asm volatile("s_nop 7\n\ts_nop 7" : "+v"(c0), "+v"(c2), "+v"(c3));
        *(LAS f32x4*)(lw2 + 0 * 16384) = c2; *(LAS f32x4*)(lw3 + 0 * 16384) = c3; *(LAS f32x4*)(lw0 + 0 * 16384) = c0;
        wait_lgkm0(); __builtin_amdgcn_s_barrier(); asm volatile("" ::: "memory");
        store_rows(fmul4(fadd4(fadd4(*(const LAS f32x4*)(lr + 0 * 16384), *(const LAS f32x4*)(lr + 0 * 16384 + 4096)), fadd4(*(const LAS f32x4*)(lr + 0 * 16384 + 8192), *(const LAS f32x4*)(lr + 0 * 16384 + 12288))), scale));
        asm volatile(".Lfk_code_end:");
    }
#undef BODY
#undef M4
#undef RECHECK
#undef STEP4
#undef SLOT
#undef XFRAG
#undef SB
#undef XLOAD
#undef POLL
#undef PIN
}
}

extern "C" void kernel_launch(int rank, int nranks, ncclComm_t comm,
                              hipStream_t stream,
                              void* const* d_in, const int* in_sizes, int n_in,
                              void* d_out, int out_size,
                              const int* in_dtypes, int out_dtype,
                              void* const* d_out_all,
                              void* const* d_ws_all, size_t ws_bytes) {
    (void)comm; (void)in_dtypes; (void)out_dtype; (void)d_out_all;
    if (nranks != NR || (unsigned)rank >= (unsigned)NR || n_in != 4 || ws_bytes < WS_END ||
        in_sizes[0] != MSH * KDIM || in_sizes[1] != KDIM * NFULL || out_size != NR * MSH * NSH) {
        fprintf(stderr, "kernel_launch: unexpected problem shape\n");
        return;
    }
    Params p;
    p.x = (const unsigned char*)d_in[0]; p.w = (const unsigned char*)d_in[1];
    p.sx = (const float*)d_in[2]; p.sw = (const float*)d_in[3]; p.out = (float*)d_out;
    for (int i = 0; i < NR; ++i) p.wsr[i] = (char*)d_ws_all[(rank + 1 + i) & 7];
    p.rank = rank; p.slow = (int)((3u - (unsigned)((size_t)p.w >> 7)) & 7u) * 2;
    fused_kernel<<<256, 256, 0, stream>>>(p);
    HIP_CHECK(hipGetLastError());
}
```

```cpp
#include <hip/hip_runtime.h>
#include <rccl/rccl.h>
#include <cstdio>

#define HIP_CHECK(expr) do { hipError_t _e = (expr); if (_e != hipSuccess) { \
    fprintf(stderr, "HIP error %s at %s:%d\n", hipGetErrorString(_e), __FILE__, __LINE__); \
    return; } } while (0)

namespace {
typedef unsigned u32x2 __attribute__((ext_vector_type(2)));
typedef unsigned u32x4 __attribute__((ext_vector_type(4)));
typedef unsigned u32x8 __attribute__((ext_vector_type(8)));
typedef float f32x4 __attribute__((ext_vector_type(4)));
#define LAS __attribute__((address_space(3)))
#define DI __device__ __forceinline__

constexpr int NR = 8;
constexpr int KDIM = 4096, NFULL = 16384, NSH = 2048, MSH = 512;
constexpr unsigned SHARD = MSH * KDIM;
constexpr int NBLK = MSH / 16;
constexpr int NUNIT = 512;
constexpr int WPP = 16;
constexpr size_t WS_X = 0;
constexpr size_t WS_FLAG = 2ull * NR * SHARD;
constexpr size_t WS_RTR = WS_FLAG + NR * 2048;
constexpr size_t WS_TICK = WS_RTR + NR * 128;
constexpr size_t WS_TMO = WS_TICK + 1024;
constexpr size_t WS_CALL = WS_TMO + 1024;
constexpr size_t WS_CNT = WS_CALL + 1024 * 4;
constexpr size_t WS_END = WS_CNT + 7 * 128 * 64;
constexpr unsigned long long DEADLINE_TICKS = 180000000ull;

constexpr unsigned OWNMASK = 0x155u;
constexpr int POLL_AHEAD = 3;
constexpr int NSPEC = 8;
constexpr unsigned NAGG = 26 * 16;
constexpr int OUT_AUX = 2;

struct Params {
    const unsigned char* x; const unsigned char* w; const float* sx; const float* sw; float* out;
    char* wsr[NR];
    int rank; int slow;
};

#define ACLOB "a0","a1","a2","a3","a4","a5","a6","a7","a8","a9","a10","a11","a12","a13","a14","a15","a16","a17","a18","a19","a20","a21","a22","a23","a24","a25","a26","a27","a28","a29","a30","a31","a32","a33","a34","a35","a36","a37","a38","a39","a40","a41","a42","a43","a44","a45","a46","a47","a48","a49","a50","a51","a52","a53","a54","a55","a56","a57","a58","a59","a60","a61","a62","a63","a64","a65","a66","a67","a68","a69","a70","a71","a72","a73","a74","a75","a76","a77","a78","a79","a80","a81","a82","a83","a84","a85","a86","a87","a88","a89","a90","a91","a92","a93","a94","a95","a96","a97","a98","a99","a100","a101","a102","a103","a104","a105","a106","a107","a108","a109","a110","a111","a112","a113","a114","a115","a116","a117","a118","a119","a120","a121","a122","a123","a124","a125","a126","a127","a128","a129","a130","a131","a132","a133","a134","a135","a136","a137","a138","a139","a140","a141","a142","a143","a144","a145","a146","a147","a148","a149","a150","a151","a152","a153","a154","a155","a156","a157","a158","a159","a160","a161","a162","a163","a164","a165","a166","a167","a168","a169","a170","a171","a172","a173","a174","a175","a176","a177","a178","a179","a180","a181","a182","a183","a184","a185","a186","a187","a188","a189","a190","a191","a192","a193","a194","a195","a196","a197","a198","a199","a200","a201","a202","a203","a204","a205","a206","a207","a208","a209","a210","a211","a212","a213","a214","a215","a216","a217","a218","a219","a220","a221","a222","a223","a224","a225","a226","a227","a228","a229","a230","a231","a232","a233","a234","a235","a236","a237","a238","a239","a240","a241","a242","a243","a244","a245","a246","a247","a248","a249","a250","a251","a252","a253","a254","a255"

template <int N> DI void wait_vm() { asm volatile("s_waitcnt vmcnt(%0)" :: "n"(N) : "memory"); }
DI void wait_lgkm0() { asm volatile("s_waitcnt lgkmcnt(0)" ::: "memory"); }
template <int B> DI void snd_load(const unsigned char* base, unsigned voff) {
    asm volatile("global_load_dwordx4 a[%c2:%c2+3], %0, %1\n\t"
                 "global_load_dwordx4 a[%c2+4:%c2+7], %0, %1 offset:64\n\t"
                 "global_load_dwordx4 a[%c2+8:%c2+11], %0, %1 offset:128\n\t"
                 "global_load_dwordx4 a[%c2+12:%c2+15], %0, %1 offset:192" :: "v"(voff), "s"(base), "i"(B) : "memory");
}
template <int B> DI void snd_store(char* base, unsigned voff) {
    asm volatile("global_store_dwordx4 %0, a[%c2:%c2+3], %1 sc0 sc1\n\t"
                 "global_store_dwordx4 %0, a[%c2+4:%c2+7], %1 offset:1024 sc0 sc1\n\t"
                 "global_store_dwordx4 %0, a[%c2+8:%c2+11], %1 offset:2048 sc0 sc1\n\t"
                 "global_store_dwordx4 %0, a[%c2+12:%c2+15], %1 offset:3072 sc0 sc1\n\ts_nop 1" :: "v"(voff), "s"(base), "i"(B) : "memory");
}
template <int A> DI void snd_ticket(char* ctr, unsigned vzero) {
    unsigned long long keep;
    asm volatile("s_mov_b64 %0, exec\n\ts_mov_b64 exec, 1\n\tglobal_atomic_add a[%c3], %1, a33, %2 sc0\n\ts_mov_b64 exec, %0"
                 : "=&s"(keep) : "v"(vzero), "s"(ctr), "i"(A) : "memory");
}
DI void code_prefetch(unsigned lane) {
    unsigned long long pc; unsigned t0, t1, t2;
    asm volatile("s_getpc_b64 %0\n"
                 ".Lfk_code_begin:\n\t"
                 "v_min_u32 %1, .Lfk_code_end-.Lfk_code_begin, %4\n\tv_min_u32 %2, .Lfk_code_end-.Lfk_code_begin, %5\n\tv_min_u32 %3, .Lfk_code_end-.Lfk_code_begin, %6\n\t"
                 "global_load_dword a252, %1, %0\n\tglobal_load_dword a253, %2, %0\n\tglobal_load_dword a254, %3, %0"
                 : "=&s"(pc), "=&v"(t0), "=&v"(t1), "=&v"(t2) : "v"(lane * 128u), "v"(lane * 128u + 8192u), "v"(lane * 128u + 16384u) : "memory", "a252", "a253", "a254");
}
template <int B, int H> DI void snd_half(char* base, unsigned voff) {
    asm volatile("global_store_dwordx4 %0, a[%c2:%c2+3], %1 offset:%c3 sc0 sc1\n\t"
                 "global_store_dwordx4 %0, a[%c2+4:%c2+7], %1 offset:%c3+1024 sc0 sc1\n\ts_nop 1" :: "v"(voff), "s"(base), "i"(B + 8 * H), "i"(2048 * H) : "memory");
}
template <int A> DI void call_load(const char* ctr, unsigned vzero) { asm volatile("global_load_dword a[%c2], %0, %1 sc0 sc1" :: "v"(vzero), "s"(ctr), "i"(A) : "memory"); }
template <int A> DI void snd_flag(char* flag, unsigned vzero) {
    unsigned long long keep;
    asm volatile("s_mov_b64 %0, exec\n\ts_mov_b64 exec, 1\n\tglobal_store_dword %1, a[%c3], %2 sc0 sc1\n\ts_mov_b64 exec, %0"
                 : "=&s"(keep) : "v"(vzero), "s"(flag), "i"(A) : "memory");
}
template <int A> DI void snd_flag4(char* flag, unsigned vzero) {
    unsigned long long keep;
    asm volatile("s_mov_b64 %0, exec\n\ts_mov_b64 exec, 1\n\tglobal_store_dwordx4 %1, a[%c3:%c3+3], %2 sc0 sc1\n\ts_mov_b64 exec, %0"
                 : "=&s"(keep) : "v"(vzero), "s"(flag), "i"(A) : "memory");
}
template <int A> DI void acc_set(unsigned v) { asm volatile("s_nop 1\n\tv_accvgpr_write_b32 a[%c1], %0\n\ts_nop 1" :: "s"(v), "i"(A)); }
DI unsigned unit_src(unsigned u) { return (u >> 4) * 65536u + (u & 15u) * 256u; }
DI unsigned unit_flag(unsigned u) { return (((u >> 2) & 3u) * 128u + (u >> 4) * 4u + (u & 3u)) * 4u; }
template <int N, int A = 32> DI unsigned snd_wait_ticket() {
    unsigned v;
    asm volatile("s_waitcnt vmcnt(%1)\n\tv_accvgpr_read_b32 %0, a[%c2]\n\ts_nop 1" : "=v"(v) : "n"(N), "i"(A) : "memory");
    return __builtin_amdgcn_readfirstlane(v);
}


template <int S, int SLOT> DI void w_read(unsigned a0, unsigned a1, unsigned a2, unsigned a3) {
    constexpr int B = S * 32, O = SLOT * 8192;
#define TRR(c, t, A) "ds_read_b64_tr_b8 a[%c4+" #c "*8+" #t "*2:%c4+" #c "*8+" #t "*2+1], " A " offset:%c5+4096*(" #t "/2)+512*(" #t "%%2)\n\t"
    asm volatile(TRR(0, 0, "%0") TRR(0, 1, "%0") TRR(0, 2, "%0") TRR(0, 3, "%0")
                 TRR(1, 0, "%1") TRR(1, 1, "%1") TRR(1, 2, "%1") TRR(1, 3, "%1")
                 TRR(2, 0, "%2") TRR(2, 1, "%2") TRR(2, 2, "%2") TRR(2, 3, "%2")
                 TRR(3, 0, "%3") TRR(3, 1, "%3") TRR(3, 2, "%3") TRR(3, 3, "%3")
                 "s_waitcnt lgkmcnt(0)"
                 :: "v"(a0), "v"(a1), "v"(a2), "v"(a3), "i"(B), "i"(O) : "memory", ACLOB);
#undef TRR
}
DI void w_stage(__amdgpu_buffer_rsrc_t rs, unsigned ve, unsigned vo, unsigned m0base, unsigned soff) {
    unsigned keep, so;
#define WST(V) "s_nop 0\n\tbuffer_load_dwordx4 " V ", %5, %1 offen lds\n\t"
#define WNX "s_add_u32 m0, m0, 0x400\n\ts_add_u32 %1, %1, 0x40000\n\t"
    asm volatile("s_mov_b32 %0, m0\n\ts_mov_b32 m0, %2\n\ts_mov_b32 %1, %6\n\t"
                 WST("%3") WNX WST("%4") WNX WST("%3") WNX WST("%4") WNX WST("%3") WNX WST("%4") WNX WST("%3") WNX WST("%4")
                 "s_mov_b32 m0, %0"
                 : "=&s"(keep), "=&s"(so) : "s"(m0base), "v"(ve), "v"(vo), "s"(rs), "s"(soff) : "scc", "memory");
#undef WST
#undef WNX
    static_assert(16 * NFULL == 0x40000, "");
}
DI void x_stage(const unsigned char* base, unsigned vdst, unsigned m0base) {
    unsigned keep, v;
#define XST(r) "v_xor_b32 %1, " #r "*16, %3\n\tv_add_u32 %1, " #r "*4096, %1\n\tglobal_load_lds_dwordx4 %1, %4\n\ts_add_u32 m0, m0, 0x400\n\t"
    asm volatile("s_mov_b32 %0, m0\n\ts_mov_b32 m0, %2\n\t"
                 XST(0) XST(1) XST(2) XST(3) XST(4) XST(5) XST(6) XST(7) XST(8) XST(9) XST(10) XST(11) XST(12) XST(13) XST(14) XST(15)
                 "s_mov_b32 m0, %0"
                 : "=&s"(keep), "=&v"(v) : "s"(m0base), "v"(vdst), "s"(base) : "scc", "memory");
#undef XST
}
template <int S, int C> DI void mf(f32x4& c, const u32x8& x, const f32x4& cinit) {
    constexpr int B = S * 32 + C * 8;
    if constexpr (S == 0) asm volatile("v_mfma_f32_16x16x128_f8f6f4 %0, a[%c3:%c3+7], %1, %2" : "+v"(c) : "v"(x), "v"(cinit), "i"(B));
    else asm volatile("v_mfma_f32_16x16x128_f8f6f4 %0, a[%c2:%c2+7], %1, %0" : "+v"(c) : "v"(x), "i"(B));
}
DI float fadd(float a, float b) { float r; asm("v_add_f32_e32 %0, %1, %2" : "=v"(r) : "v"(a), "v"(b)); return r; }
DI float fmul(float a, float b) { float r; asm("v_mul_f32_e32 %0, %1, %2" : "=v"(r) : "v"(a), "v"(b)); return r; }
DI f32x4 fadd4(f32x4 a, f32x4 b) { return (f32x4){fadd(a[0], b[0]), fadd(a[1], b[1]), fadd(a[2], b[2]), fadd(a[3], b[3])}; }
DI f32x4 fmul4(f32x4 a, float b) { return (f32x4){fmul(a[0], b), fmul(a[1], b), fmul(a[2], b), fmul(a[3], b)}; }

constexpr int S_K1 = 4, S_BAR = 6, S_RD = 8, S_K2 = 10, S_A1 = 12, S_A2 = 14, S_POLL = 16, S_A3 = 18, S_MU = 20, S_ST = 22, S_K3 = 24, S_K4 = 26, S_K5 = 28, S_K6 = 30;

struct XBlk { u32x4 h[16]; };

__global__ __launch_bounds__(256, 1) void fused_kernel(Params p) {
    __shared__ __attribute__((aligned(16))) unsigned char lds[4 * 32768 + 32768];
    const int lane = threadIdx.x & 63;
    const int wid = __builtin_amdgcn_readfirstlane(threadIdx.x >> 6);
    const int rank = p.rank;
    char* const me = p.wsr[7];
    const int bid = blockIdx.x;
    code_prefetch(lane);
    const bool sender = bid < 24 || (bid < 32 && (bid & 2));
    const int w = (bid < 24 ? bid : 24 + (bid & 1) + ((bid >> 1) & 2)) * 4 + wid, pi = w % 7;
    char* const pws = p.wsr[pi];
    asm volatile("" :: "s"(p.x), "s"(p.w), "s"(me), "s"(rank), "s"(pws));

    char* const callp = me + WS_CALL + (bid * 4 + wid) * 4;

    const unsigned vsrc = (lane & 15) * KDIM + (lane >> 4) * 16, vdst = lane * 16;

    if (__builtin_expect(sender, 1)) {
        const int j = w / 7;
        unsigned vzero = 0;
        asm volatile("v_accvgpr_write_b32 a33, 1" : "+v"(vzero) :: "a32", "a33", "a35", "a36", "a37", "a38", "a43", "a111");
        auto srcp = [&](unsigned u) { return p.x + unit_src(u < NUNIT ? u : NUNIT - 1); };
        unsigned u0 = j, u1 = j + WPP, u2 = j + 2 * WPP, prev, pp = NUNIT;
        char* const ctr = me + WS_TICK + pi * 128;
        asm volatile("s_load_dword s96, %0, 0x0\n\ts_load_dword s97, %1, 0x0\n\ts_load_dword s98, %2, 0x0"
                     :: "s"(pws + WS_X + (size_t)rank * SHARD), "s"(pws + WS_X + (size_t)(NR + rank) * SHARD), "s"(pws + WS_FLAG + rank * 2048) : "memory", "s96", "s97", "s98");
        call_load<35>(callp, vzero);
        snd_load<64>(srcp(u0), vsrc);
        snd_load<80>(srcp(u1), vsrc);
        const unsigned call = snd_wait_ticket<4, 35>() + 1u;
        char* const dst = pws + WS_X + (size_t)((call & 1u) * NR + rank) * SHARD;
        snd_half<64, 0>(dst + u0 * 4096u, vdst);
        snd_ticket<32>(ctr, vzero);
        snd_load<96>(srcp(u2), vsrc);
        snd_half<64, 1>(dst + u0 * 4096u, vdst);
        acc_set<40>(call); acc_set<41>(call); acc_set<42>(call); acc_set<43>(call);
        const unsigned tbase = (call - 1u) * NUNIT - 3u * WPP;
        char* const fl = pws + WS_FLAG + rank * 2048;
        char* const own = me + WS_X + (size_t)((call & 1u) * NR + rank) * SHARD;
        prev = u0; u0 = u1; u1 = u2;
        wait_vm<7>();
        wait_lgkm0();
        int fill = 0;
        char* const cnt = me + WS_CNT + pi * (128 * 64);
        auto filler = [&]() { asm volatile("global_load_dword a38, %0, %1" :: "v"(vzero), "s"(p.x) : "memory"); };
        auto publish = [&](unsigned u) {
            if (fill == 4 || u >= NUNIT) filler(); else if (u < NAGG) snd_ticket<37>(cnt + (u >> 2) * 64, vzero); else snd_flag<40>(fl + unit_flag(u), vzero);
        };
        auto publish4 = [&](unsigned u) {
            unsigned c; asm volatile("v_accvgpr_read_b32 %0, a37\n\ts_nop 1" : "=v"(c));
            if (u < NAGG && fill != 4 && (__builtin_amdgcn_readfirstlane(c) & 3) == 3) snd_flag4<40>(fl + unit_flag(u & ~3u), vzero); else filler();
        };
        const char* const rtr = pws + WS_RTR + rank * 128;
#define SEND_STEP(BS, BL, NA) { \
            if (u0 >= NUNIT - 16 * NSPEC - 64 && fill == 0) { asm volatile("global_load_dword a36, %0, %1 sc0 sc1" :: "v"((lane & 31) * 4), "s"(rtr) : "memory"); fill = 1; } \
            if (u0 >= NUNIT - 16 * NSPEC && (fill & 1)) { \
                  \
                wait_vm<0>(); publish4(pp); publish(prev); wait_vm<0>(); publish4(prev); pp = prev = NUNIT; \
                const unsigned long long t0 = __builtin_amdgcn_s_memrealtime(); \
                for (fill = 2;;) { \
                    unsigned f, tmo; \
                    asm volatile("global_load_dword %0, %2, %3 sc0 sc1\n\tglobal_load_dword %1, %4, %5 sc1\n\ts_waitcnt vmcnt(0)" \
                                 : "=&v"(f), "=&v"(tmo) : "v"((lane & 31) * 4), "s"(rtr), "v"(vzero), "s"(me + WS_TMO) : "memory"); \
                    if (__all(f >= call)) break; \
                    if (__builtin_amdgcn_readfirstlane(tmo) == call || __builtin_amdgcn_s_memrealtime() - t0 > DEADLINE_TICKS) { snd_flag<40>(me + WS_TMO, vzero); fill = 4; break; } \
                    __builtin_amdgcn_s_sleep(4); \
                } \
            } \
            char* const d = (fill == 4 ? own : dst) + u0 * 4096u; \
            snd_half<BS, 0>(d, vdst); \
            u2 = snd_wait_ticket<NA>() - tbase; \
            snd_ticket<32>(ctr, vzero); \
            snd_load<BL>(srcp(u2), vsrc); \
            wait_vm<7>();                                                \
            publish4(pp); publish(prev); \
            snd_half<BS, 1>(d, vdst); \
            pp = prev; prev = u0; u0 = u1; u1 = u2; \
            wait_vm<9>();                                                \
            if (fill == 1) { unsigned f; asm volatile("v_accvgpr_read_b32 %0, a36" : "=v"(f)); fill = __all(f >= call) ? 2 : 3; } }
        SEND_STEP(80, 64, 8)
        for (;;) {
            SEND_STEP(96, 80, 10) if (u0 >= NUNIT) break;
            SEND_STEP(64, 96, 10) if (u0 >= NUNIT) break;
            SEND_STEP(80, 64, 10) if (u0 >= NUNIT) break;
        }
#undef SEND_STEP
        wait_vm<0>();
        publish4(pp); publish(prev);
        if (prev < NAGG) { wait_vm<0>(); publish4(prev); }
        snd_flag<40>(callp, vzero);
        return;
    }

    int cg, rg;
    {
        const int xcd = bid & 7, i = (xcd & 1) * 28 + (bid >> 3) - 4;
        rg = bid < 32 ? 7 : i >> 3;
        cg = bid < 32 ? p.slow + (xcd & 1) + (xcd >> 2) * 16 : (xcd >> 1) * 8 + (i & 7);
    }
    const int q = wid;
    const float scale = p.sx[0] * p.sw[0];
    unsigned call;
    const unsigned ldsw = (unsigned)(size_t)(LAS unsigned char*)lds + q * 32768u;
    const unsigned peer = (unsigned)(rank + 1 + rg) & 7u;
    const unsigned sfl = peer * 2048 + q * 512, sflo = rank * 2048 + q * 512, vfl = lane * 8;
    static_assert(__builtin_popcount(OWNMASK) == 5 && (OWNMASK & 3) == 1, "");
    const bool slowcg = (cg & 14) == p.slow;
    const int n = rg == 7 ? NBLK - 7 : slowcg ? NBLK + 1 : NBLK + 5, ostep = rg == 7 ? 1 : 7;
    const unsigned ownpos = rg == 7 ? ~0u : slowcg ? 1u : OWNMASK;
    unsigned tabx; int tabr;
    {
        const int before = __builtin_amdgcn_mbcnt_lo(ownpos, 0u);
        const bool isown = lane < 32 && ((ownpos >> (lane & 31)) & 1u);
        const int b = isown ? rg + ostep * before : lane - before;
        tabx = (((isown ? rank : peer) * NBLK + (b & 31)) << 16) | (b < NBLK ? 0u : 1u);
        tabr = isown || b >= NBLK ? -1 : b;
    }
    const unsigned orow = q * 4 + (lane >> 4);
    unsigned voffO = orow * (NSH * 4) + (cg * 64 + (lane & 15) * 4) * 4;
    const unsigned lwb = 131072 + q * 4096 + (lane & 15) * 256 + (((lane >> 4) ^ (lane & 15)) << 4);
    LAS unsigned char* lw0 = (LAS unsigned char*)lds + lwb; LAS unsigned char* lw1 = (LAS unsigned char*)lds + (lwb ^ 64);
    LAS unsigned char* lw2 = (LAS unsigned char*)lds + (lwb ^ 128); LAS unsigned char* lw3 = (LAS unsigned char*)lds + (lwb ^ 192);
    LAS unsigned char* lr = (LAS unsigned char*)lds + 131072 + orow * 256 + (((lane & 15) ^ orow) << 4);
    f32x4 cinit = {0.f, 0.f, 0.f, 0.f};
    unsigned voffR[8];
#pragma unroll
    for (int k = 0; k < 8; ++k) voffR[k] = vdst + q * 16384 + (unsigned)((k + rg) & 7) * 2048u;
    f32x4 c0 = cinit, c1 = cinit, c2 = cinit, c3 = cinit;
    XBlk X0, X1;
    u32x2 pf;
    {
        const __amdgpu_buffer_rsrc_t rw = __builtin_amdgcn_make_buffer_rsrc((void*)p.w, 0, (unsigned)KDIM * NFULL, 0x00020000);
        const unsigned ve = (lane >> 2) * NFULL + (((lane & 3) ^ ((lane >> 4) & 1)) * 16), vo = ve ^ 32u;
        const unsigned so = (unsigned)(q * 1024) * NFULL + rank * NSH + cg * 64;
        const unsigned ab = ldsw + ((lane >> 4) * 16 + ((lane & 15) >> 1)) * 64 + (lane & 1) * 8, f = (lane >> 3) & 3;
        const unsigned a0 = ab + ((0 ^ f) << 4), a1 = ab + ((1 ^ f) << 4), a2 = ab + ((2 ^ f) << 4), a3 = ab + ((3 ^ f) << 4);
        constexpr unsigned SS = 128u * NFULL;
        unsigned vzero = 0;
        asm volatile("" : "+v"(vzero));
        call_load<32>(callp, vzero);
#define WOFF(S) (so + (unsigned)(((S) + rg) & 7) * SS)
#define WFIRST w_stage(rw, ve, vo, ldsw + 0 * 8192, WOFF(0)); w_stage(rw, ve, vo, ldsw + 1 * 8192, WOFF(1)); \
               w_stage(rw, ve, vo, ldsw + 2 * 8192, WOFF(2)); w_stage(rw, ve, vo, ldsw + 3 * 8192, WOFF(3));
        const bool copier = rg >= 3 && rg < 7;
        const unsigned uo = (unsigned)(((rg - 3) * 32 + cg) * 4 + q);
        if (copier) snd_load<224>(p.x + unit_src(uo), vsrc);
        if (rg < 7) {
            const unsigned l0 = (unsigned)(((bid >> 3) - 4) * 4 + q) * 64u + lane, cb = rank * NSH + (cg & 24) * 64;
            const unsigned l2 = l0 + 2 * 7168u;
            asm volatile("buffer_load_dword a249, %0, %3, %4 offen\n\tbuffer_load_dword a250, %1, %3, %4 offen\n\tbuffer_load_dword a251, %2, %3, %4 offen"
                         :: "v"((l0 >> 2) * NFULL + (l0 & 3) * 128u), "v"(((l0 + 7168u) >> 2) * NFULL + (l0 & 3) * 128u),
                            "v"(((l2 < 16384u ? l2 : l0) >> 2) * NFULL + (l0 & 3) * 128u), "s"(rw), "s"(cb) : "memory", "a249", "a250", "a251");
        }
        if (copier) {
            call = snd_wait_ticket<3>() + 1u;
            acc_set<240>(call);
            snd_store<224>(me + WS_X + (size_t)((call & 1u) * NR + rank) * SHARD + uo * 4096u, vdst);
            WFIRST
            wait_vm<32>();
            snd_flag<240>(me + WS_FLAG + rank * 2048 + unit_flag(uo), vzero);
            wait_vm<24>();
        } else {
            WFIRST
            call = snd_wait_ticket<24>() + 1u;
        }
#undef WFIRST
        w_read<0, 0>(a0, a1, a2, a3); w_stage(rw, ve, vo, ldsw + 0 * 8192, WOFF(4));
        asm volatile("" : "+v"(tabx), "+v"(tabr), "+v"(voffO), "+v"(lw0), "+v"(lw1), "+v"(lw2), "+v"(lw3), "+v"(lr), "+v"(voffR[0]), "+v"(voffR[1]), "+v"(voffR[2]), "+v"(voffR[3]),
                          "+v"(voffR[4]), "+v"(voffR[5]), "+v"(voffR[6]), "+v"(voffR[7]));
        asm volatile("s_nop 1" : "+v"(cinit), "+v"(c0), "+v"(c1), "+v"(c2), "+v"(c3));
        wait_vm<24>(); w_read<1, 1>(a0, a1, a2, a3); w_stage(rw, ve, vo, ldsw + 1 * 8192, WOFF(5));
        unsigned xr[16];
        {
            const unsigned xb = q * 32768 + (lane & 15) * 1024 + (((lane >> 4) ^ (lane & 3)) << 4), xs = (lane & 12) << 4;
#pragma unroll
            for (int k = 0; k < 16; ++k) { xr[k] = xb + (((unsigned)((k + 2 * rg) & 15) << 6) ^ xs); asm volatile("" : "+v"(xr[k])); }
        }
        wait_vm<24>(); w_read<2, 2>(a0, a1, a2, a3); w_stage(rw, ve, vo, ldsw + 2 * 8192, WOFF(6));
        wait_vm<24>(); w_read<3, 3>(a0, a1, a2, a3); w_stage(rw, ve, vo, ldsw + 3 * 8192, WOFF(7));
#undef WOFF
        wait_vm<24>(); w_read<4, 0>(a0, a1, a2, a3);
        wait_vm<16>(); w_read<5, 1>(a0, a1, a2, a3);
        pf = __builtin_bit_cast(u32x2, __builtin_amdgcn_raw_buffer_load_b64(__builtin_amdgcn_make_buffer_rsrc(me + WS_FLAG, 0, NR * 2048, 0x00020000), vfl, sfl, 17));
        x_stage(p.x + rg * 65536 + q * 1024, vdst, ldsw);
        wait_vm<8 + 17>(); w_read<6, 2>(a0, a1, a2, a3);
        wait_vm<17>();     w_read<7, 3>(a0, a1, a2, a3);
        wait_vm<0>();
        asm volatile("s_mov_b64 exec, 1\n\tglobal_store_dword %0, %1, %2 sc0 sc1\n\ts_mov_b64 exec, -1" :: "v"(vzero), "v"(call), "s"(callp) : "memory");
#pragma unroll
        for (int k = 0; k < 16; ++k) X1.h[k] = *(const LAS u32x4*)((LAS unsigned char*)lds + xr[k]);
    }
    void* const xall = me + WS_X + (size_t)(call & 1u) * NR * SHARD;
    unsigned xlim = NR * SHARD;
    unsigned xsz = NR * SHARD;

    auto count = [&](u32x2 f) { unsigned r; asm("s_ff1_i32_b64 %0, %1" : "=s"(r) : "s"(__ballot(f.x < call || f.y < call))); return (int)(r >> 1); };
    auto spin = [&](unsigned soff, int need) {
        const char* const pf = me + WS_FLAG + soff;
        const unsigned long long t0 = __builtin_amdgcn_s_memrealtime();
        asm volatile("" :: "s"(t0));
        for (;;) {
            u32x2 f; unsigned tmo;
            asm volatile("global_load_dwordx2 %0, %2, %3 sc0 sc1\n\tglobal_load_dword %1, %4, %5 sc1\n\ts_waitcnt vmcnt(0)"
                         : "=&v"(f), "=&v"(tmo) : "v"(vfl), "s"(pf), "v"(0u), "s"(me + WS_TMO) : "memory");
            const int c = count(f);
            if (c > need) return c;
            if (__builtin_amdgcn_readfirstlane(tmo) == call || __builtin_amdgcn_s_memrealtime() - t0 > DEADLINE_TICKS) {
                if (lane == 0) __hip_atomic_store((unsigned*)(me + WS_TMO), call, __ATOMIC_RELAXED, __HIP_MEMORY_SCOPE_AGENT);
                xlim = 0; xsz = 0; const float qn = __builtin_nanf(""); cinit = (f32x4){qn, qn, qn, qn};
                asm volatile("s_nop 1" : "+v"(cinit));
                return 2 * NBLK;
            }
            __builtin_amdgcn_s_sleep(1);
        }
    };
#define XLOAD(X, k) (X).h[k] = __builtin_bit_cast(u32x4, __builtin_amdgcn_raw_buffer_load_b128(rx, voffR[(k) >> 1], sx + ((k) & 1) * 1024u, 17))
#define POLL(size, off) __builtin_bit_cast(u32x2, __builtin_amdgcn_raw_buffer_load_b64(__builtin_amdgcn_make_buffer_rsrc(me + WS_FLAG, 0, size, 0x00020000), vfl, off, 17))
    int rdy = rg == 7 ? 2 * NBLK : count(pf);
    unsigned xm = 1u, x0 = __builtin_amdgcn_readlane(tabx, 0), x1 = __builtin_amdgcn_readlane(tabx, 1);
    int r1 = __builtin_amdgcn_readlane(tabr, 1), need = r1 + 1, thr = need;
    int sp0 = NBLK;
    unsigned sx, oso = 0, osz = 0, psz = NR * 2048;
    auto store_rows = [&](const f32x4& r) {
        __builtin_amdgcn_raw_buffer_store_b128(__builtin_bit_cast(u32x4, r), __builtin_amdgcn_make_buffer_rsrc((void*)p.out, 0, osz, 0x00020000), voffO, oso, OUT_AUX);
    };
    static_assert(NSPEC % 2 == 0 && NSPEC <= 14, "");
    char* const pz = (char*)xall + (rg < 7 ? (size_t)peer * SHARD + (NBLK - NSPEC) * 65536 + cg * 512 : (size_t)rank * SHARD) + q * 16384;
    const unsigned pzs = rg < 7 ? 131072u : 0u, vpz = rg < 7 ? (lane >> 5) * 65536u + (lane & 31) * 16u : vdst;
    auto fill_piece = [&](int k) {
        const u32x4 ff = {~0u, ~0u, ~0u, ~0u};
        asm volatile("global_store_dwordx4 %0, %1, %2 sc0 sc1\n\ts_nop 1" :: "v"(vpz), "v"(ff), "s"(pz + k * pzs) : "memory");
    };
#define SB __builtin_amdgcn_sched_barrier(0);
#define XFRAG(X, S) __builtin_shufflevector((X).h[2 * (S)], (X).h[2 * (S) + 1], 0, 1, 2, 3, 4, 5, 6, 7)
#define PIN(v) asm volatile("" : "+s"(v));
#define SLOT(FIRST, CUR, NXT, PP, PO, PN, J, C) \
        mf<(J) / 4, C>(c##C, XFRAG(CUR, (J) / 4), cinit); SB \
        if constexpr ((J) == 0) *(LAS f32x4*)(lw2 + PP * 16384) = c2; \
        if constexpr ((J) == 1) *(LAS f32x4*)(lw3 + PP * 16384) = c3; \
        if constexpr ((J) == 2) *(LAS f32x4*)(lw0 + PP * 16384) = c0; \
        if constexpr ((J) == 31) *(LAS f32x4*)(lw1 + (1 - PP) * 16384) = c1; \
        if constexpr ((J) == S_POLL) PN = POLL(psz, sfl); \
        if constexpr (FIRST && (J) == S_POLL - 2) pf = POLL(NR * 2048, sflo); \
        if constexpr (FIRST && ((J) & 3) == 2 && (J) / 4 < NSPEC / 2) fill_piece((J) / 4); \
        if constexpr ((J) & 1) XLOAD(NXT, (J) >> 1); \
        if constexpr ((J) == S_BAR && PP == 1 && !(FIRST)) wait_vm<20>();     \
        if constexpr ((J) == S_BAR) { wait_lgkm0(); __builtin_amdgcn_s_barrier(); asm volatile("" ::: "memory"); } \
        if constexpr ((J) == S_RD) { q0 = *(const LAS f32x4*)(lr + PP * 16384); q1 = *(const LAS f32x4*)(lr + PP * 16384 + 4096); \
                                     q2 = *(const LAS f32x4*)(lr + PP * 16384 + 8192); q3 = *(const LAS f32x4*)(lr + PP * 16384 + 12288); } \
        if constexpr ((J) == S_A1) q1 = fadd4(q0, q1); \
        if constexpr ((J) == S_A2) q2 = fadd4(q2, q3); \
        if constexpr ((J) == S_A3) q1 = fadd4(q1, q2); \
        if constexpr ((J) == S_MU) q1 = fmul4(q1, scale); \
        if constexpr ((J) == S_ST) store_rows(q1); \
        if constexpr ((J) == S_K1) { PIN(r1) sp1 = r1 >= rdy ? r1 : NBLK; PIN(sp1) PIN(i) i2 = i + 2; PIN(i2) } \
        if constexpr ((J) == S_K2) { PIN(xm) oso = xm << 1; osz = (xm & 1u) ? 0u : (unsigned)(NR * MSH) * NSH * 4u; PIN(oso) PIN(osz) } \
        if constexpr ((J) == S_K3) { PIN(i2) xm = x0; x0 = x1; x1 = __builtin_amdgcn_readlane(tabx, i2); r1 = __builtin_amdgcn_readlane(tabr, i2); PIN(x1) PIN(r1) } \
        if constexpr ((J) == S_K3 + 1) { PIN(x1) xszn = (x1 & 1u) ? 0u : xlim; PIN(xszn) } \
        if constexpr ((J) == S_K4) { PIN(r1) int r2 = r1 + 1; PIN(r2) need = r1 < NBLK - NSPEC ? r2 : r1; PIN(need) } \
        if constexpr ((J) == S_K4 + 1) { PIN(need) thr = sp0 < NBLK ? -1 : need; ++i; PIN(thr) PIN(i) } \
        if constexpr ((J) == S_K5) { asm volatile("" : "+v"(PO)); const int c = count(PO); rdy = c > rdy ? c : rdy; PIN(rdy) }     \
        if constexpr ((J) == S_K6) { PIN(need) psz = rdy < need + POLL_AHEAD ? NR * 2048 : 0u; PIN(psz) } \
        SB
#define STEP4(FIRST, CUR, NXT, PP, PO, PN, S) SLOT(FIRST, CUR, NXT, PP, PO, PN, 4 * S, 1) SLOT(FIRST, CUR, NXT, PP, PO, PN, 4 * S + 1, 2) SLOT(FIRST, CUR, NXT, PP, PO, PN, 4 * S + 2, 3) SLOT(FIRST, CUR, NXT, PP, PO, PN, 4 * S + 3, 0)
#define RECHECK(PP) \
        if (spm < NBLK) { \
            asm volatile("s_nop 7\n\ts_nop 7" : "+v"(c0)); \
            if (__any(c0[0] != c0[0])) { \
                rdy = spin(sfl, spm); \
                const __amdgpu_buffer_rsrc_t rx = __builtin_amdgcn_make_buffer_rsrc(xall, 0, xlim, 0x00020000); \
                sx = xm; \
                XBlk T; \
                _Pragma("unroll") for (int k = 0; k < 16; ++k) XLOAD(T, k); \
                M4(T, 0) M4(T, 1) M4(T, 2) M4(T, 3) M4(T, 4) M4(T, 5) M4(T, 6) M4(T, 7) \
                asm volatile("s_nop 7\n\ts_nop 7" : "+v"(c0), "+v"(c1), "+v"(c2), "+v"(c3)); \
                *(LAS f32x4*)(lw1 + PP * 16384) = c1; \
            } \
        }
#define M4(X, S) mf<S, 1>(c1, XFRAG(X, S), cinit); mf<S, 2>(c2, XFRAG(X, S), cinit); mf<S, 3>(c3, XFRAG(X, S), cinit); mf<S, 0>(c0, XFRAG(X, S), cinit);
#define BODY(FIRST, CUR, NXT, PP, PO, PN) { \
        if (__builtin_expect((unsigned)rdy < (unsigned)thr, 0)) { \
            RECHECK(PP) \
            if (rdy < need) { const int c = count(PO); rdy = c > rdy ? c : rdy; }     \
            if (rdy < need) rdy = spin(sfl, need - 1); \
        } \
        sx = x1; \
        const __amdgpu_buffer_rsrc_t rx = __builtin_amdgcn_make_buffer_rsrc(xall, 0, xsz, 0x00020000); \
        f32x4 q0, q1, q2, q3; unsigned xszn; int sp1, i2; \
        STEP4(FIRST, CUR, NXT, PP, PO, PN, 0) STEP4(FIRST, CUR, NXT, PP, PO, PN, 1) STEP4(FIRST, CUR, NXT, PP, PO, PN, 2) STEP4(FIRST, CUR, NXT, PP, PO, PN, 3) \
        STEP4(FIRST, CUR, NXT, PP, PO, PN, 4) STEP4(FIRST, CUR, NXT, PP, PO, PN, 5) STEP4(FIRST, CUR, NXT, PP, PO, PN, 6) STEP4(FIRST, CUR, NXT, PP, PO, PN, 7) \
        xsz = xszn; spm = sp0; sp0 = sp1; }

    int i = 0, spm = NBLK;
    u32x2 P0 = {0u, 0u}, P1;
    if (rg == 7) spin(sflo, NBLK - 1);
    BODY(true, X1, X0, 1, P0, P1)
    if (ownpos != 1u && count(pf) < NBLK) spin(sflo, NBLK - 1);
    do {
        BODY(false, X0, X1, 0, P1, P0) BODY(false, X1, X0, 1, P0, P1)
        if (__builtin_expect(i == 3 && q == 0 && rg < 7, 0))
            asm volatile("s_mov_b64 exec, 1\n\tglobal_store_dword %0, %1, %2 sc0 sc1\n\ts_mov_b64 exec, -1" :: "v"(0u), "v"(call), "s"(me + WS_RTR + peer * 128 + cg * 4) : "memory");
    } while (i < n);
    RECHECK(0)
    oso = xm << 1; osz = (unsigned)(NR * MSH) * NSH * 4u;
    {
        asm volatile("s_nop 7\n\ts_nop 7" : "+v"(c0), "+v"(c2), "+v"(c3));
        *(LAS f32x4*)(lw2 + 0 * 16384) = c2; *(LAS f32x4*)(lw3 + 0 * 16384) = c3; *(LAS f32x4*)(lw0 + 0 * 16384) = c0;
        wait_lgkm0(); __builtin_amdgcn_s_barrier(); asm volatile("" ::: "memory");
        store_rows(fmul4(fadd4(fadd4(*(const LAS f32x4*)(lr + 0 * 16384), *(const LAS f32x4*)(lr + 0 * 16384 + 4096)), fadd4(*(const LAS f32x4*)(lr + 0 * 16384 + 8192), *(const LAS f32x4*)(lr + 0 * 16384 + 12288))), scale));
        asm volatile(".Lfk_code_end:");
    }
#undef BODY
#undef M4
#undef RECHECK
#undef STEP4
#undef SLOT
#undef XFRAG
#undef SB
#undef XLOAD
#undef POLL
#undef PIN
}
}

extern "C" void kernel_launch(int rank, int nranks, ncclComm_t comm,
                              hipStream_t stream,
                              void* const* d_in, const int* in_sizes, int n_in,
                              void* d_out, int out_size,
                              const int* in_dtypes, int out_dtype,
                              void* const* d_out_all,
                              void* const* d_ws_all, size_t ws_bytes) {
    (void)comm; (void)in_dtypes; (void)out_dtype; (void)d_out_all;
    if (nranks != NR || (unsigned)rank >= (unsigned)NR || n_in != 4 || ws_bytes < WS_END ||
        in_sizes[0] != MSH * KDIM || in_sizes[1] != KDIM * NFULL || out_size != NR * MSH * NSH) {
        fprintf(stderr, "kernel_launch: unexpected problem shape\n");
        return;
    }
    Params p;
    p.x = (const unsigned char*)d_in[0]; p.w = (const unsigned char*)d_in[1];
    p.sx = (const float*)d_in[2]; p.sw = (const float*)d_in[3]; p.out = (float*)d_out;
    for (int i = 0; i < NR; ++i) p.wsr[i] = (char*)d_ws_all[(rank + 1 + i) & 7];
    p.rank = rank; p.slow = (int)((3u - (unsigned)((size_t)p.w >> 7)) & 7u) * 2;
    fused_kernel<<<256, 256, 0, stream>>>(p);
    HIP_CHECK(hipGetLastError());
}
```
